```python
import numpy as np

kernel_name = "gpu_gemm_rs_m4096_k4096_n32768_f32_relu"


ABSMAX_RTOL = 0.001

COMPUTE_DTYPE = "float32"
EPILOGUE = "relu"
IN_QUANT = None
OUT_QUANT = None
IN_DTYPES = {"x": "float32", "w_mat": "float32"}
WORLD_SIZE = 8

def setup_inputs():

    x = np.random.randn(4096, 4096).astype(np.float32)
    w_mat = (np.random.randn(4096, 32768) / np.sqrt(4096)).astype(np.float32)
    return {"x": x, "w_mat": w_mat}

def _epilogue(y):
    return np.maximum(y, 0.0).astype(np.float32)

def reference(x, w_mat):
    y = (x @ w_mat).astype(np.float32)
    return _epilogue(y)
```

```cpp
#include <hip/hip_runtime.h>
#include <rccl/rccl.h>
#include <cstdio>

#define HIP_CHECK(expr) do { hipError_t _e = (expr); if (_e != hipSuccess) { \
    fprintf(stderr, "HIP error %s at %s:%d\n", hipGetErrorString(_e), __FILE__, __LINE__); \
    return; } } while (0)

namespace {

typedef _Float16 half8 __attribute__((ext_vector_type(8)));
typedef _Float16 half2v __attribute__((ext_vector_type(2)));
typedef float f32x4 __attribute__((ext_vector_type(4)));
typedef unsigned u32x4 __attribute__((ext_vector_type(4)));
#define LAS __attribute__((address_space(3)))

constexpr int NR = 8;
constexpr int GM = 4096, GK = 4096, GN = 32768;
constexpr int KS = GK / NR;
constexpr int NB = GN / NR;
constexpr int MS = GM / NR;
constexpr int NWG = 256;
constexpr unsigned PIECE = 32768;
constexpr unsigned HTB = 16384;
constexpr unsigned PANEL = 64 * PIECE;
constexpr size_t WS_CALL = 0;
constexpr size_t WS_TMO = 16384;
constexpr size_t WS_FX = 32768;
constexpr size_t WS_FW = WS_FX + 4096;
constexpr size_t WS_FO = 65536;
constexpr size_t WS_STAGE = 1u << 20;
constexpr size_t STAGE_PAR = 32ull * PANEL;
constexpr size_t WS_NEED = WS_STAGE + 2 * STAGE_PAR;
constexpr unsigned long long DEADLINE = 50000000ull;
constexpr int AUX_SYS = 17;
constexpr unsigned LDS_BYTES = 8 * HTB + 256;

struct Params {
    const float* x;
    const float* w;
    float* out_all[NR];
    char* ws_all[NR];
    int rank;
};

__device__ __forceinline__ void piece_of(int g, int q, int& type, int& panel, int& j) {
    if (g < 3) {
        j = 2 * g + (q >> 5);
        const int idx = q & 31;
        type = (idx >> 3) & 1;
        panel = ((idx >> 4) << 3) | (idx & 7);
    } else {
        const int step = q >> 3, e = q & 7;
        if (step >= 1 && step <= 4) { type = 0; j = 6 + ((step - 1) >> 1); panel = ((step - 1) & 1) * 8 + e; }
        else { type = 1; const int c = step == 0 ? 0 : step - 4; panel = 4 * c + (e & 3); j = 6 + (e >> 2); }
    }
}

__device__ __forceinline__ unsigned pack2(float a, float b) {
    half2v h; h.x = (_Float16)a; h.y = (_Float16)b;
    return __builtin_bit_cast(unsigned, h);
}

__device__ __forceinline__ const void* uniform_ptr(const void* p) {
    const unsigned long long v = (unsigned long long)p;
    const unsigned lo = __builtin_amdgcn_readfirstlane((unsigned)v), hi = __builtin_amdgcn_readfirstlane((unsigned)(v >> 32));
    return (const void*)(((unsigned long long)hi << 32) | lo);
}

__global__ void __launch_bounds__(512, 2) fused_gemm_rs(const Params p) {
    __shared__ __attribute__((aligned(16))) unsigned char smem[LDS_BYTES];
    LAS unsigned char* lds = (LAS unsigned char*)smem;
    LAS unsigned* lcnt = (LAS unsigned*)(lds + 8 * HTB);

    const int tid = threadIdx.x, lane = tid & 63;
    const int wid = __builtin_amdgcn_readfirstlane(tid >> 6);
    const int wr = wid >> 2, wc = wid & 3, fr = lane & 15, fq = lane >> 4;
    const int b = blockIdx.x, xcd = b & 7, grp = b >> 6, v = (b >> 3) & 7;
    const int pm = 2 * v + (xcd & 1), pn = 4 * grp + (xcd >> 1);
    const int rank = p.rank;
    char* const ws = p.ws_all[rank];
    unsigned* const tmo = (unsigned*)(ws + WS_TMO);

    if (tid < 16) lcnt[tid] = 0u;
    if (tid == 64) {
        unsigned* cp = (unsigned*)(ws + WS_CALL + (size_t)b * 64);
        const unsigned c = __hip_atomic_load(cp, __ATOMIC_RELAXED, __HIP_MEMORY_SCOPE_AGENT) + 1u;
        __hip_atomic_store(cp, c, __ATOMIC_RELAXED, __HIP_MEMORY_SCOPE_AGENT);
        lcnt[16] = c;
    }
    __syncthreads();
    const unsigned call = __builtin_amdgcn_readfirstlane(lcnt[16]);
    const size_t stage_off = WS_STAGE + (size_t)(call & 1u) * STAGE_PAR;

    f32x4 acc[2][2][4][2];
#pragma unroll
    for (int a = 0; a < 2; ++a)
#pragma unroll
        for (int c = 0; c < 2; ++c)
#pragma unroll
            for (int m = 0; m < 4; ++m)
#pragma unroll
                for (int n = 0; n < 2; ++n) acc[a][c][m][n] = (f32x4){0.f, 0.f, 0.f, 0.f};

    const char* const gA = ws + stage_off + (size_t)pm * PANEL;
    const char* const gB = ws + stage_off + (size_t)(16 + pn) * PANEL;
    const unsigned* const fX = (const unsigned*)(ws + WS_FX) + pm * 64;
    const unsigned* const fW = (const unsigned*)(ws + WS_FW) + pn * 64;
    unsigned long long ready = 0ull;

    const int split = grp == 0 ? 0 : (2 * grp - 1) * 8;

#define WAIT_V(n) asm volatile("s_waitcnt vmcnt(" #n ")" ::: "memory")
#define WAIT_L(n) asm volatile("s_waitcnt lgkmcnt(" #n ")" ::: "memory")
#define BAR __builtin_amdgcn_s_barrier()
#define SCHED __builtin_amdgcn_sched_barrier(0)
#define SA(bb, h) (((bb) * 2 + (h)) * HTB)
#define SB(bb, h) ((4 + (bb) * 2 + (h)) * HTB)
#define STAGE(bufoff, gbase) do { _Pragma("unroll") for (int _i = 0; _i < 2; ++_i) \
        __builtin_amdgcn_global_load_lds((const unsigned*)((gbase) + voff + _i * 8192), (LAS unsigned*)(lds + (bufoff) + ldsw + _i * 8192), 16, 0, 0); } while (0)
#define LDA(dst, bb, h) do { _Pragma("unroll") for (int m = 0; m < 4; ++m) _Pragma("unroll") for (int k = 0; k < 2; ++k) \
        dst[m][k] = *(const LAS half8*)(lds + SA(bb, h) + aoff + m * 256 + k * 8192); } while (0)
#define LDB(dst, bb, h) do { _Pragma("unroll") for (int n = 0; n < 2; ++n) _Pragma("unroll") for (int k = 0; k < 2; ++k) \
        dst[n][k] = *(const LAS half8*)(lds + SB(bb, h) + boff + n * 256 + k * 8192); } while (0)
#define MMA(ai, bj, At, Bt) do { __builtin_amdgcn_s_setprio(1); \
        _Pragma("unroll") for (int m = 0; m < 4; ++m) _Pragma("unroll") for (int n = 0; n < 2; ++n) _Pragma("unroll") for (int k = 0; k < 2; ++k) \
        acc[ai][bj][m][n] = __builtin_amdgcn_mfma_f32_16x16x32_f16(Bt[n][k], At[m][k], acc[ai][bj][m][n], 0, 0, 0); \
        __builtin_amdgcn_s_setprio(0); } while (0)
#define POLL(need) do { if (wid == 0 && (ready & (need)) != (need)) { \
        const unsigned long long _t0 = __builtin_amdgcn_s_memrealtime(); \
        for (;;) { \
            const unsigned _fx = __hip_atomic_load(fX + lane, __ATOMIC_RELAXED, __HIP_MEMORY_SCOPE_SYSTEM); \
            const unsigned _fw = __hip_atomic_load(fW + lane, __ATOMIC_RELAXED, __HIP_MEMORY_SCOPE_SYSTEM); \
            ready = __ballot(_fx >= call && _fw >= call); \
            if ((ready & (need)) == (need)) break; \
            if (__builtin_amdgcn_readfirstlane(__hip_atomic_load(tmo, __ATOMIC_RELAXED, __HIP_MEMORY_SCOPE_AGENT)) == call \
                || __builtin_amdgcn_s_memrealtime() - _t0 > DEADLINE) { \
                if (lane == 0) __hip_atomic_store(tmo, call, __ATOMIC_RELAXED, __HIP_MEMORY_SCOPE_AGENT); \
                ready = ~0ull; break; } \
            __builtin_amdgcn_s_sleep(8); \
        } \
        asm volatile("s_waitcnt vmcnt(0)" ::: "memory"); } } while (0)

    for (int seg = 0; seg < 2; ++seg) {
        const int t0 = seg == 0 ? 0 : split, t1 = seg == 0 ? split : 64;
        if (seg == 1) {
            int tz = tid; asm volatile("" : "+v"(tz));
            const int ln = tz & 63;
            float f[32];
            int dest_of[3]; unsigned flag_of[3];
            auto job = [&](int i, int& type, const float*& src, char*& dstp, int& dest, unsigned& flagoff) {
                int panel, j; piece_of(grp, i * 8 + xcd, type, panel, j);
                dest = (v + i + rank) & 7;
                const int slot = j * 8 + ((rank - dest) & 7);
                dstp = p.ws_all[dest] + stage_off + (size_t)(type * 16 + panel) * PANEL + (size_t)slot * PIECE;
                flagoff = (unsigned)(type ? WS_FW : WS_FX) + (unsigned)(panel * 64 + slot) * 4u;
                src = type ? p.w + (size_t)(j * 64 + wid * 8) * GN + dest * NB + panel * 256
                           : p.x + (size_t)(panel * 256) * KS + j * 64 + wid * 8;
            };
            auto load = [&](int type, const float* src) {
                if (type) {
#pragma unroll
                    for (int r = 0; r < 8; ++r)
#pragma unroll
                        for (int c = 0; c < 4; ++c) f[c * 8 + r] = src[(size_t)r * GN + c * 64 + ln];
                } else {
#pragma unroll
                    for (int c = 0; c < 4; ++c) {
                        const f32x4 a = *(const f32x4*)(src + (size_t)(c * 64 + ln) * KS), bq = *(const f32x4*)(src + (size_t)(c * 64 + ln) * KS + 4);
                        f[c * 8 + 0] = a[0]; f[c * 8 + 1] = a[1]; f[c * 8 + 2] = a[2]; f[c * 8 + 3] = a[3];
                        f[c * 8 + 4] = bq[0]; f[c * 8 + 5] = bq[1]; f[c * 8 + 6] = bq[2]; f[c * 8 + 7] = bq[3];
                    }
                }
            };
            auto publish = [&](int i, int dest, unsigned flagoff) {
                if (ln == 0) {
                    const unsigned o = __hip_atomic_fetch_add(lcnt + i, 1u, __ATOMIC_RELAXED, __HIP_MEMORY_SCOPE_WORKGROUP);
                    if (o == 7u) __hip_atomic_store((unsigned*)(p.ws_all[dest] + flagoff), call, __ATOMIC_RELAXED, __HIP_MEMORY_SCOPE_SYSTEM);
                }
            };
            int type, dest; const float* src; char* dstp; unsigned flagoff;
            job(0, type, src, dstp, dest, flagoff);
            load(type, src);
            dest_of[0] = dest_of[1] = dest_of[2] = 0; flag_of[0] = flag_of[1] = flag_of[2] = 0;
            for (int i = 0; i < 8; ++i) {
                WAIT_V(4);
                u32x4 pk[4];
#pragma unroll
                for (int c = 0; c < 4; ++c)
                    pk[c] = (u32x4){pack2(f[c * 8], f[c * 8 + 1]), pack2(f[c * 8 + 2], f[c * 8 + 3]), pack2(f[c * 8 + 4], f[c * 8 + 5]), pack2(f[c * 8 + 6], f[c * 8 + 7])};
                asm volatile("" : "+v"(pk[0]), "+v"(pk[1]), "+v"(pk[2]), "+v"(pk[3]));
                if (i >= 2) publish(i - 2, dest_of[1], flag_of[1]);
                dest_of[1] = dest_of[0]; flag_of[1] = flag_of[0];
                dest_of[0] = dest; flag_of[0] = flagoff;
                char* const cur = dstp;
                if (i + 1 < 8) { job(i + 1, type, src, dstp, dest, flagoff); load(type, src); }
                asm volatile("" ::: "memory");
                const __amdgpu_buffer_rsrc_t rs = __builtin_amdgcn_make_buffer_rsrc((void*)uniform_ptr(cur), 0, PIECE, 0x00020000);
#pragma unroll
                for (int c = 0; c < 4; ++c)
                    __builtin_amdgcn_raw_buffer_store_b128(pk[c], rs, (c >> 1) * HTB + wid * 2048 + ((c & 1) * 64 + ln) * 16, 0, AUX_SYS);
                asm volatile("" ::: "memory");
            }
            WAIT_V(4);
            publish(6, dest_of[1], flag_of[1]);
            WAIT_V(0);
            publish(7, dest_of[0], flag_of[0]);
        }
        if (t0 >= t1) continue;
        const unsigned voff = (unsigned)tid * 16u;
        const unsigned ldsw = (unsigned)wid * 1024u;
        const int aoff = fq * 2048 + (wr * 64 + fr) * 16, boff = fq * 2048 + (wc * 32 + fr) * 16;
        half8 At[4][2], B0[2][2], B1[2][2];
        const char* const cA = gA + (size_t)t0 * PIECE;
        const char* const cB = gB + (size_t)t0 * PIECE;
        POLL(3ull << t0);
        BAR;
        STAGE(SB(0, 0), cB); STAGE(SA(0, 0), cA); STAGE(SB(0, 1), cB + HTB); STAGE(SA(0, 1), cA + HTB);
        if (wr == 1) BAR;
        WAIT_V(4); BAR;
        STAGE(SB(1, 0), cB + PIECE); STAGE(SA(1, 0), cA + PIECE); STAGE(SB(1, 1), cB + PIECE + HTB);
        WAIT_V(6); BAR;
        for (int t = t0; t < t1; t += 2) {
            const bool last = (t == t1 - 2);
            const char* a1 = gA + (size_t)(t + 1) * PIECE;
            const char* a2 = last ? cA : gA + (size_t)(t + 2) * PIECE;
            const char* b2 = last ? cB : gB + (size_t)(t + 2) * PIECE;
            const char* a3 = a2 + PIECE; const char* b3 = b2 + PIECE;
            if (!last) POLL(3ull << (t + 2));
            LDB(B0, 0, 0); SCHED; LDA(At, 0, 0); STAGE(SA(1, 1), a1 + HTB);
            WAIT_L(8); BAR; WAIT_L(0); MMA(0, 0, At, B0); BAR; SCHED;
            LDB(B1, 0, 1); STAGE(SB(0, 0), b2);
            BAR; WAIT_L(0); MMA(0, 1, At, B1); BAR;
            LDA(At, 0, 1); STAGE(SA(0, 0), a2);
            BAR; WAIT_L(0); MMA(1, 0, At, B0); BAR; SCHED;
            STAGE(SB(0, 1), b2 + HTB);
            WAIT_V(6); BAR; MMA(1, 1, At, B1); BAR;
            LDB(B0, 1, 0); SCHED; LDA(At, 1, 0); STAGE(SA(0, 1), a2 + HTB);
            WAIT_L(8); BAR; WAIT_L(0); MMA(0, 0, At, B0); BAR; SCHED;
            LDB(B1, 1, 1); STAGE(SB(1, 0), b3);
            BAR; WAIT_L(0); MMA(0, 1, At, B1); BAR;
            LDA(At, 1, 1); STAGE(SA(1, 0), a3);
            BAR; WAIT_L(0); MMA(1, 0, At, B0); BAR; SCHED;
            STAGE(SB(1, 1), b3 + HTB);
            WAIT_V(6); BAR; MMA(1, 1, At, B1); BAR;
        }
        WAIT_V(0);
        if (wr == 0) BAR;
        BAR;
    }

    {
        int tz = tid; asm volatile("" : "+v"(tz));
        const int ln = tz & 63, efr = ln & 15, efq = ln >> 4;
        const bool bad = __builtin_amdgcn_readfirstlane(__hip_atomic_load(tmo, __ATOMIC_RELAXED, __HIP_MEMORY_SCOPE_AGENT)) == call;
        const int dest = pm >> 1;
        const __amdgpu_buffer_rsrc_t rs = __builtin_amdgcn_make_buffer_rsrc((void*)uniform_ptr(p.out_all[dest]), 0, (unsigned)MS * GN * 4u, 0x00020000);
        const unsigned base = (unsigned)(((pm & 1) * 256 + wr * 64 + efr) * GN + rank * NB + pn * 256 + wc * 32 + efq * 4) * 4u;
        const float qnan = __builtin_nanf("");
#pragma unroll
        for (int ai = 0; ai < 2; ++ai)
#pragma unroll
            for (int m = 0; m < 4; ++m)
#pragma unroll
                for (int bj = 0; bj < 2; ++bj)
#pragma unroll
                    for (int n = 0; n < 2; ++n) {
                        f32x4 o = acc[ai][bj][m][n];
#pragma unroll
                        for (int e = 0; e < 4; ++e) o[e] = bad ? qnan : (o[e] < 0.f ? 0.f : o[e]);
                        __builtin_amdgcn_raw_buffer_store_b128(__builtin_bit_cast(u32x4, o), rs,
                            base + (unsigned)((ai * 128 + m * 16) * GN + bj * 128 + n * 16) * 4u, 0, AUX_SYS);
                    }
        WAIT_V(0);
        __syncthreads();
        if (tz == 0) {
            if (dest != rank) {
                __builtin_amdgcn_fence(__ATOMIC_RELEASE, "");
                WAIT_V(0);
                __hip_atomic_store((unsigned*)(p.ws_all[dest] + WS_FO + (size_t)(rank * 32 + (pm & 1) * 16 + pn) * 128), call,
                                   __ATOMIC_RELAXED, __HIP_MEMORY_SCOPE_SYSTEM);
            }
            if ((b >> 5) != rank) {
                const unsigned* fo = (const unsigned*)(ws + WS_FO + (size_t)b * 128);
                const unsigned long long t0 = __builtin_amdgcn_s_memrealtime();
                while (__hip_atomic_load(fo, __ATOMIC_RELAXED, __HIP_MEMORY_SCOPE_SYSTEM) < call) {
                    if (__hip_atomic_load(tmo, __ATOMIC_RELAXED, __HIP_MEMORY_SCOPE_AGENT) == call
                        || __builtin_amdgcn_s_memrealtime() - t0 > DEADLINE) {
                        __hip_atomic_store(tmo, call, __ATOMIC_RELAXED, __HIP_MEMORY_SCOPE_AGENT);
                        __hip_atomic_store((unsigned*)(p.out_all[rank] + rank * NB + b), 0x7fc00000u, __ATOMIC_RELAXED, __HIP_MEMORY_SCOPE_SYSTEM);
                        break;
                    }
                    __builtin_amdgcn_s_sleep(4);
                }
                __builtin_amdgcn_fence(__ATOMIC_ACQUIRE, "");
            }
        }
    }
}

}

extern "C" void kernel_launch(int rank, int nranks, ncclComm_t comm,
                              hipStream_t stream,
                              void* const* d_in, const int* in_sizes, int n_in,
                              void* d_out, int out_size,
                              const int* in_dtypes, int out_dtype,
                              void* const* d_out_all,
                              void* const* d_ws_all, size_t ws_bytes) {
    if (nranks != NR || n_in < 2 || (unsigned)rank >= (unsigned)NR || ws_bytes < WS_NEED ||
        in_sizes[0] != GM * KS || in_sizes[1] != KS * GN || out_size != MS * GN) {
        fprintf(stderr, "kernel_launch: unexpected problem shape\n");
        return;
    }
    Params p;
    p.x = (const float*)d_in[0];
    p.w = (const float*)d_in[1];
    for (int r = 0; r < NR; ++r) { p.out_all[r] = (float*)d_out_all[r]; p.ws_all[r] = (char*)d_ws_all[r]; }
    p.rank = rank;
    hipLaunchKernelGGL(fused_gemm_rs, dim3(NWG), dim3(512), 0, stream, p);
    HIP_CHECK(hipGetLastError());
}
```

```cpp
#include <hip/hip_runtime.h>
#include <rccl/rccl.h>
#include <cstdio>

#define HIP_CHECK(expr) do { hipError_t _e = (expr); if (_e != hipSuccess) { \
    fprintf(stderr, "HIP error %s at %s:%d\n", hipGetErrorString(_e), __FILE__, __LINE__); \
    return; } } while (0)

namespace {

typedef _Float16 half8 __attribute__((ext_vector_type(8)));
typedef _Float16 half2v __attribute__((ext_vector_type(2)));
typedef float f32x4 __attribute__((ext_vector_type(4)));
typedef unsigned u32x4 __attribute__((ext_vector_type(4)));
typedef unsigned long long u64;
#define LAS __attribute__((address_space(3)))

constexpr int NR = 8;
constexpr int GM = 4096, GK = 4096, GN = 32768;
constexpr int KS = GK / NR;
constexpr int NB = GN / NR;
constexpr int MS = GM / NR;
constexpr int NWG = 256;
constexpr unsigned PIECE = 32768;
constexpr unsigned HTB = 16384;
constexpr unsigned PANEL = 64 * PIECE;
constexpr unsigned WS_GLOB = 0;
constexpr unsigned WS_KDONE = 256;
constexpr unsigned WS_Q = 1024;
constexpr unsigned WS_FX = 8192;
constexpr unsigned WS_FW = WS_FX + 4096;
constexpr unsigned WS_CTRL = 65536;
constexpr unsigned WS_STAMP = 131072;
constexpr unsigned WS_FO = 262144;
constexpr unsigned WS_CTL_BYTES = 1u << 20;
constexpr size_t WS_STAGE = 1u << 20;
constexpr size_t STAGE_PAR = 32ull * PANEL;
constexpr unsigned CHUNK = 17408;
constexpr size_t WS_OBOX = WS_STAGE + 2 * STAGE_PAR;
constexpr size_t WS_NEED = WS_OBOX + 256ull * 16 * CHUNK;
constexpr u64 DEADLINE = 50000000ull;
constexpr int AUX_SYS = 17;
constexpr unsigned LDS_SNAP = 8 * CHUNK + 4096;
constexpr unsigned LDS_MISC = LDS_SNAP + 2048;
constexpr unsigned LDS_DUMMY = LDS_MISC + 64;
constexpr unsigned LDS_BYTES = LDS_DUMMY + 256;
constexpr unsigned OOB = 0x40000000u;

#define QUOTA0 8
#define QUOTA1 8
#define QUOTA2 8
#define PASS_AHEAD 2
#define POLL_SLEEP 6
#define IN_BYTES PIECE
#define SRC_BYTES (8u * GN * 4u)
#define OUT_BYTES(x) (x)
#define AUX_OUT 0
#define PNMAP 0
#define OUT_PASS 6

struct Params {
    const float* x;
    const float* w;
    float* out_all[NR];
    char* ws_all[NR];
    unsigned order[64];
    int rank;
};

static unsigned piece_code(int q) {
    int type, panel, j;
    if (q < 192) {
        j = q >> 5;
        const int idx = q & 31;
        type = (idx >> 3) & 1;
        panel = ((idx >> 4) << 3) | (idx & 7);
    } else {
        const int step = (q - 192) >> 3, e = q & 7;
        if (step >= 1 && step <= 4) { type = 0; j = 6 + ((step - 1) >> 1); panel = ((step - 1) & 1) * 8 + e; }
        else { type = 1; const int c = step == 0 ? 0 : step - 4; panel = 4 * c + (e & 3); j = 6 + (e >> 2); }
    }
    return (unsigned)(type << 7 | panel << 3 | j);
}

__device__ __forceinline__ unsigned pack2(float a, float b) {
    half2v h; h.x = (_Float16)a; h.y = (_Float16)b;
    return __builtin_bit_cast(unsigned, h);
}
__device__ __forceinline__ __amdgpu_buffer_rsrc_t make_rsrc(const void* p, unsigned bytes) {
    const u64 v = (u64)p;
    const unsigned lo = __builtin_amdgcn_readfirstlane((unsigned)v), hi = __builtin_amdgcn_readfirstlane((unsigned)(v >> 32));
    return __builtin_amdgcn_make_buffer_rsrc((void*)(((u64)hi << 32) | lo), 0, bytes, 0x00020000);
}

__global__ void __launch_bounds__(512, 2) fused_gemm_rs(const Params p) {
    __shared__ __attribute__((aligned(16))) unsigned char smem[LDS_BYTES];
    LAS unsigned char* lds = (LAS unsigned char*)smem;
    LAS unsigned* misc = (LAS unsigned*)(lds + LDS_MISC);

    const int tid = threadIdx.x, lane = tid & 63;
    const int wid = __builtin_amdgcn_readfirstlane(tid >> 6);
    const int wr = wid >> 2, wc = wid & 3, fr = lane & 15, fq = lane >> 4;
    const int b = blockIdx.x, xcd = b & 7, grp = b >> 6, v = (b >> 3) & 7;
    const int pm = 2 * v + (xcd & 1), pn = 4 * grp + (xcd >> 1);
    const int rank = p.rank;
    char* const ws = p.ws_all[rank];
    unsigned* const tmo = (unsigned*)(ws + WS_GLOB);
    u64* const queue = (u64*)(ws + WS_Q + v * 128);

#define STAMP(i) do { } while (0)
#define WAIT_V(n) asm volatile("s_waitcnt vmcnt(" #n ")" ::: "memory")
#define WAIT_L(n) asm volatile("s_waitcnt lgkmcnt(" #n ")" ::: "memory")
#define BAR __builtin_amdgcn_s_barrier()
#define SCHED __builtin_amdgcn_sched_barrier(0)
#define CFENCE asm volatile("" ::: "memory")

    if (tid == 0) {
        unsigned* cp = (unsigned*)(ws + WS_CTRL + b * 256) + 32;
        const unsigned c = __hip_atomic_load(cp, __ATOMIC_RELAXED, __HIP_MEMORY_SCOPE_AGENT) + 1u;
        __hip_atomic_store(cp, c, __ATOMIC_RELAXED, __HIP_MEMORY_SCOPE_AGENT);
        __hip_atomic_fetch_max(queue, (u64)c << 32, __ATOMIC_RELAXED, __HIP_MEMORY_SCOPE_AGENT);
        misc[0] = c;
        misc[5] = 0u;
    }
    __syncthreads();
    const unsigned call = __builtin_amdgcn_readfirstlane(misc[0]);
    const size_t stage_off = WS_STAGE + (size_t)(call & 1u) * STAGE_PAR;
    STAMP(0);

    f32x4 acc[2][2][4][2];
#pragma unroll
    for (int a = 0; a < 2; ++a)
#pragma unroll
        for (int c = 0; c < 2; ++c)
#pragma unroll
            for (int m = 0; m < 4; ++m)
#pragma unroll
                for (int n = 0; n < 2; ++n) acc[a][c][m][n] = (f32x4){0.f, 0.f, 0.f, 0.f};

    const __amdgpu_buffer_rsrc_t rA = make_rsrc(ws + stage_off + (size_t)pm * PANEL, PANEL);
    const __amdgpu_buffer_rsrc_t rB = make_rsrc(ws + stage_off + (size_t)(16 + pn) * PANEL, PANEL);
    const __amdgpu_buffer_rsrc_t rC = make_rsrc(ws, WS_CTL_BYTES);
    const unsigned snap_src = wid == 0 ? WS_FX + pm * 256 : wid == 1 ? WS_FW + pn * 256 : wid == 2 ? WS_CTRL + b * 256 : wid == 3 ? WS_GLOB : OOB;

#define SA(bb, h) (((bb) * 2 + (h)) * HTB)
#define SB(bb, h) ((4 + (bb) * 2 + (h)) * HTB)
#define STAGE(bufoff, rs, so) do { _Pragma("unroll") for (int _i = 0; _i < 2; ++_i) \
        __builtin_amdgcn_raw_ptr_buffer_load_lds(rs, (LAS void*)(lds + (bufoff) + ldsw + _i * 8192), 16, voff, (so) + _i * 8192, 0, 0); } while (0)
#define SNAP() __builtin_amdgcn_raw_ptr_buffer_load_lds(rC, (LAS void*)(lds + LDS_SNAP + wid * 256), 4, lane * 4, snap_src, 0, AUX_SYS)
#define LDA(dst, bb, h) do { _Pragma("unroll") for (int m = 0; m < 4; ++m) _Pragma("unroll") for (int k = 0; k < 2; ++k) \
        dst[m][k] = *(const LAS half8*)(lds + SA(bb, h) + (k ? aoff1 : aoff0) + m * 2048); } while (0)
#define LDB(dst, bb, h) do { _Pragma("unroll") for (int n = 0; n < 2; ++n) _Pragma("unroll") for (int k = 0; k < 2; ++k) \
        dst[n][k] = *(const LAS half8*)(lds + SA(bb, h) + boff + n * 1024 + k * 8192); } while (0)
#define MMA(ai, bj, At, Bt) do { __builtin_amdgcn_s_setprio(1); \
        _Pragma("unroll") for (int m = 0; m < 4; ++m) _Pragma("unroll") for (int n = 0; n < 2; ++n) _Pragma("unroll") for (int k = 0; k < 2; ++k) \
        acc[ai][bj][m][n] = __builtin_amdgcn_mfma_f32_16x16x32_f16(Bt[n][k], At[m][k], acc[ai][bj][m][n], 0, 0, 0); \
        __builtin_amdgcn_s_setprio(0); } while (0)
#define SNAP_ADDR(tt) (LDS_SNAP + (lane < 4 ? (lane >> 1) * 256 + (((tt) + (lane & 1)) & 63) * 4 : lane == 4 ? 512 : 768))

    int t = 0;
    int state = grp == 0 ? 1 : 0;
    do {
        if (state != 1) {
            const u64 t0 = __builtin_amdgcn_s_memrealtime();
            unsigned bits;
            for (;;) {
                SNAP(); WAIT_V(0); BAR; CFENCE;
                const unsigned sv = *(const LAS unsigned*)(lds + SNAP_ADDR(t));
                bits = (unsigned)__ballot(lane == 5 ? sv == call : sv >= call) & (state == 0 ? 63u : 47u);
                if (bits >= 16u || (t < 64 && bits == 15u)) break;
                if (tid == 0 && __builtin_amdgcn_s_memrealtime() - t0 > DEADLINE)
                    __hip_atomic_store(tmo, call, __ATOMIC_RELAXED, __HIP_MEMORY_SCOPE_AGENT);
                __builtin_amdgcn_s_sleep(POLL_SLEEP);
                CFENCE; BAR;
            }
            if (bits & 16u) state = 1;
            if (bits & 32u) { state = 3; t = 64; }
        }
        const bool run = state != 1 && t < 64;
        if (state == 1) {
            STAMP(3);
            state = 2;
            int tz = tid; asm volatile("" : "+v"(tz));
            const int ln = tz & 63;
            const int quota = grp == 0 ? QUOTA0 : grp == 1 ? QUOTA1 : grp == 2 ? QUOTA2 : 1000;
            char* const dws = p.ws_all[v];
            const int srel = (rank - v) & 7;
            const unsigned vlW = ln * 16, vlX = (ln >> 3) * 2048 + (ln & 7) * 32;
            const unsigned vsW = wid * 2048 + ((ln & 48) + (ln & 3) * 4 + ((ln >> 2) & 3)) * 16;
            const unsigned vsX = wid * 4096 + (ln >> 3) * 128 + ((ln & 7) ^ (ln >> 4)) * 16;
#define DRAW(got, cnt, on) do { u64 _sv; const unsigned _m = __builtin_amdgcn_readfirstlane((wid == 0 && (on)) ? 1u : 0u); got = (u64)(cnt); \
                asm volatile("s_cmp_eq_u32 %3, 0\n\ts_cbranch_scc1 .Lskip%=\n\ts_mov_b64 %1, exec\n\ts_mov_b64 exec, 1\n\t" \
                             "global_atomic_add_x2 %0, %2, %0, off sc0\n\ts_mov_b64 exec, %1\n.Lskip%=:" \
                             : "+v"(got), "=&s"(_sv) : "v"(queue), "s"(_m) : "memory", "scc"); } while (0)
            int id0 = xcd, id1 = 8 + xcd;
            if (grp != 0) {
                u64 got; DRAW(got, 2, true);
                asm volatile("s_waitcnt vmcnt(0)" : "+v"(got) :: "memory");
                BAR;
                if (tz == 0) misc[1] = 16u + (unsigned)got;
                WAIT_L(0); BAR; CFENCE;
                id0 = __builtin_amdgcn_readfirstlane(misc[1]); id1 = id0 + 1;
            }
            f32x4 f[8];
            u32x4 pk[4];
            auto code_of = [&](int id) { return (int)((p.order[(id >> 2) & 63] >> ((id & 3) * 8)) & 255u); };
            auto load = [&](int code, bool on) {
                const int type = code >> 7, panel = (code >> 3) & 15, j = code & 7;
                const float* src = type ? p.w + (size_t)(j * 64 + wid * 8) * GN + v * NB + panel * 256
                                        : p.x + (size_t)(panel * 256 + wid * 32) * KS + j * 64;
                const __amdgpu_buffer_rsrc_t rs = make_rsrc(src, on ? SRC_BYTES : 0u);
                const unsigned vo = type ? vlW : vlX;
                const unsigned s1 = type ? GN * 4 : 16, s2 = type ? 2 * GN * 4 : 16384;
#pragma unroll
                for (int i = 0; i < 8; ++i)
                    f[i] = __builtin_bit_cast(f32x4, __builtin_amdgcn_raw_buffer_load_b128(rs, vo, (i >> 1) * s2 + (i & 1) * s1, 0));
            };
            auto convert = [&](int code) {
                if (code >> 7) {
#pragma unroll
                    for (int e = 0; e < 4; ++e)
                        pk[e] = (u32x4){pack2(f[0][e], f[1][e]), pack2(f[2][e], f[3][e]), pack2(f[4][e], f[5][e]), pack2(f[6][e], f[7][e])};
                } else {
#pragma unroll
                    for (int c = 0; c < 4; ++c)
                        pk[c] = (u32x4){pack2(f[2 * c][0], f[2 * c][1]), pack2(f[2 * c][2], f[2 * c][3]), pack2(f[2 * c + 1][0], f[2 * c + 1][1]), pack2(f[2 * c + 1][2], f[2 * c + 1][3])};
                }
            };
            auto store = [&](int code) {
                const int type = code >> 7;
                const __amdgpu_buffer_rsrc_t rs = make_rsrc(dws + stage_off + (size_t)(code >> 3) * PANEL + (size_t)((code & 7) * 8 + srel) * PIECE, IN_BYTES);
                const unsigned vo0 = type ? vsW : vsX, vo1 = (type ? vsW : vsX ^ 64u) + 1024u;
                const unsigned s2 = type ? HTB : 2048;
#pragma unroll
                for (int c = 0; c < 4; ++c)
                    __builtin_amdgcn_raw_buffer_store_b128(pk[c], rs, ((c & 1) ? vo1 : vo0) + (c >> 1) * s2, 0, AUX_SYS);
            };
            auto publish = [&](int code) {
                __hip_atomic_store((unsigned*)(dws + WS_FX) + (code >> 3) * 64 + (code & 7) * 8 + srel, call, __ATOMIC_RELAXED, __HIP_MEMORY_SCOPE_SYSTEM);
            };
            unsigned* const next_tok = (unsigned*)(ws + WS_CTRL + ((b + 64) & 255) * 256) + (grp == 3 ? 1 : 0);
            bool passed = false;
            if (id0 < 256) {
                int c0 = code_of(id0), cp = -1;
                load(c0, true);
                convert(c0);
                for (int n = 0;; ++n) {
                    const bool more = id1 < 256;
                    const bool fetch = more && n + 2 < quota;
                    const bool pass = !passed && (!more || n + PASS_AHEAD >= quota || (grp == 3 && id1 >= 256 - 8));
                    const int c1 = code_of(id1);
                    u64 got; DRAW(got, 1, fetch);
                    if (pass && tz == 0) __hip_atomic_store(next_tok, call, __ATOMIC_RELAXED, __HIP_MEMORY_SCOPE_AGENT);
                    passed = passed || pass;
                    CFENCE;
                    load(c1, more);
                    CFENCE;
                    store(c0);
                    __builtin_amdgcn_s_waitcnt(0x0F74);
                    asm volatile("s_waitcnt vmcnt(4)" : "+v"(got) :: "memory");
                    if (tz == 0) misc[2 + (n & 1)] = fetch ? 16u + (unsigned)got : 256u;
                    WAIT_L(0); BAR; CFENCE;
                    if (cp >= 0 && tz == 64) publish(cp);
                    if (!more) break;
                    convert(c1);
                    cp = c0; c0 = c1; id1 = __builtin_amdgcn_readfirstlane(misc[2 + (n & 1)]);
                }
                __builtin_amdgcn_s_waitcnt(0x0F70);
                WAIT_V(0); BAR; CFENCE;
                if (tz == 64) publish(c0);
            }
            if (!passed && tz == 0) __hip_atomic_store(next_tok, call, __ATOMIC_RELAXED, __HIP_MEMORY_SCOPE_AGENT);
            STAMP(4);
        }
        if (run) {
            const unsigned voff = (unsigned)tid * 16u;
            const unsigned ldsw = (unsigned)wid * 1024u;
            const int aoff0 = (wr * 64 + fr) * 128 + (fq ^ (fr >> 1)) * 16, aoff1 = aoff0 ^ 64;
            int boff = SB(0, 0) + fq * 2048 + (wc * 16 + fr) * 16;
            asm volatile("" : "+v"(boff));
            half8 At[4][2], B0[2][2], B1[2][2];
            unsigned so = (unsigned)t * PIECE;
            STAGE(SB(0, 0), rB, so); STAGE(SA(0, 0), rA, so); STAGE(SB(0, 1), rB, so + HTB); STAGE(SA(0, 1), rA, so + HTB);
            if (wr == 1) BAR;
            WAIT_V(4); BAR;
            STAGE(SB(1, 0), rB, so + PIECE); STAGE(SA(1, 0), rA, so + PIECE); STAGE(SB(1, 1), rB, so + PIECE + HTB);
            WAIT_V(6); BAR;
            bool go;
            do {
                const unsigned sv = *(const LAS unsigned*)(lds + SNAP_ADDR(t + 2));
                SCHED;
                LDB(B0, 0, 0); SCHED; LDA(At, 0, 0); STAGE(SA(1, 1), rA, so + PIECE + HTB);
                WAIT_L(8); BAR; WAIT_L(0); MMA(0, 0, At, B0);
                const unsigned bits = (unsigned)__ballot(sv >= call);
                go = t + 2 < 64 && (bits & (state == 0 ? 31u : 15u)) == 15u;
                const unsigned s2 = go ? so + 2 * PIECE : OOB;
                BAR; SCHED;
                LDB(B1, 0, 1); STAGE(SB(0, 0), rB, s2);
                BAR; WAIT_L(0); MMA(0, 1, At, B1); BAR;
                LDA(At, 0, 1); STAGE(SA(0, 0), rA, s2);
                BAR; WAIT_L(0); MMA(1, 0, At, B0); BAR; SCHED;
                STAGE(SB(0, 1), rB, s2 + HTB);
                if (!go) WAIT_V(0);
                WAIT_V(6); BAR; MMA(1, 1, At, B1); BAR;
                SNAP();
                LDB(B0, 1, 0); SCHED; LDA(At, 1, 0); STAGE(SA(0, 1), rA, s2 + HTB);
                WAIT_L(8); BAR; WAIT_L(0); MMA(0, 0, At, B0); BAR; SCHED;
                LDB(B1, 1, 1); STAGE(SB(1, 0), rB, s2 + PIECE);
                BAR; WAIT_L(0); MMA(0, 1, At, B1); BAR;
                LDA(At, 1, 1); STAGE(SA(1, 0), rA, s2 + PIECE);
                BAR; WAIT_L(0); MMA(1, 0, At, B0); BAR; SCHED;
                STAGE(SB(1, 1), rB, s2 + PIECE + HTB);
                WAIT_V(6); BAR; MMA(1, 1, At, B1); BAR;
                t += 2; so += 2 * PIECE;
            } while (go);
            WAIT_V(0);
            if (wr == 0) BAR;
            BAR;
        }
    } while (t < 64 || state < 2);
    STAMP(6);

    {
        int tz = tid; asm volatile("" : "+v"(tz));
        const int ln = tz & 63, efr = ln & 15, efq = ln >> 4;
        const float qnan = __builtin_nanf("");
        const __amdgpu_buffer_rsrc_t rO = make_rsrc(p.out_all[rank], (unsigned)MS * GN * 4u);
        const unsigned reg0 = (unsigned)wid * CHUNK;
        if (v == rank) {
            const bool bad = __builtin_amdgcn_readfirstlane(__hip_atomic_load(tmo, __ATOMIC_RELAXED, __HIP_MEMORY_SCOPE_AGENT)) == call;
            const unsigned base = (unsigned)(((pm & 1) * 256 + wr * 64 + efr) * GN + rank * NB + pn * 256 + wc * 64 + efq * 4) * 4u;
#pragma unroll
            for (int ai = 0; ai < 2; ++ai)
#pragma unroll
                for (int m = 0; m < 4; ++m)
#pragma unroll
                    for (int r = 0; r < 4; ++r) {
                        f32x4 o = (f32x4){acc[ai][0][m][0][r], acc[ai][0][m][1][r], acc[ai][1][m][0][r], acc[ai][1][m][1][r]};
#pragma unroll
                        for (int e = 0; e < 4; ++e) o[e] = bad ? qnan : (o[e] < 0.f ? 0.f : o[e]);
                        __builtin_amdgcn_raw_buffer_store_b128(__builtin_bit_cast(u32x4, o), rO, base + r * 64 + (unsigned)((ai * 128 + m * 16) * GN) * 4u, 0, AUX_OUT);
                    }
        } else {
            char* const dws = p.ws_all[v];
            const unsigned slot = (unsigned)((rank * 32 + grp * 8 + xcd) * 16 + wid * 2);
            const unsigned dummy = LDS_DUMMY + ln * 4;
            unsigned cnt[2];
#pragma unroll
            for (int ai = 0; ai < 2; ++ai) {
                unsigned mk[2] = {0u, 0u};
                unsigned pos = reg0 + 512u;
#pragma unroll
                for (int m = 0; m < 4; ++m)
#pragma unroll
                    for (int r = 0; r < 4; ++r)
#pragma unroll
                        for (int e = 0; e < 4; ++e) {
                            const float x = acc[ai][e >> 1][m][e & 1][r];
                            const bool keep = !(x <= 0.f);
                            const u64 bal = __builtin_amdgcn_ballot_w64(keep);
                            const unsigned rk = __builtin_amdgcn_mbcnt_hi((unsigned)(bal >> 32), __builtin_amdgcn_mbcnt_lo((unsigned)bal, 0u));
                            *(LAS float*)(lds + (keep ? pos + rk * 4u : dummy)) = x;
                            mk[m >> 1] = mk[m >> 1] * 2u + (keep ? 1u : 0u);
                            pos += (unsigned)__builtin_popcountll(bal) * 4u;
                        }
                *(LAS u64*)(lds + reg0 + ln * 8) = ((u64)mk[1] << 32) | mk[0];
                const unsigned bytes = pos - reg0;
                cnt[ai] = bytes;
                if (ai == 0) {
                    const unsigned* tok = (const unsigned*)(ws + WS_CTRL + b * 256) + 1;
                    const u64 t0 = __builtin_amdgcn_s_memrealtime();
                    while (state != 3 && __builtin_amdgcn_readfirstlane(__hip_atomic_load(tok, __ATOMIC_RELAXED, __HIP_MEMORY_SCOPE_AGENT)) < call) {
                        if (__builtin_amdgcn_readfirstlane(__hip_atomic_load(tmo, __ATOMIC_RELAXED, __HIP_MEMORY_SCOPE_AGENT)) == call
                            || __builtin_amdgcn_s_memrealtime() - t0 > DEADLINE) {
                            if (ln == 0) __hip_atomic_store(tmo, call, __ATOMIC_RELAXED, __HIP_MEMORY_SCOPE_AGENT);
                            break;
                        }
                        __builtin_amdgcn_s_sleep(16);
                    }
                    STAMP(5);
                }
                const __amdgpu_buffer_rsrc_t rs = make_rsrc(dws + WS_OBOX + (size_t)(slot + ai) * CHUNK, OUT_BYTES((bytes + 63u) & ~63u));
                for (unsigned o = ln * 16; o - ln * 16 < bytes; o += 2048) {
                    const u32x4 d0 = *(const LAS u32x4*)(lds + reg0 + o), d1 = *(const LAS u32x4*)(lds + reg0 + o + 1024);
                    __builtin_amdgcn_raw_buffer_store_b128(d0, rs, o, 0, AUX_SYS);
                    __builtin_amdgcn_raw_buffer_store_b128(d1, rs, o + 1024, 0, AUX_SYS);
                }
                if (ai == 1) {
                    if (ln == 0 && grp < 3 && __hip_atomic_fetch_add(misc + 5, 1u, __ATOMIC_RELAXED, __HIP_MEMORY_SCOPE_WORKGROUP) == OUT_PASS - 1)
                        __hip_atomic_store((unsigned*)(ws + WS_CTRL + (b + 64) * 256) + 1, call, __ATOMIC_RELAXED, __HIP_MEMORY_SCOPE_AGENT);
                }
                CFENCE;
            }
            STAMP(7);
            WAIT_V(0);
            const bool bad = __builtin_amdgcn_readfirstlane(__hip_atomic_load(tmo, __ATOMIC_RELAXED, __HIP_MEMORY_SCOPE_AGENT)) == call;
            if (ln < 2) __hip_atomic_store((u64*)(dws + WS_FO + (slot + ln) * 64), ((u64)call << 32) | (bad ? 0xffffffffu : (ln ? cnt[1] : cnt[0])),
                                           __ATOMIC_RELAXED, __HIP_MEMORY_SCOPE_SYSTEM);
            STAMP(8);
            const int njobs = grp < 2 ? 3 : grp == 2 ? 2 : 0;
            for (int jb = 0; jb < njobs; ++jb) {
                const int g2 = jb < 2 ? grp : 3, ai = jb < 2 ? jb : grp;
                const unsigned sl = (unsigned)((v * 32 + g2 * 8 + xcd) * 16 + wid * 2 + ai);
                const u64* fo = (const u64*)(ws + WS_FO + sl * 64);
                const u64 t0 = __builtin_amdgcn_s_memrealtime();
                unsigned bytes;
                for (;;) {
                    const u64 fv = __hip_atomic_load(fo, __ATOMIC_RELAXED, __HIP_MEMORY_SCOPE_SYSTEM);
                    bytes = __builtin_amdgcn_readfirstlane((unsigned)fv);
                    if (__builtin_amdgcn_readfirstlane((unsigned)(fv >> 32)) >= call) break;
                    if (__builtin_amdgcn_readfirstlane(__hip_atomic_load(tmo, __ATOMIC_RELAXED, __HIP_MEMORY_SCOPE_AGENT)) == call
                        || __builtin_amdgcn_s_memrealtime() - t0 > DEADLINE) {
                        if (ln == 0) __hip_atomic_store(tmo, call, __ATOMIC_RELAXED, __HIP_MEMORY_SCOPE_AGENT);
                        bytes = 0xffffffffu;
                        break;
                    }
                    __builtin_amdgcn_s_sleep(8);
                }
                const bool nodata = bytes > 512u + 16384u;
                if (nodata) bytes = 0u;
                const __amdgpu_buffer_rsrc_t rs = make_rsrc(ws + WS_OBOX + (size_t)sl * CHUNK, bytes);
                for (unsigned o = 0; o < bytes; o += 1024)
                    __builtin_amdgcn_raw_ptr_buffer_load_lds(rs, (LAS void*)(lds + reg0 + o), 16, ln * 16, o, 0, 0);
                WAIT_V(0);
                const u64 mm = *(const LAS u64*)(lds + reg0 + ln * 8);
                unsigned mk[2] = {(unsigned)mm, (unsigned)(mm >> 32)};
                unsigned pos = reg0 + 512u;
                const unsigned base = (unsigned)(((xcd & 1) * 256 + ai * 128 + wr * 64 + efr) * GN + v * NB + (4 * g2 + (xcd >> 1)) * 256 + wc * 64 + efq * 4) * 4u;
#pragma unroll
                for (int m = 0; m < 4; ++m)
#pragma unroll
                    for (int r = 0; r < 4; ++r) {
                        f32x4 o;
#pragma unroll
                        for (int e = 0; e < 4; ++e) {
                            const bool keep = (int)mk[m >> 1] < 0;
                            mk[m >> 1] <<= 1;
                            const u64 bal = __builtin_amdgcn_ballot_w64(keep);
                            const unsigned rk = __builtin_amdgcn_mbcnt_hi((unsigned)(bal >> 32), __builtin_amdgcn_mbcnt_lo((unsigned)bal, 0u));
                            const float x = *(const LAS float*)(lds + pos + rk * 4u);
                            o[e] = nodata ? qnan : (keep ? x : 0.f);
                            pos += (unsigned)__builtin_popcountll(bal) * 4u;
                        }
                        __builtin_amdgcn_raw_buffer_store_b128(__builtin_bit_cast(u32x4, o), rO, base + r * 64 + (unsigned)(m * 16 * GN) * 4u, 0, AUX_OUT);
                    }
                WAIT_L(0);
            }
        }
        STAMP(9);
    }
}

}

extern "C" void kernel_launch(int rank, int nranks, ncclComm_t comm,
                              hipStream_t stream,
                              void* const* d_in, const int* in_sizes, int n_in,
                              void* d_out, int out_size,
                              const int* in_dtypes, int out_dtype,
                              void* const* d_out_all,
                              void* const* d_ws_all, size_t ws_bytes) {
    if (nranks != NR || n_in < 2 || (unsigned)rank >= (unsigned)NR || ws_bytes < WS_NEED ||
        in_sizes[0] != GM * KS || in_sizes[1] != KS * GN || out_size != MS * GN) {
        fprintf(stderr, "kernel_launch: unexpected problem shape\n");
        return;
    }
    Params p;
    p.x = (const float*)d_in[0];
    p.w = (const float*)d_in[1];
    for (int r = 0; r < NR; ++r) { p.out_all[r] = (float*)d_out_all[r]; p.ws_all[r] = (char*)d_ws_all[r]; }
    for (int i = 0; i < 64; ++i)
        p.order[i] = piece_code(4 * i) | piece_code(4 * i + 1) << 8 | piece_code(4 * i + 2) << 16 | piece_code(4 * i + 3) << 24;
    p.rank = rank;
    hipLaunchKernelGGL(fused_gemm_rs, dim3(NWG), dim3(512), 0, stream, p);
    HIP_CHECK(hipGetLastError());
}
```

```cpp
#include <hip/hip_runtime.h>
#include <rccl/rccl.h>
#include <cstdio>

#define HIP_CHECK(expr) do { hipError_t _e = (expr); if (_e != hipSuccess) { \
    fprintf(stderr, "HIP error %s at %s:%d\n", hipGetErrorString(_e), __FILE__, __LINE__); \
    return; } } while (0)

namespace {

typedef _Float16 half8 __attribute__((ext_vector_type(8)));
typedef _Float16 half2v __attribute__((ext_vector_type(2)));
typedef float f32x4 __attribute__((ext_vector_type(4)));
typedef unsigned u32x4 __attribute__((ext_vector_type(4)));
typedef unsigned long long u64;
#define LAS __attribute__((address_space(3)))

constexpr int NR = 8;
constexpr int GM = 4096, GK = 4096, GN = 32768;
constexpr int KS = GK / NR;
constexpr int NB = GN / NR;
constexpr int MS = GM / NR;
constexpr int NWG = 256;
constexpr unsigned PIECE = 32768;
constexpr unsigned HTB = 16384;
constexpr unsigned PANEL = 64 * PIECE;
constexpr unsigned WS_GLOB = 0;
constexpr unsigned WS_Q = 1024;
constexpr unsigned WS_OST = 2048;
constexpr unsigned WS_FX = 8192;
constexpr unsigned WS_FW = WS_FX + 4096;
constexpr unsigned WS_CTRL = 65536;
constexpr unsigned WS_STAMP = 131072;
constexpr unsigned WS_FO = 262144;
constexpr unsigned WS_CTL_BYTES = 1u << 20;
constexpr size_t WS_STAGE = 1u << 20;
constexpr size_t STAGE_PAR = 32ull * PANEL;
constexpr unsigned CHUNK = 17408;
constexpr size_t WS_OBOX = WS_STAGE + 2 * STAGE_PAR;
constexpr size_t WS_NEED = WS_OBOX + 256ull * 16 * CHUNK;
constexpr u64 DEADLINE = 50000000ull;
constexpr int AUX_SYS = 17;
constexpr unsigned REG = 19968;
constexpr unsigned LDS_SNAP = 8 * REG + 1024;
constexpr unsigned LDS_MISC = LDS_SNAP + 2048;
constexpr unsigned LDS_BYTES = LDS_MISC + 64;
static_assert(LDS_BYTES <= 163840 && 8 * REG >= 8 * HTB && REG >= CHUNK + 1024, "LDS layout");
constexpr unsigned OOB = 0x40000000u;

#define QUOTA0 8
#define QUOTA1 8
#define QUOTA2 8
#define PASS_AHEAD 1
#define POLL_SLEEP 6
#define IN_BYTES PIECE
#define SRC_BYTES (8u * GN * 4u)
#define OUT_BYTES(x) (x)
#define AUX_OUT 2
#define LAYOUT 0
#define OUT_AHEAD 8
#define OUT_EXTRA 0
#define TOK_SLEEP 16
#define ALIGN 100
#define ALIGN3 ALIGN
#define DEF1 6
#define DEF2 4
#define DEF3 5
#define OUT_TOK 0
#define OUT_LEFT 4
#define CHAINS 1
#define PW 1
#define OUT_REM 1024
#define OUT_WIN 0
#define PNMAP 0
#define OUT_PASS 4

struct Params {
    const float* x;
    const float* w;
    float* out_all[NR];
    char* ws_all[NR];
    unsigned order[64];
    int rank;
};

static void make_order(unsigned* order) {
    const int def[4] = {8, DEF1, DEF2, DEF3};
    unsigned char q[256]; int n = 0;
    for (int j = 0; j < 8; ++j) {
        for (int pn = 0; pn < 16; ++pn) if (j < def[pn >> 2]) q[n++] = (unsigned char)(128 | pn << 3 | j);
        for (int pm = 0; pm < 16; ++pm) q[n++] = (unsigned char)(pm << 3 | j);
    }
    for (int g = 1; g < 4; ++g)
        for (int j = def[g]; j < 8; ++j)
            for (int pn = 4 * g; pn < 4 * g + 4; ++pn) q[n++] = (unsigned char)(128 | pn << 3 | j);
    for (int i = 0; i < 64; ++i) order[i] = q[4 * i] | q[4 * i + 1] << 8 | q[4 * i + 2] << 16 | (unsigned)q[4 * i + 3] << 24;
}

__device__ __forceinline__ unsigned pack2(float a, float b) {
    half2v h; h.x = (_Float16)a; h.y = (_Float16)b;
    return __builtin_bit_cast(unsigned, h);
}
__device__ __forceinline__ __amdgpu_buffer_rsrc_t make_rsrc(const void* p, unsigned bytes) {
    const u64 v = (u64)p;
    const unsigned lo = __builtin_amdgcn_readfirstlane((unsigned)v), hi = __builtin_amdgcn_readfirstlane((unsigned)(v >> 32));
    return __builtin_amdgcn_make_buffer_rsrc((void*)(((u64)hi << 32) | lo), 0, bytes, 0x00020000);
}

__device__ __forceinline__ void compact4(float x0, float x1, float x2, float x3, unsigned& mk, unsigned& pos) {
    unsigned a0, a1, a2, a3, c, p1, p2, p3;
    asm volatile(
        "v_cmp_nle_f32 s[88:89], %10, 0\n\t"
        "v_cmp_nle_f32 s[90:91], %11, 0\n\t"
        "v_cmp_nle_f32 s[92:93], %12, 0\n\t"
        "v_cmp_nle_f32 s[94:95], %13, 0\n\t"
        "s_bcnt1_i32_b64 %4, s[88:89]\n\t"
        "v_mbcnt_lo_u32_b32 %0, s88, 0\n\t"
        "s_lshl2_add_u32 %5, %4, %9\n\t"
        "v_mbcnt_lo_u32_b32 %1, s90, 0\n\t"
        "s_bcnt1_i32_b64 %4, s[90:91]\n\t"
        "v_mbcnt_lo_u32_b32 %2, s92, 0\n\t"
        "s_lshl2_add_u32 %6, %4, %5\n\t"
        "v_mbcnt_lo_u32_b32 %3, s94, 0\n\t"
        "s_bcnt1_i32_b64 %4, s[92:93]\n\t"
        "v_mbcnt_hi_u32_b32 %0, s89, %0\n\t"
        "s_lshl2_add_u32 %7, %4, %6\n\t"
        "v_mbcnt_hi_u32_b32 %1, s91, %1\n\t"
        "s_bcnt1_i32_b64 %4, s[94:95]\n\t"
        "v_mbcnt_hi_u32_b32 %2, s93, %2\n\t"
        "v_mbcnt_hi_u32_b32 %3, s95, %3\n\t"
        "v_lshl_add_u32 %0, %0, 2, %9\n\t"
        "v_lshl_add_u32 %1, %1, 2, %5\n\t"
        "v_lshl_add_u32 %2, %2, 2, %6\n\t"
        "v_lshl_add_u32 %3, %3, 2, %7\n\t"
        "s_lshl2_add_u32 %9, %4, %7\n\t"
        "v_addc_co_u32 %8, vcc, %8, %8, s[88:89]\n\t"
        "v_addc_co_u32 %8, vcc, %8, %8, s[90:91]\n\t"
        "v_addc_co_u32 %8, vcc, %8, %8, s[92:93]\n\t"
        "v_addc_co_u32 %8, vcc, %8, %8, s[94:95]\n\t"
        "s_mov_b64 exec, s[88:89]\n\t"
        "ds_write_b32 %0, %10\n\t"
        "s_mov_b64 exec, s[90:91]\n\t"
        "ds_write_b32 %1, %11\n\t"
        "s_mov_b64 exec, s[92:93]\n\t"
        "ds_write_b32 %2, %12\n\t"
        "s_mov_b64 exec, s[94:95]\n\t"
        "ds_write_b32 %3, %13\n\t"
        "s_mov_b64 exec, -1"
        : "=&v"(a0), "=&v"(a1), "=&v"(a2), "=&v"(a3), "=&s"(c), "=&s"(p1), "=&s"(p2), "=&s"(p3), "+v"(mk), "+s"(pos)
        : "v"(x0), "v"(x1), "v"(x2), "v"(x3)
        : "vcc", "scc", "memory", "s88", "s89", "s90", "s91", "s92", "s93", "s94", "s95");
}

__device__ __forceinline__ void expand16(f32x4 (&o)[4], unsigned& mk, unsigned& pos, float fill) {
    float x[16]; unsigned a0, a1, a2, a3, c, p1, p2, p3;
    asm volatile(
        "v_add_co_u32 %24, s[88:89], %24, %24\n\t"
        "v_add_co_u32 %24, s[90:91], %24, %24\n\t"
        "v_add_co_u32 %24, s[92:93], %24, %24\n\t"
        "v_add_co_u32 %24, s[94:95], %24, %24\n\t"
        "v_mov_b32 %0, %26\n\t"
        "v_mov_b32 %1, %26\n\t"
        "s_bcnt1_i32_b64 %20, s[88:89]\n\t"
        "v_mbcnt_lo_u32_b32 %16, s88, 0\n\t"
        "s_lshl2_add_u32 %21, %20, %25\n\t"
        "v_mbcnt_lo_u32_b32 %17, s90, 0\n\t"
        "s_bcnt1_i32_b64 %20, s[90:91]\n\t"
        "v_mbcnt_lo_u32_b32 %18, s92, 0\n\t"
        "s_lshl2_add_u32 %22, %20, %21\n\t"
        "v_mbcnt_lo_u32_b32 %19, s94, 0\n\t"
        "s_bcnt1_i32_b64 %20, s[92:93]\n\t"
        "v_mbcnt_hi_u32_b32 %16, s89, %16\n\t"
        "s_lshl2_add_u32 %23, %20, %22\n\t"
        "v_mbcnt_hi_u32_b32 %17, s91, %17\n\t"
        "s_bcnt1_i32_b64 %20, s[94:95]\n\t"
        "v_mbcnt_hi_u32_b32 %18, s93, %18\n\t"
        "v_mbcnt_hi_u32_b32 %19, s95, %19\n\t"
        "v_lshl_add_u32 %16, %16, 2, %25\n\t"
        "v_lshl_add_u32 %17, %17, 2, %21\n\t"
        "v_lshl_add_u32 %18, %18, 2, %22\n\t"
        "v_lshl_add_u32 %19, %19, 2, %23\n\t"
        "s_lshl2_add_u32 %25, %20, %23\n\t"
        "v_mov_b32 %2, %26\n\t"
        "v_mov_b32 %3, %26\n\t"
        "s_mov_b64 exec, s[88:89]\n\t"
        "ds_read_b32 %0, %16\n\t"
        "s_mov_b64 exec, s[90:91]\n\t"
        "ds_read_b32 %1, %17\n\t"
        "s_mov_b64 exec, s[92:93]\n\t"
        "ds_read_b32 %2, %18\n\t"
        "s_mov_b64 exec, s[94:95]\n\t"
        "ds_read_b32 %3, %19\n\t"
        "s_mov_b64 exec, -1\n\t"
        "v_add_co_u32 %24, s[88:89], %24, %24\n\t"
        "v_add_co_u32 %24, s[90:91], %24, %24\n\t"
        "v_add_co_u32 %24, s[92:93], %24, %24\n\t"
        "v_add_co_u32 %24, s[94:95], %24, %24\n\t"
        "v_mov_b32 %4, %26\n\t"
        "v_mov_b32 %5, %26\n\t"
        "s_bcnt1_i32_b64 %20, s[88:89]\n\t"
        "v_mbcnt_lo_u32_b32 %16, s88, 0\n\t"
        "s_lshl2_add_u32 %21, %20, %25\n\t"
        "v_mbcnt_lo_u32_b32 %17, s90, 0\n\t"
        "s_bcnt1_i32_b64 %20, s[90:91]\n\t"
        "v_mbcnt_lo_u32_b32 %18, s92, 0\n\t"
        "s_lshl2_add_u32 %22, %20, %21\n\t"
        "v_mbcnt_lo_u32_b32 %19, s94, 0\n\t"
        "s_bcnt1_i32_b64 %20, s[92:93]\n\t"
        "v_mbcnt_hi_u32_b32 %16, s89, %16\n\t"
        "s_lshl2_add_u32 %23, %20, %22\n\t"
        "v_mbcnt_hi_u32_b32 %17, s91, %17\n\t"
        "s_bcnt1_i32_b64 %20, s[94:95]\n\t"
        "v_mbcnt_hi_u32_b32 %18, s93, %18\n\t"
        "v_mbcnt_hi_u32_b32 %19, s95, %19\n\t"
        "v_lshl_add_u32 %16, %16, 2, %25\n\t"
        "v_lshl_add_u32 %17, %17, 2, %21\n\t"
        "v_lshl_add_u32 %18, %18, 2, %22\n\t"
        "v_lshl_add_u32 %19, %19, 2, %23\n\t"
        "s_lshl2_add_u32 %25, %20, %23\n\t"
        "v_mov_b32 %6, %26\n\t"
        "v_mov_b32 %7, %26\n\t"
        "s_mov_b64 exec, s[88:89]\n\t"
        "ds_read_b32 %4, %16\n\t"
        "s_mov_b64 exec, s[90:91]\n\t"
        "ds_read_b32 %5, %17\n\t"
        "s_mov_b64 exec, s[92:93]\n\t"
        "ds_read_b32 %6, %18\n\t"
        "s_mov_b64 exec, s[94:95]\n\t"
        "ds_read_b32 %7, %19\n\t"
        "s_mov_b64 exec, -1\n\t"
        "v_add_co_u32 %24, s[88:89], %24, %24\n\t"
        "v_add_co_u32 %24, s[90:91], %24, %24\n\t"
        "v_add_co_u32 %24, s[92:93], %24, %24\n\t"
        "v_add_co_u32 %24, s[94:95], %24, %24\n\t"
        "v_mov_b32 %8, %26\n\t"
        "v_mov_b32 %9, %26\n\t"
        "s_bcnt1_i32_b64 %20, s[88:89]\n\t"
        "v_mbcnt_lo_u32_b32 %16, s88, 0\n\t"
        "s_lshl2_add_u32 %21, %20, %25\n\t"
        "v_mbcnt_lo_u32_b32 %17, s90, 0\n\t"
        "s_bcnt1_i32_b64 %20, s[90:91]\n\t"
        "v_mbcnt_lo_u32_b32 %18, s92, 0\n\t"
        "s_lshl2_add_u32 %22, %20, %21\n\t"
        "v_mbcnt_lo_u32_b32 %19, s94, 0\n\t"
        "s_bcnt1_i32_b64 %20, s[92:93]\n\t"
        "v_mbcnt_hi_u32_b32 %16, s89, %16\n\t"
        "s_lshl2_add_u32 %23, %20, %22\n\t"
        "v_mbcnt_hi_u32_b32 %17, s91, %17\n\t"
        "s_bcnt1_i32_b64 %20, s[94:95]\n\t"
        "v_mbcnt_hi_u32_b32 %18, s93, %18\n\t"
        "v_mbcnt_hi_u32_b32 %19, s95, %19\n\t"
        "v_lshl_add_u32 %16, %16, 2, %25\n\t"
        "v_lshl_add_u32 %17, %17, 2, %21\n\t"
        "v_lshl_add_u32 %18, %18, 2, %22\n\t"
        "v_lshl_add_u32 %19, %19, 2, %23\n\t"
        "s_lshl2_add_u32 %25, %20, %23\n\t"
        "v_mov_b32 %10, %26\n\t"
        "v_mov_b32 %11, %26\n\t"
        "s_mov_b64 exec, s[88:89]\n\t"
        "ds_read_b32 %8, %16\n\t"
        "s_mov_b64 exec, s[90:91]\n\t"
        "ds_read_b32 %9, %17\n\t"
        "s_mov_b64 exec, s[92:93]\n\t"
        "ds_read_b32 %10, %18\n\t"
        "s_mov_b64 exec, s[94:95]\n\t"
        "ds_read_b32 %11, %19\n\t"
        "s_mov_b64 exec, -1\n\t"
        "v_add_co_u32 %24, s[88:89], %24, %24\n\t"
        "v_add_co_u32 %24, s[90:91], %24, %24\n\t"
        "v_add_co_u32 %24, s[92:93], %24, %24\n\t"
        "v_add_co_u32 %24, s[94:95], %24, %24\n\t"
        "v_mov_b32 %12, %26\n\t"
        "v_mov_b32 %13, %26\n\t"
        "s_bcnt1_i32_b64 %20, s[88:89]\n\t"
        "v_mbcnt_lo_u32_b32 %16, s88, 0\n\t"
        "s_lshl2_add_u32 %21, %20, %25\n\t"
        "v_mbcnt_lo_u32_b32 %17, s90, 0\n\t"
        "s_bcnt1_i32_b64 %20, s[90:91]\n\t"
        "v_mbcnt_lo_u32_b32 %18, s92, 0\n\t"
        "s_lshl2_add_u32 %22, %20, %21\n\t"
        "v_mbcnt_lo_u32_b32 %19, s94, 0\n\t"
        "s_bcnt1_i32_b64 %20, s[92:93]\n\t"
        "v_mbcnt_hi_u32_b32 %16, s89, %16\n\t"
        "s_lshl2_add_u32 %23, %20, %22\n\t"
        "v_mbcnt_hi_u32_b32 %17, s91, %17\n\t"
        "s_bcnt1_i32_b64 %20, s[94:95]\n\t"
        "v_mbcnt_hi_u32_b32 %18, s93, %18\n\t"
        "v_mbcnt_hi_u32_b32 %19, s95, %19\n\t"
        "v_lshl_add_u32 %16, %16, 2, %25\n\t"
        "v_lshl_add_u32 %17, %17, 2, %21\n\t"
        "v_lshl_add_u32 %18, %18, 2, %22\n\t"
        "v_lshl_add_u32 %19, %19, 2, %23\n\t"
        "s_lshl2_add_u32 %25, %20, %23\n\t"
        "v_mov_b32 %14, %26\n\t"
        "v_mov_b32 %15, %26\n\t"
        "s_mov_b64 exec, s[88:89]\n\t"
        "ds_read_b32 %12, %16\n\t"
        "s_mov_b64 exec, s[90:91]\n\t"
        "ds_read_b32 %13, %17\n\t"
        "s_mov_b64 exec, s[92:93]\n\t"
        "ds_read_b32 %14, %18\n\t"
        "s_mov_b64 exec, s[94:95]\n\t"
        "ds_read_b32 %15, %19\n\t"
        "s_mov_b64 exec, -1\n\t"
        "s_waitcnt lgkmcnt(0)"
        : "=&v"(x[0]), "=&v"(x[1]), "=&v"(x[2]), "=&v"(x[3]), "=&v"(x[4]), "=&v"(x[5]), "=&v"(x[6]), "=&v"(x[7]), "=&v"(x[8]), "=&v"(x[9]), "=&v"(x[10]), "=&v"(x[11]), "=&v"(x[12]), "=&v"(x[13]), "=&v"(x[14]), "=&v"(x[15]),
          "=&v"(a0), "=&v"(a1), "=&v"(a2), "=&v"(a3), "=&s"(c), "=&s"(p1), "=&s"(p2), "=&s"(p3), "+v"(mk), "+s"(pos)
        : "v"(fill)
        : "scc", "memory", "s88", "s89", "s90", "s91", "s92", "s93", "s94", "s95");
#pragma unroll
    for (int r = 0; r < 4; ++r) o[r] = (f32x4){x[4 * r], x[4 * r + 1], x[4 * r + 2], x[4 * r + 3]};
}

__global__ void __launch_bounds__(512, 2) fused_gemm_rs(const Params p) {
    __shared__ __attribute__((aligned(16))) unsigned char smem[LDS_BYTES];
    LAS unsigned char* lds = (LAS unsigned char*)smem;
    LAS unsigned* misc = (LAS unsigned*)(lds + LDS_MISC);

    const int tid = threadIdx.x, lane = tid & 63;
    const int wid = __builtin_amdgcn_readfirstlane(tid >> 6);
    const int wr = wid >> 2, wc = wid & 3, fr = lane & 15, fq = lane >> 4;
    const int b = blockIdx.x, xcd = b & 7, grp = b >> 6, v = (b >> 3) & 7;
    const int pm = 2 * v + (xcd & 1), pn = 4 * grp + (xcd >> 1);
    const int rank = p.rank;
    char* const ws = p.ws_all[rank];
    unsigned* const tmo = (unsigned*)(ws + WS_GLOB);
    u64* const queue = (u64*)(ws + WS_Q + v * 128);

#define STAMP(i) do { } while (0)
#define WAIT_V(n) asm volatile("s_waitcnt vmcnt(" #n ")" ::: "memory")
#define WAIT_VX(n) WAIT_V(n)
#define WAIT_L(n) asm volatile("s_waitcnt lgkmcnt(" #n ")" ::: "memory")
#define BAR __builtin_amdgcn_s_barrier()
#define SCHED __builtin_amdgcn_sched_barrier(0)
#define CFENCE asm volatile("" ::: "memory")

    if (tid == 0) {
        unsigned* cp = (unsigned*)(ws + WS_CTRL + b * 256) + 32;
        const unsigned c = __hip_atomic_load(cp, __ATOMIC_RELAXED, __HIP_MEMORY_SCOPE_AGENT) + 1u;
        __hip_atomic_store(cp, c, __ATOMIC_RELAXED, __HIP_MEMORY_SCOPE_AGENT);
        __hip_atomic_fetch_max(queue, (u64)c << 32, __ATOMIC_RELAXED, __HIP_MEMORY_SCOPE_AGENT);
        misc[0] = c;
        misc[5] = 0u; misc[8] = 0u; misc[9] = 0u;
    }
    __syncthreads();
    const unsigned call = __builtin_amdgcn_readfirstlane(misc[0]);
    const size_t stage_off = WS_STAGE + (size_t)(call & 1u) * STAGE_PAR;
    STAMP(0);

    f32x4 acc[2][2][4][2];
#pragma unroll
    for (int a = 0; a < 2; ++a)
#pragma unroll
        for (int c = 0; c < 2; ++c)
#pragma unroll
            for (int m = 0; m < 4; ++m)
#pragma unroll
                for (int n = 0; n < 2; ++n) acc[a][c][m][n] = (f32x4){0.f, 0.f, 0.f, 0.f};

    const __amdgpu_buffer_rsrc_t rA = make_rsrc(ws + stage_off + (size_t)pm * PANEL, PANEL);
    const __amdgpu_buffer_rsrc_t rB = make_rsrc(ws + stage_off + (size_t)(16 + pn) * PANEL, PANEL);
    const __amdgpu_buffer_rsrc_t rC = make_rsrc(ws, WS_CTL_BYTES);
    const unsigned snap_src = wid == 0 ? WS_FX + pm * 256 : wid == 1 ? WS_FW + pn * 256 : wid == 2 ? WS_CTRL + b * 256 : wid == 3 ? WS_GLOB : OOB;

#define SA(bb, h) (((bb) * 2 + (h)) * HTB)
#define SB(bb, h) ((4 + (bb) * 2 + (h)) * HTB)
#define STAGE(bufoff, rs, so) do { _Pragma("unroll") for (int _i = 0; _i < 2; ++_i) \
        __builtin_amdgcn_raw_ptr_buffer_load_lds(rs, (LAS void*)(lds + (bufoff) + ldsw + _i * 8192), 16, voff, (so) + _i * 8192, 0, 0); } while (0)
#define SNAP() __builtin_amdgcn_raw_ptr_buffer_load_lds(rC, (LAS void*)(lds + LDS_SNAP + wid * 256), 4, lane * 4, snap_src, 0, AUX_SYS)
#define LDA(dst, bb, h) do { _Pragma("unroll") for (int m = 0; m < 4; ++m) _Pragma("unroll") for (int k = 0; k < 2; ++k) \
        dst[m][k] = *(const LAS half8*)(lds + SA(bb, h) + (k ? aoff1 : aoff0) + m * 2048); } while (0)
#define LDB(dst, bb, h) do { _Pragma("unroll") for (int n = 0; n < 2; ++n) _Pragma("unroll") for (int k = 0; k < 2; ++k) \
        dst[n][k] = *(const LAS half8*)(lds + SA(bb, h) + boff + n * 1024 + k * 8192); } while (0)
#define MMA(ai, bj, At, Bt) do { __builtin_amdgcn_s_setprio(1); \
        _Pragma("unroll") for (int m = 0; m < 4; ++m) _Pragma("unroll") for (int n = 0; n < 2; ++n) _Pragma("unroll") for (int k = 0; k < 2; ++k) \
        acc[ai][bj][m][n] = __builtin_amdgcn_mfma_f32_16x16x32_f16(At[m][k], Bt[n][k], acc[ai][bj][m][n], 0, 0, 0); \
        __builtin_amdgcn_s_setprio(0); } while (0)
#define SNAP_ADDR(tt) (LDS_SNAP + (lane < 4 ? (lane >> 1) * 256 + (((tt) + (lane & 1)) & 63) * 4 : lane == 4 ? 512 : 768))

    int t = 0;
    int state = grp == 0 ? 1 : 0;
    do {
        if (state != 1) {
            const u64 t0 = __builtin_amdgcn_s_memrealtime();
            unsigned bits;
            for (;;) {
                SNAP(); WAIT_V(0); BAR; CFENCE;
                const unsigned sv = *(const LAS unsigned*)(lds + SNAP_ADDR(t));
                bits = (unsigned)__ballot(lane == 5 ? sv == call : sv >= call) & (state == 0 ? 63u : 47u);
                if (bits >= 16u || (t < 64 && bits == 15u)) break;
                if (tid == 0 && __builtin_amdgcn_s_memrealtime() - t0 > DEADLINE)
                    __hip_atomic_store(tmo, call, __ATOMIC_RELAXED, __HIP_MEMORY_SCOPE_AGENT);
                __builtin_amdgcn_s_sleep(POLL_SLEEP);
                CFENCE; BAR;
            }
            if (bits & 16u) state = 1;
            if (bits & 32u) { state = 3; t = 64; }
        }
        const bool run = state != 1 && t < 64;
        if (state == 1) {
            STAMP(3);
            state = 2;
            int tz = tid; asm volatile("" : "+v"(tz));
            const int ln = tz & 63;
            const int quota = grp == 0 ? QUOTA0 : grp == 1 ? QUOTA1 : grp == 2 ? QUOTA2 : 1000;
            char* const dws = p.ws_all[v];
            const int srel = (rank - v) & 7;
            const unsigned vlW = ln * 16, vlX = (ln >> 3) * 2048 + (ln & 7) * 32;
            const unsigned vsW = wid * 2048 + ln * 16;
            const unsigned vsX = wid * 4096 + (ln >> 3) * 128 + ((ln & 7) ^ (ln >> 4)) * 16;
#define DRAW(got, cnt, on) do { u64 _sv; const unsigned _m = __builtin_amdgcn_readfirstlane((wid == 0 && (on)) ? 1u : 0u); got = (u64)(cnt); \
                asm volatile("s_cmp_eq_u32 %3, 0\n\ts_cbranch_scc1 .Lskip%=\n\ts_mov_b64 %1, exec\n\ts_mov_b64 exec, 1\n\t" \
                             "global_atomic_add_x2 %0, %2, %0, off sc0\n\ts_mov_b64 exec, %1\n.Lskip%=:" \
                             : "+v"(got), "=&s"(_sv) : "v"(queue), "s"(_m) : "memory", "scc"); } while (0)
            int id0 = xcd, id1 = 8 + xcd;
            if (grp != 0) {
                u64 got; DRAW(got, 2, true);
                asm volatile("s_waitcnt vmcnt(0)" : "+v"(got) :: "memory");
                BAR;
                if (tz == 0) misc[1] = 16u + (unsigned)got;
                WAIT_L(0); BAR; CFENCE;
                id0 = __builtin_amdgcn_readfirstlane(misc[1]); id1 = id0 + 1;
            }
            f32x4 f[8];
            u32x4 pk[4];
            auto code_of = [&](int id) { return (int)((p.order[(id >> 2) & 63] >> ((id & 3) * 8)) & 255u); };
            __amdgpu_buffer_rsrc_t rL, rS; unsigned voL, voS0, voS1, sL1, sL2, sS2;
            auto load_setup = [&](int code) {
                const int type = code >> 7, panel = (code >> 3) & 15, j = code & 7;
                const float* src = type ? p.w + (size_t)(j * 64 + wid * 8) * GN + v * NB + panel * 256
                                        : p.x + (size_t)(panel * 256 + wid * 32) * KS + j * 64;
                rL = make_rsrc(src, 8u * GN * 4u);
                voL = type ? vlW : vlX;
                sL1 = type ? GN * 4 : 16; sL2 = type ? 2 * GN * 4 : 16384;
            };
#define LOAD(i) f[i] = __builtin_bit_cast(f32x4, __builtin_amdgcn_raw_buffer_load_b128(rL, voL, ((i) >> 1) * sL2 + ((i) & 1) * sL1, 0))
            auto convert = [&](int code) {
                if (code >> 7) {
#pragma unroll
                    for (int e = 0; e < 4; ++e)
                        pk[e] = (u32x4){pack2(f[0][e], f[1][e]), pack2(f[2][e], f[3][e]), pack2(f[4][e], f[5][e]), pack2(f[6][e], f[7][e])};
                } else {
#pragma unroll
                    for (int c = 0; c < 4; ++c)
                        pk[c] = (u32x4){pack2(f[2 * c][0], f[2 * c][1]), pack2(f[2 * c][2], f[2 * c][3]), pack2(f[2 * c + 1][0], f[2 * c + 1][1]), pack2(f[2 * c + 1][2], f[2 * c + 1][3])};
                }
            };
            auto store_setup = [&](int code) {
                const int type = code >> 7;
                rS = make_rsrc(dws + stage_off + (size_t)(code >> 3) * PANEL + (size_t)((code & 7) * 8 + srel) * PIECE, IN_BYTES);
                voS0 = type ? vsW : vsX; voS1 = (type ? vsW : vsX ^ 64u) + 1024u;
                sS2 = type ? HTB : 2048;
            };
#define STORE(c) __builtin_amdgcn_raw_buffer_store_b128(pk[c], rS, (((c) & 1) ? voS1 : voS0) + ((c) >> 1) * sS2, 0, AUX_SYS)
            auto publish = [&](int code) {
                __hip_atomic_store((unsigned*)(dws + WS_FX) + (code >> 3) * 64 + (code & 7) * 8 + srel, call, __ATOMIC_RELAXED, __HIP_MEMORY_SCOPE_SYSTEM);
            };
            unsigned* const next_tok = (unsigned*)(ws + WS_CTRL + ((b + 64) & 255) * 256) + (grp == 3 ? 1 : 0);
            auto hand_on = [&]() { __hip_atomic_store(next_tok, call, __ATOMIC_RELAXED, __HIP_MEMORY_SCOPE_AGENT); };
            bool passed = false;
            if (id0 < 256) {
                int c0 = code_of(id0), cp = -1;
                load_setup(c0);
                LOAD(0); LOAD(1); LOAD(2); LOAD(3); LOAD(4); LOAD(5); LOAD(6); LOAD(7);
                convert(c0);
                for (int n = 0;; ++n) {
                    const bool more = id1 < 256;
                    const bool fetch = more && n + 2 < quota;
                    const bool pass = !passed && grp < 3 && (!more || n + PASS_AHEAD >= quota);
                    const int c1 = code_of(id1);
                    store_setup(c0); load_setup(c1);
                    CFENCE;
                    STORE(0);
                    CFENCE;
                    u64 got; DRAW(got, 1, fetch);
                    if (pass && tz == 0) hand_on();
                    passed = passed || pass;
                    CFENCE;
                    LOAD(0); LOAD(1); LOAD(2);
                    WAIT_V(3);
                    if (cp >= 0 && ln == 0 && __hip_atomic_fetch_add(misc + 8 + (n & 1), 1u, __ATOMIC_RELAXED, __HIP_MEMORY_SCOPE_WORKGROUP) % 8u == 7u) publish(cp);
                    CFENCE;
                    STORE(1);
                    CFENCE;
                    LOAD(3); LOAD(4); LOAD(5);
                    WAIT_V(3);
                    STORE(2);
                    CFENCE;
                    LOAD(6); LOAD(7);
                    WAIT_V(2);
                    STORE(3);
                    __builtin_amdgcn_s_waitcnt(0x0F71);
                    asm volatile("s_waitcnt vmcnt(1)" : "+v"(got) :: "memory");
                    if (tz == 0) misc[2 + (n & 1)] = fetch ? 16u + (unsigned)got : 256u;
                    if (more) convert(c1);
                    WAIT_L(0); BAR; CFENCE;
                    if (!more) break;
                    cp = c0; c0 = c1; id1 = __builtin_amdgcn_readfirstlane(misc[2 + (n & 1)]);
                }
                if (OUT_TOK == 1 && !passed) { if (tz == 0) hand_on(); passed = true; }
                __builtin_amdgcn_s_waitcnt(0x0F70);
                WAIT_V(0); BAR; CFENCE;
                if (tz == 64) publish(c0);
            }
            if (!passed && tz == 0) hand_on();
            STAMP(4);
        }
        if (run) {
            const unsigned voff = (unsigned)tid * 16u;
            const unsigned ldsw = (unsigned)wid * 1024u;
            const int aoff0 = (wr * 64 + fr) * 128 + (fq ^ (fr >> 1)) * 16, aoff1 = aoff0 ^ 64;
            int boff = SB(0, 0) + fq * 2048 + (wc * 16 + fr) * 16;
            asm volatile("" : "+v"(boff));
            half8 At[4][2], B0[2][2], B1[2][2];
            unsigned so = (unsigned)t * PIECE;
            STAGE(SB(0, 0), rB, so); STAGE(SA(0, 0), rA, so); STAGE(SB(0, 1), rB, so + HTB); STAGE(SA(0, 1), rA, so + HTB);
            if (wr == 1) BAR;
            WAIT_V(4); BAR;
            STAGE(SB(1, 0), rB, so + PIECE); STAGE(SA(1, 0), rA, so + PIECE); STAGE(SB(1, 1), rB, so + PIECE + HTB);
            WAIT_V(6); BAR;
            bool go;
            do {
                const unsigned sv = *(const LAS unsigned*)(lds + SNAP_ADDR(t + 2));
                SCHED;
                LDB(B0, 0, 0); SCHED; LDA(At, 0, 0); STAGE(SA(1, 1), rA, so + PIECE + HTB);
                WAIT_L(8); BAR; WAIT_L(0); MMA(0, 0, At, B0);
                const unsigned bits = (unsigned)__ballot(sv >= call);
                go = t + 2 < 64 && (bits & (state == 0 ? 31u : 15u)) == 15u;
                const unsigned s2 = go ? so + 2 * PIECE : OOB;
                BAR; SCHED;
                LDB(B1, 0, 1); STAGE(SB(0, 0), rB, s2);
                BAR; WAIT_L(0); MMA(0, 1, At, B1); BAR;
                LDA(At, 0, 1); STAGE(SA(0, 0), rA, s2);
                BAR; WAIT_L(0); MMA(1, 0, At, B0); BAR; SCHED;
                STAGE(SB(0, 1), rB, s2 + HTB);
                if (!go) WAIT_V(0);
                WAIT_V(6); BAR; MMA(1, 1, At, B1); BAR;
                SNAP();
                LDB(B0, 1, 0); SCHED; LDA(At, 1, 0); STAGE(SA(0, 1), rA, s2 + HTB);
                WAIT_L(8); BAR; WAIT_L(0); MMA(0, 0, At, B0); BAR; SCHED;
                LDB(B1, 1, 1); STAGE(SB(1, 0), rB, s2 + PIECE);
                BAR; WAIT_L(0); MMA(0, 1, At, B1); BAR;
                LDA(At, 1, 1); STAGE(SA(1, 0), rA, s2 + PIECE);
                BAR; WAIT_L(0); MMA(1, 0, At, B0); BAR; SCHED;
                STAGE(SB(1, 1), rB, s2 + PIECE + HTB);
                WAIT_V(6); BAR; MMA(1, 1, At, B1); BAR;
                t += 2; so += 2 * PIECE;
            } while (go);
            WAIT_V(0);
            if (wr == 0) BAR;
            BAR;
        }
    } while (t < 64 || state < 2);
    STAMP(6);

    {
        int tz = tid; asm volatile("" : "+v"(tz));
        const int ln = tz & 63, efr = ln & 15, efq = ln >> 4;
        const float qnan = __builtin_nanf("");
        const __amdgpu_buffer_rsrc_t rO = make_rsrc(p.out_all[rank], (unsigned)MS * GN * 4u);
        const unsigned lds0 = (unsigned)(size_t)lds;
        const unsigned reg0 = (unsigned)wid * REG;
        const unsigned lane_off = (unsigned)((wr * 64 + efq * 4) * GN + wc * 64 + efr * 4) * 4u;
        if (v == rank) {
            const bool bad = __builtin_amdgcn_readfirstlane(__hip_atomic_load(tmo, __ATOMIC_RELAXED, __HIP_MEMORY_SCOPE_AGENT)) == call;
            const unsigned base = (unsigned)((pm & 1) * 256 * GN + rank * NB + pn * 256) * 4u + lane_off;
#pragma unroll
            for (int ai = 0; ai < 2; ++ai)
#pragma unroll
                for (int m = 0; m < 4; ++m)
#pragma unroll
                    for (int r = 0; r < 4; ++r) {
                        f32x4 o = (f32x4){acc[ai][0][m][0][r], acc[ai][0][m][1][r], acc[ai][1][m][0][r], acc[ai][1][m][1][r]};
#pragma unroll
                        for (int e = 0; e < 4; ++e) o[e] = bad ? qnan : (o[e] < 0.f ? 0.f : o[e]);
                        __builtin_amdgcn_raw_buffer_store_b128(__builtin_bit_cast(u32x4, o), rO, base + (unsigned)((ai * 128 + m * 16 + r) * GN) * 4u, 0, AUX_OUT);
                    }
        } else {
            char* const dws = p.ws_all[v];
            const unsigned slot = (unsigned)((rank * 32 + grp * 8 + xcd) * 16 + wid * 2);
            unsigned n1 = 0;
#pragma unroll
            for (int bj = 0; bj < 2; ++bj)
#pragma unroll
                for (int m = 0; m < 4; ++m)
#pragma unroll
                    for (int n = 0; n < 2; ++n)
#pragma unroll
                        for (int r = 0; r < 4; ++r) n1 += (unsigned)__builtin_popcountll(__builtin_amdgcn_ballot_w64(!(acc[1][bj][m][n][r] <= 0.f)));
            unsigned badm = 0;
            auto admit = [&]() {
                const unsigned* tok = (const unsigned*)(ws + WS_CTRL + (grp < CHAINS ? (b & 63) : b) * 256) + 1;
                unsigned* ost = (unsigned*)(ws + WS_OST + (v * 4 + grp) * 128);
                const u64 t0 = __builtin_amdgcn_s_memrealtime();
                for (;;) {
                    const unsigned tk = __hip_atomic_load(tok, __ATOMIC_RELAXED, __HIP_MEMORY_SCOPE_AGENT);
                    const unsigned tm = __hip_atomic_load(tmo, __ATOMIC_RELAXED, __HIP_MEMORY_SCOPE_AGENT);
                    const unsigned st = __hip_atomic_load(ost, __ATOMIC_RELAXED, __HIP_MEMORY_SCOPE_AGENT);
                    if (__builtin_amdgcn_readfirstlane(tm) == call) { badm = 0xffffffffu; break; }
                    if (__builtin_amdgcn_readfirstlane(tk) >= call) break;
                    if (__builtin_amdgcn_readfirstlane(st) >= (call - 1u) * 8u + (grp == 3 ? ALIGN3 : ALIGN)) break;
                    if (ln == 0 && __builtin_amdgcn_s_memrealtime() - t0 > DEADLINE) __hip_atomic_store(tmo, call, __ATOMIC_RELAXED, __HIP_MEMORY_SCOPE_AGENT);
                    __builtin_amdgcn_s_sleep(TOK_SLEEP);
                }
                if (tz == 0) __hip_atomic_fetch_add(ost, 1u, __ATOMIC_RELAXED, __HIP_MEMORY_SCOPE_AGENT);
                STAMP(5);
            };
            auto send = [&](unsigned from, unsigned bytes, unsigned sl, unsigned o0, unsigned o1) {
                const __amdgpu_buffer_rsrc_t rs = make_rsrc(dws + WS_OBOX + (size_t)sl * CHUNK, OUT_BYTES((bytes + 63u) & ~63u));
                for (unsigned o = o0; o < o1; o += 1024) {
                    const u32x4 d = *(const LAS u32x4*)(lds + from + ln * 16 + o);
                    WAIT_VX(OUT_WIN);
                    __builtin_amdgcn_raw_buffer_store_b128(d, rs, ln * 16 + o, 0, AUX_SYS);
                }
                CFENCE;
            };
            auto flag = [&](unsigned bytes, unsigned sl) {
                if (ln == 0) __hip_atomic_store((u64*)(dws + WS_FO + sl * 64), ((u64)call << 32) | (bytes | badm), __ATOMIC_RELAXED, __HIP_MEMORY_SCOPE_SYSTEM);
            };
            unsigned start[2], bytes[2];
            bool both = true;
#pragma unroll
            for (int ai = 0; ai < 2; ++ai) {
                start[ai] = (ai == 1 && both) ? reg0 + ((bytes[0] + 15u) & ~15u) : reg0;
                unsigned mk0 = 0u, mk1 = 0u;
                unsigned pos = lds0 + start[ai] + 512u;
#pragma unroll
                for (int m = 0; m < 4; ++m)
#pragma unroll
                    for (int r = 0; r < 4; ++r) {
                        if (m < 2) compact4(acc[ai][0][m][0][r], acc[ai][0][m][1][r], acc[ai][1][m][0][r], acc[ai][1][m][1][r], mk0, pos);
                        else compact4(acc[ai][0][m][0][r], acc[ai][0][m][1][r], acc[ai][1][m][0][r], acc[ai][1][m][1][r], mk1, pos);
                    }
                *(LAS u64*)(lds + start[ai] + ln * 8) = ((u64)mk1 << 32) | mk0;
                WAIT_L(0);
                bytes[ai] = pos - lds0 - start[ai];
                if (ai == 0) {
                    both = bytes[0] + 16u + 512u + n1 * 4u <= REG;
                    if (!both) { admit(); send(reg0, bytes[0], slot, 0u, bytes[0]); WAIT_L(0); }
                }
            }
            if (both) { admit(); send(reg0, bytes[0], slot, 0u, bytes[0]); }
            send(start[1], bytes[1], slot + 1, 0u, 1024u);
            WAIT_V(1);
            flag(bytes[0], slot);
            const unsigned cut = bytes[1] > 1024u + OUT_REM ? (bytes[1] - OUT_REM) & ~1023u : 1024u;
            send(start[1], bytes[1], slot + 1, 1024u, cut);
            if (ln == 0 && grp + CHAINS < 4 && __hip_atomic_fetch_add(misc + 5, 1u, __ATOMIC_RELAXED, __HIP_MEMORY_SCOPE_WORKGROUP) == OUT_PASS - 1)
                __hip_atomic_store((unsigned*)(ws + WS_CTRL + (b + 64 * CHAINS) * 256) + 1, call, __ATOMIC_RELAXED, __HIP_MEMORY_SCOPE_AGENT);
            send(start[1], bytes[1], slot + 1, cut, bytes[1]);
            STAMP(7);
            WAIT_V(0);
            flag(bytes[1], slot + 1);
            STAMP(8);
            const int njobs = grp < 2 ? 3 : grp == 2 ? 2 : 0;
            for (int jb = 0; jb < njobs; ++jb) {
                const int g2 = jb < 2 ? grp : 3, ai = jb < 2 ? jb : grp;
                const unsigned sl = (unsigned)((v * 32 + g2 * 8 + xcd) * 16 + wid * 2 + ai);
                const u64* fo = (const u64*)(ws + WS_FO + sl * 64);
                const u64 t0 = __builtin_amdgcn_s_memrealtime();
                unsigned bytes;
                for (;;) {
                    const u64 fv = __hip_atomic_load(fo, __ATOMIC_RELAXED, __HIP_MEMORY_SCOPE_SYSTEM);
                    bytes = __builtin_amdgcn_readfirstlane((unsigned)fv);
                    if (__builtin_amdgcn_readfirstlane((unsigned)(fv >> 32)) >= call) break;
                    if (__builtin_amdgcn_readfirstlane(__hip_atomic_load(tmo, __ATOMIC_RELAXED, __HIP_MEMORY_SCOPE_AGENT)) == call
                        || __builtin_amdgcn_s_memrealtime() - t0 > DEADLINE) {
                        if (ln == 0) __hip_atomic_store(tmo, call, __ATOMIC_RELAXED, __HIP_MEMORY_SCOPE_AGENT);
                        bytes = 0xffffffffu;
                        break;
                    }
                    __builtin_amdgcn_s_sleep(8);
                }
                const bool nodata = bytes > 512u + 16384u;
                if (nodata) bytes = 0u;
                const __amdgpu_buffer_rsrc_t rs = make_rsrc(ws + WS_OBOX + (size_t)sl * CHUNK, bytes);
                for (unsigned o = 0; o < bytes; o += 1024)
                    __builtin_amdgcn_raw_ptr_buffer_load_lds(rs, (LAS void*)(lds + reg0 + o), 16, ln * 16, o, 0, 0);
                WAIT_V(0);
                const u64 mm = *(const LAS u64*)(lds + reg0 + ln * 8);
                unsigned mk = nodata ? 0u : (unsigned)mm;
                unsigned pos = lds0 + reg0 + 512u;
                unsigned off = (unsigned)(((xcd & 1) * 256 + ai * 128) * GN + v * NB + (4 * g2 + (xcd >> 1)) * 256) * 4u + lane_off;
                const float fill = nodata ? qnan : 0.f;
                for (int m = 0; m < 4; ++m) {
                    f32x4 o[4];
                    expand16(o, mk, pos, fill);
#pragma unroll
                    for (int r = 0; r < 4; ++r)
                        __builtin_amdgcn_raw_buffer_store_b128(__builtin_bit_cast(u32x4, o[r]), rO, off + (unsigned)(r * GN) * 4u, 0, AUX_OUT);
                    off += 16u * GN * 4u;
                    if (m == 1) mk = nodata ? 0u : (unsigned)(mm >> 32);
                }
            }
        }
        STAMP(9);
    }
}

}

extern "C" void kernel_launch(int rank, int nranks, ncclComm_t comm,
                              hipStream_t stream,
                              void* const* d_in, const int* in_sizes, int n_in,
                              void* d_out, int out_size,
                              const int* in_dtypes, int out_dtype,
                              void* const* d_out_all,
                              void* const* d_ws_all, size_t ws_bytes) {
    if (nranks != NR || n_in < 2 || (unsigned)rank >= (unsigned)NR || ws_bytes < WS_NEED ||
        in_sizes[0] != GM * KS || in_sizes[1] != KS * GN || out_size != MS * GN) {
        fprintf(stderr, "kernel_launch: unexpected problem shape\n");
        return;
    }
    Params p;
    p.x = (const float*)d_in[0];
    p.w = (const float*)d_in[1];
    for (int r = 0; r < NR; ++r) { p.out_all[r] = (float*)d_out_all[r]; p.ws_all[r] = (char*)d_ws_all[r]; }
    make_order(p.order);
    p.rank = rank;
    hipLaunchKernelGGL(fused_gemm_rs, dim3(NWG), dim3(512), 0, stream, p);
    HIP_CHECK(hipGetLastError());
}
```

```cpp
#include <hip/hip_runtime.h>
#include <rccl/rccl.h>
#include <cstdio>

#define HIP_CHECK(expr) do { hipError_t _e = (expr); if (_e != hipSuccess) { \
    fprintf(stderr, "HIP error %s at %s:%d\n", hipGetErrorString(_e), __FILE__, __LINE__); \
    return; } } while (0)

namespace {

typedef _Float16 half8 __attribute__((ext_vector_type(8)));
typedef _Float16 half2v __attribute__((ext_vector_type(2)));
typedef float f32x4 __attribute__((ext_vector_type(4)));
typedef unsigned u32x4 __attribute__((ext_vector_type(4)));
typedef unsigned long long u64;
#define LAS __attribute__((address_space(3)))

constexpr int NR = 8;
constexpr int GM = 4096, GK = 4096, GN = 32768;
constexpr int KS = GK / NR;
constexpr int NB = GN / NR;
constexpr int MS = GM / NR;
constexpr int NWG = 256;
constexpr unsigned PIECE = 32768;
constexpr unsigned HTB = 16384;
constexpr unsigned PANEL = 64 * PIECE;
constexpr unsigned CHUNK = 17408;
constexpr unsigned WS_GLOB = 0;
constexpr unsigned WS_Q = 1024;
constexpr unsigned WS_FX = 8192;
constexpr unsigned WS_FW = WS_FX + 4096;
constexpr unsigned WS_RDY = 16384;
constexpr unsigned WS_CTRL = 65536;
constexpr unsigned WS_STAMP = 131072;
constexpr unsigned WS_FO = 262144;
constexpr unsigned WS_CTL_BYTES = 1u << 20;
constexpr size_t WS_STAGE = 1u << 20;
constexpr size_t STAGE_PAR = 48ull * PANEL;
constexpr size_t SRCW = 32ull * PANEL;
constexpr size_t WS_OBOX = WS_STAGE + 2 * STAGE_PAR;
constexpr size_t WS_NEED = WS_OBOX + 4096ull * CHUNK;
constexpr u64 DEADLINE = 50000000ull;
constexpr int AUX_SYS = 17;
constexpr int AUX_NT = 2;
constexpr unsigned LDS_SNAP = 8 * CHUNK + 1024;
constexpr unsigned LDS_MISC = LDS_SNAP + 2048;
constexpr unsigned LDS_BYTES = LDS_MISC + 64;
static_assert(LDS_BYTES <= 163840 && CHUNK >= HTB, "LDS layout");
constexpr unsigned OOB = 0x40000000u;

#define QUOTA0 8
#define QUOTA1 8
#define QUOTA2 8
#define PASS_AHEAD 1
#define DEF1 6
#define DEF2 5
#define DEF3 4
#define PP 0
#define CONV_SLEEP 0
#define DIRECT_WIN 2
#define OUT_TOK 1
#define PULLW1 1
#define PULLW2 0
#define PULLW3 0
#define PULL_W1 32u

struct Params {
    const float* x;
    const float* w;
    float* out;
    char* ws_all[NR];
    unsigned order[64];
    unsigned worder[32];
    int rank;
};

static void make_order(unsigned* order, unsigned* worder) {
    const int def[4] = {8, DEF1, DEF2, DEF3};
    unsigned char q[256], wq[128]; int n = 0, m = 0;
    for (int j = 0; j < 8; ++j) {
        for (int pn = 0; pn < 16; ++pn) if (j < def[pn >> 2]) q[n++] = (unsigned char)(128 | pn << 3 | j);
        for (int pm = 0; pm < 16; ++pm) q[n++] = (unsigned char)(pm << 3 | j);
    }
    for (int g = 1; g < 4; ++g)
        for (int j = def[g]; j < 8; ++j)
            for (int pn = 4 * g; pn < 4 * g + 4; ++pn) q[n++] = (unsigned char)(128 | pn << 3 | j);
    for (int i = 0; i < 256; ++i) if (q[i] & 128) wq[m++] = q[i];
    for (int i = 0; i < 64; ++i) order[i] = q[4 * i] | q[4 * i + 1] << 8 | q[4 * i + 2] << 16 | (unsigned)q[4 * i + 3] << 24;
    for (int i = 0; i < 32; ++i) worder[i] = wq[4 * i] | wq[4 * i + 1] << 8 | wq[4 * i + 2] << 16 | (unsigned)wq[4 * i + 3] << 24;
}

__device__ __forceinline__ unsigned pack2(float a, float b) {
    half2v h; h.x = (_Float16)a; h.y = (_Float16)b;
    return __builtin_bit_cast(unsigned, h);
}
__device__ __forceinline__ __amdgpu_buffer_rsrc_t make_rsrc(const void* p, unsigned bytes) {
    const u64 v = (u64)p;
    const unsigned lo = __builtin_amdgcn_readfirstlane((unsigned)v), hi = __builtin_amdgcn_readfirstlane((unsigned)(v >> 32));
    return __builtin_amdgcn_make_buffer_rsrc((void*)(((u64)hi << 32) | lo), 0, bytes, 0x00020000);
}

__device__ __forceinline__ void compact4(float x0, float x1, float x2, float x3, unsigned& mk, unsigned& pos) {
    unsigned a0, a1, a2, a3, c, p1, p2, p3;
    asm volatile(
        "v_cmp_nle_f32 s[88:89], %10, 0\n\t"
        "v_cmp_nle_f32 s[90:91], %11, 0\n\t"
        "v_cmp_nle_f32 s[92:93], %12, 0\n\t"
        "v_cmp_nle_f32 s[94:95], %13, 0\n\t"
        "s_bcnt1_i32_b64 %4, s[88:89]\n\t"
        "v_mbcnt_lo_u32_b32 %0, s88, 0\n\t"
        "s_lshl2_add_u32 %5, %4, %9\n\t"
        "v_mbcnt_lo_u32_b32 %1, s90, 0\n\t"
        "s_bcnt1_i32_b64 %4, s[90:91]\n\t"
        "v_mbcnt_lo_u32_b32 %2, s92, 0\n\t"
        "s_lshl2_add_u32 %6, %4, %5\n\t"
        "v_mbcnt_lo_u32_b32 %3, s94, 0\n\t"
        "s_bcnt1_i32_b64 %4, s[92:93]\n\t"
        "v_mbcnt_hi_u32_b32 %0, s89, %0\n\t"
        "s_lshl2_add_u32 %7, %4, %6\n\t"
        "v_mbcnt_hi_u32_b32 %1, s91, %1\n\t"
        "s_bcnt1_i32_b64 %4, s[94:95]\n\t"
        "v_mbcnt_hi_u32_b32 %2, s93, %2\n\t"
        "v_mbcnt_hi_u32_b32 %3, s95, %3\n\t"
        "v_lshl_add_u32 %0, %0, 2, %9\n\t"
        "v_lshl_add_u32 %1, %1, 2, %5\n\t"
        "v_lshl_add_u32 %2, %2, 2, %6\n\t"
        "v_lshl_add_u32 %3, %3, 2, %7\n\t"
        "s_lshl2_add_u32 %9, %4, %7\n\t"
        "v_addc_co_u32 %8, vcc, %8, %8, s[88:89]\n\t"
        "v_addc_co_u32 %8, vcc, %8, %8, s[90:91]\n\t"
        "v_addc_co_u32 %8, vcc, %8, %8, s[92:93]\n\t"
        "v_addc_co_u32 %8, vcc, %8, %8, s[94:95]\n\t"
        "s_mov_b64 exec, s[88:89]\n\t"
        "ds_write_b32 %0, %10\n\t"
        "s_mov_b64 exec, s[90:91]\n\t"
        "ds_write_b32 %1, %11\n\t"
        "s_mov_b64 exec, s[92:93]\n\t"
        "ds_write_b32 %2, %12\n\t"
        "s_mov_b64 exec, s[94:95]\n\t"
        "ds_write_b32 %3, %13\n\t"
        "s_mov_b64 exec, -1"
        : "=&v"(a0), "=&v"(a1), "=&v"(a2), "=&v"(a3), "=&s"(c), "=&s"(p1), "=&s"(p2), "=&s"(p3), "+v"(mk), "+s"(pos)
        : "v"(x0), "v"(x1), "v"(x2), "v"(x3)
        : "vcc", "scc", "memory", "s88", "s89", "s90", "s91", "s92", "s93", "s94", "s95");
}

__device__ __forceinline__ void expand16(f32x4 (&o)[4], unsigned& mk, unsigned& pos, float fill) {
    float x[16]; unsigned a0, a1, a2, a3, c, p1, p2, p3;
    asm volatile(
        "v_add_co_u32 %24, s[88:89], %24, %24\n\t"
        "v_add_co_u32 %24, s[90:91], %24, %24\n\t"
        "v_add_co_u32 %24, s[92:93], %24, %24\n\t"
        "v_add_co_u32 %24, s[94:95], %24, %24\n\t"
        "v_mov_b32 %0, %26\n\t"
        "v_mov_b32 %1, %26\n\t"
        "s_bcnt1_i32_b64 %20, s[88:89]\n\t"
        "v_mbcnt_lo_u32_b32 %16, s88, 0\n\t"
        "s_lshl2_add_u32 %21, %20, %25\n\t"
        "v_mbcnt_lo_u32_b32 %17, s90, 0\n\t"
        "s_bcnt1_i32_b64 %20, s[90:91]\n\t"
        "v_mbcnt_lo_u32_b32 %18, s92, 0\n\t"
        "s_lshl2_add_u32 %22, %20, %21\n\t"
        "v_mbcnt_lo_u32_b32 %19, s94, 0\n\t"
        "s_bcnt1_i32_b64 %20, s[92:93]\n\t"
        "v_mbcnt_hi_u32_b32 %16, s89, %16\n\t"
        "s_lshl2_add_u32 %23, %20, %22\n\t"
        "v_mbcnt_hi_u32_b32 %17, s91, %17\n\t"
        "s_bcnt1_i32_b64 %20, s[94:95]\n\t"
        "v_mbcnt_hi_u32_b32 %18, s93, %18\n\t"
        "v_mbcnt_hi_u32_b32 %19, s95, %19\n\t"
        "v_lshl_add_u32 %16, %16, 2, %25\n\t"
        "v_lshl_add_u32 %17, %17, 2, %21\n\t"
        "v_lshl_add_u32 %18, %18, 2, %22\n\t"
        "v_lshl_add_u32 %19, %19, 2, %23\n\t"
        "s_lshl2_add_u32 %25, %20, %23\n\t"
        "v_mov_b32 %2, %26\n\t"
        "v_mov_b32 %3, %26\n\t"
        "s_mov_b64 exec, s[88:89]\n\t"
        "ds_read_b32 %0, %16\n\t"
        "s_mov_b64 exec, s[90:91]\n\t"
        "ds_read_b32 %1, %17\n\t"
        "s_mov_b64 exec, s[92:93]\n\t"
        "ds_read_b32 %2, %18\n\t"
        "s_mov_b64 exec, s[94:95]\n\t"
        "ds_read_b32 %3, %19\n\t"
        "s_mov_b64 exec, -1\n\t"
        "v_add_co_u32 %24, s[88:89], %24, %24\n\t"
        "v_add_co_u32 %24, s[90:91], %24, %24\n\t"
        "v_add_co_u32 %24, s[92:93], %24, %24\n\t"
        "v_add_co_u32 %24, s[94:95], %24, %24\n\t"
        "v_mov_b32 %4, %26\n\t"
        "v_mov_b32 %5, %26\n\t"
        "s_bcnt1_i32_b64 %20, s[88:89]\n\t"
        "v_mbcnt_lo_u32_b32 %16, s88, 0\n\t"
        "s_lshl2_add_u32 %21, %20, %25\n\t"
        "v_mbcnt_lo_u32_b32 %17, s90, 0\n\t"
        "s_bcnt1_i32_b64 %20, s[90:91]\n\t"
        "v_mbcnt_lo_u32_b32 %18, s92, 0\n\t"
        "s_lshl2_add_u32 %22, %20, %21\n\t"
        "v_mbcnt_lo_u32_b32 %19, s94, 0\n\t"
        "s_bcnt1_i32_b64 %20, s[92:93]\n\t"
        "v_mbcnt_hi_u32_b32 %16, s89, %16\n\t"
        "s_lshl2_add_u32 %23, %20, %22\n\t"
        "v_mbcnt_hi_u32_b32 %17, s91, %17\n\t"
        "s_bcnt1_i32_b64 %20, s[94:95]\n\t"
        "v_mbcnt_hi_u32_b32 %18, s93, %18\n\t"
        "v_mbcnt_hi_u32_b32 %19, s95, %19\n\t"
        "v_lshl_add_u32 %16, %16, 2, %25\n\t"
        "v_lshl_add_u32 %17, %17, 2, %21\n\t"
        "v_lshl_add_u32 %18, %18, 2, %22\n\t"
        "v_lshl_add_u32 %19, %19, 2, %23\n\t"
        "s_lshl2_add_u32 %25, %20, %23\n\t"
        "v_mov_b32 %6, %26\n\t"
        "v_mov_b32 %7, %26\n\t"
        "s_mov_b64 exec, s[88:89]\n\t"
        "ds_read_b32 %4, %16\n\t"
        "s_mov_b64 exec, s[90:91]\n\t"
        "ds_read_b32 %5, %17\n\t"
        "s_mov_b64 exec, s[92:93]\n\t"
        "ds_read_b32 %6, %18\n\t"
        "s_mov_b64 exec, s[94:95]\n\t"
        "ds_read_b32 %7, %19\n\t"
        "s_mov_b64 exec, -1\n\t"
        "v_add_co_u32 %24, s[88:89], %24, %24\n\t"
        "v_add_co_u32 %24, s[90:91], %24, %24\n\t"
        "v_add_co_u32 %24, s[92:93], %24, %24\n\t"
        "v_add_co_u32 %24, s[94:95], %24, %24\n\t"
        "v_mov_b32 %8, %26\n\t"
        "v_mov_b32 %9, %26\n\t"
        "s_bcnt1_i32_b64 %20, s[88:89]\n\t"
        "v_mbcnt_lo_u32_b32 %16, s88, 0\n\t"
        "s_lshl2_add_u32 %21, %20, %25\n\t"
        "v_mbcnt_lo_u32_b32 %17, s90, 0\n\t"
        "s_bcnt1_i32_b64 %20, s[90:91]\n\t"
        "v_mbcnt_lo_u32_b32 %18, s92, 0\n\t"
        "s_lshl2_add_u32 %22, %20, %21\n\t"
        "v_mbcnt_lo_u32_b32 %19, s94, 0\n\t"
        "s_bcnt1_i32_b64 %20, s[92:93]\n\t"
        "v_mbcnt_hi_u32_b32 %16, s89, %16\n\t"
        "s_lshl2_add_u32 %23, %20, %22\n\t"
        "v_mbcnt_hi_u32_b32 %17, s91, %17\n\t"
        "s_bcnt1_i32_b64 %20, s[94:95]\n\t"
        "v_mbcnt_hi_u32_b32 %18, s93, %18\n\t"
        "v_mbcnt_hi_u32_b32 %19, s95, %19\n\t"
        "v_lshl_add_u32 %16, %16, 2, %25\n\t"
        "v_lshl_add_u32 %17, %17, 2, %21\n\t"
        "v_lshl_add_u32 %18, %18, 2, %22\n\t"
        "v_lshl_add_u32 %19, %19, 2, %23\n\t"
        "s_lshl2_add_u32 %25, %20, %23\n\t"
        "v_mov_b32 %10, %26\n\t"
        "v_mov_b32 %11, %26\n\t"
        "s_mov_b64 exec, s[88:89]\n\t"
        "ds_read_b32 %8, %16\n\t"
        "s_mov_b64 exec, s[90:91]\n\t"
        "ds_read_b32 %9, %17\n\t"
        "s_mov_b64 exec, s[92:93]\n\t"
        "ds_read_b32 %10, %18\n\t"
        "s_mov_b64 exec, s[94:95]\n\t"
        "ds_read_b32 %11, %19\n\t"
        "s_mov_b64 exec, -1\n\t"
        "v_add_co_u32 %24, s[88:89], %24, %24\n\t"
        "v_add_co_u32 %24, s[90:91], %24, %24\n\t"
        "v_add_co_u32 %24, s[92:93], %24, %24\n\t"
        "v_add_co_u32 %24, s[94:95], %24, %24\n\t"
        "v_mov_b32 %12, %26\n\t"
        "v_mov_b32 %13, %26\n\t"
        "s_bcnt1_i32_b64 %20, s[88:89]\n\t"
        "v_mbcnt_lo_u32_b32 %16, s88, 0\n\t"
        "s_lshl2_add_u32 %21, %20, %25\n\t"
        "v_mbcnt_lo_u32_b32 %17, s90, 0\n\t"
        "s_bcnt1_i32_b64 %20, s[90:91]\n\t"
        "v_mbcnt_lo_u32_b32 %18, s92, 0\n\t"
        "s_lshl2_add_u32 %22, %20, %21\n\t"
        "v_mbcnt_lo_u32_b32 %19, s94, 0\n\t"
        "s_bcnt1_i32_b64 %20, s[92:93]\n\t"
        "v_mbcnt_hi_u32_b32 %16, s89, %16\n\t"
        "s_lshl2_add_u32 %23, %20, %22\n\t"
        "v_mbcnt_hi_u32_b32 %17, s91, %17\n\t"
        "s_bcnt1_i32_b64 %20, s[94:95]\n\t"
        "v_mbcnt_hi_u32_b32 %18, s93, %18\n\t"
        "v_mbcnt_hi_u32_b32 %19, s95, %19\n\t"
        "v_lshl_add_u32 %16, %16, 2, %25\n\t"
        "v_lshl_add_u32 %17, %17, 2, %21\n\t"
        "v_lshl_add_u32 %18, %18, 2, %22\n\t"
        "v_lshl_add_u32 %19, %19, 2, %23\n\t"
        "s_lshl2_add_u32 %25, %20, %23\n\t"
        "v_mov_b32 %14, %26\n\t"
        "v_mov_b32 %15, %26\n\t"
        "s_mov_b64 exec, s[88:89]\n\t"
        "ds_read_b32 %12, %16\n\t"
        "s_mov_b64 exec, s[90:91]\n\t"
        "ds_read_b32 %13, %17\n\t"
        "s_mov_b64 exec, s[92:93]\n\t"
        "ds_read_b32 %14, %18\n\t"
        "s_mov_b64 exec, s[94:95]\n\t"
        "ds_read_b32 %15, %19\n\t"
        "s_mov_b64 exec, -1\n\t"
        "s_waitcnt lgkmcnt(0)"
        : "=&v"(x[0]), "=&v"(x[1]), "=&v"(x[2]), "=&v"(x[3]), "=&v"(x[4]), "=&v"(x[5]), "=&v"(x[6]), "=&v"(x[7]), "=&v"(x[8]), "=&v"(x[9]), "=&v"(x[10]), "=&v"(x[11]), "=&v"(x[12]), "=&v"(x[13]), "=&v"(x[14]), "=&v"(x[15]),
          "=&v"(a0), "=&v"(a1), "=&v"(a2), "=&v"(a3), "=&s"(c), "=&s"(p1), "=&s"(p2), "=&s"(p3), "+v"(mk), "+s"(pos)
        : "v"(fill)
        : "scc", "memory", "s88", "s89", "s90", "s91", "s92", "s93", "s94", "s95");
#pragma unroll
    for (int r = 0; r < 4; ++r) o[r] = (f32x4){x[4 * r], x[4 * r + 1], x[4 * r + 2], x[4 * r + 3]};
}

__global__ void __launch_bounds__(512, 2) fused_gemm_rs(const Params p) {
    __shared__ __attribute__((aligned(16))) unsigned char smem[LDS_BYTES];
    LAS unsigned char* lds = (LAS unsigned char*)smem;
    LAS unsigned* misc = (LAS unsigned*)(lds + LDS_MISC);

    const int tid = threadIdx.x, lane = tid & 63;
    const int wid = __builtin_amdgcn_readfirstlane(tid >> 6);
    const int wr = wid >> 2, wc = wid & 3, fr = lane & 15, fq = lane >> 4;
    const int b = blockIdx.x, xcd = b & 7, grp = b >> 6, v = (b >> 3) & 7;
    const int pm = 2 * v + (xcd & 1), pn = 4 * grp + (xcd >> 1);
    const int rank = p.rank;
    char* const ws = p.ws_all[rank];
    char* const pws = p.ws_all[v];
    unsigned* const tmo = (unsigned*)(ws + WS_GLOB);
    u64* const queue = (u64*)(ws + WS_Q + v * 128);

#define STAMP(i) do { } while (0)
#define WAIT_V(n) asm volatile("s_waitcnt vmcnt(" #n ")" ::: "memory")
#define WAIT_VC(n) do { __builtin_amdgcn_s_waitcnt(0x0F70 | (n)); WAIT_V(n); } while (0)
#define WAIT_L(n) asm volatile("s_waitcnt lgkmcnt(" #n ")" ::: "memory")
#define BAR __builtin_amdgcn_s_barrier()
#define SCHED __builtin_amdgcn_sched_barrier(0)
#define CFENCE asm volatile("" ::: "memory")
#define RLX_AGENT __ATOMIC_RELAXED, __HIP_MEMORY_SCOPE_AGENT
#define RLX_SYS __ATOMIC_RELAXED, __HIP_MEMORY_SCOPE_SYSTEM

    unsigned call;
    size_t stage_off;
    {
        const int njobs = grp == 0 ? 0 : 6;
        auto job_code = [&](int n) {
            if (grp == 3 && n == 0) { const int u = v * 8 + xcd; return (u & 15) << 3 | (u >> 4); }
            if (grp == 2 && n == 5) { const int u = 64 + v * 8 + xcd; return (u & 15) << 3 | (u >> 4); }
            const int i = xcd + (grp - 1) * 8 + 24 * (grp == 3 ? n - 1 : n);
            return (int)((p.worder[(i >> 2) & 31] >> ((i & 3) * 8)) & 255u);
        };
        f32x4 f[8];
        auto load = [&](int code) {
            const int type = code >> 7, panel = (code >> 3) & 15, j = code & 7;
            const float* src = type ? p.w + (size_t)(j * 64 + wid * 8) * GN + v * NB + panel * 256
                                    : p.x + (size_t)(panel * 256 + wid * 32) * KS + j * 64;
            const __amdgpu_buffer_rsrc_t rs = make_rsrc(src, 8u * GN * 4u);
            const unsigned vo = type ? lane * 16 : (lane >> 3) * 2048 + (lane & 7) * 32;
            const unsigned s1 = type ? GN * 4 : 16, s2 = type ? 2 * GN * 4 : 16384;
#pragma unroll
            for (int i = 0; i < 8; ++i)
                f[i] = __builtin_bit_cast(f32x4, __builtin_amdgcn_raw_buffer_load_b128(rs, vo, (i >> 1) * s2 + (i & 1) * s1, 0));
        };
        int code = job_code(0);
        if (grp != 0) load(code);
        char* const lws = p.ws_all[tid & 7];
        CFENCE;
        if (tid == 0) {
            unsigned* cp = (unsigned*)(ws + WS_CTRL + b * 256) + 32;
            const unsigned c = __hip_atomic_load(cp, RLX_AGENT) + 1u;
            __hip_atomic_store(cp, c, RLX_AGENT);
            __hip_atomic_fetch_max(queue, (u64)c << 32, RLX_AGENT);
            __hip_atomic_fetch_max(queue + 1, (u64)c << 32, RLX_AGENT);
            misc[0] = c; misc[8] = 0u; misc[9] = 0u;
        }
        WAIT_L(0); BAR; CFENCE;
        call = __builtin_amdgcn_readfirstlane(misc[0]);
        stage_off = WS_STAGE + (size_t)(call & 1u) * STAGE_PAR;
        STAMP(0);
#pragma unroll 1
        for (int n = 0; n < njobs; ++n) {
            const int type = code >> 7;
            u32x4 pk[4];
            if (type) {
#pragma unroll
                for (int e = 0; e < 4; ++e)
                    pk[e] = (u32x4){pack2(f[0][e], f[1][e]), pack2(f[2][e], f[3][e]), pack2(f[4][e], f[5][e]), pack2(f[6][e], f[7][e])};
            } else {
#pragma unroll
                for (int c = 0; c < 4; ++c)
                    pk[c] = (u32x4){pack2(f[2 * c][0], f[2 * c][1]), pack2(f[2 * c][2], f[2 * c][3]), pack2(f[2 * c + 1][0], f[2 * c + 1][1]), pack2(f[2 * c + 1][2], f[2 * c + 1][3])};
            }
            const bool direct = grp == 1 && n == 0;
            const bool staged = !type || v == rank || direct;
            const int srel = direct ? (rank - v) & 7 : 0;
            char* const dbase = direct ? pws : ws;
            const __amdgpu_buffer_rsrc_t rs = make_rsrc(dbase + stage_off + (staged ? (size_t)(code >> 3) * PANEL + (size_t)((code & 7) * 8 + srel) * PIECE
                                                                                    : SRCW + (size_t)(v * 128 + (code & 127)) * PIECE), PIECE);
            const unsigned vsX = wid * 4096 + (lane >> 3) * 128 + ((lane & 7) ^ (lane >> 4)) * 16;
            const unsigned vo0 = type ? wid * 2048 + lane * 16 : vsX, vo1 = (type ? vo0 : vsX ^ 64u) + 1024u, s2 = type ? HTB : 2048;
            const int cur = code;
            CFENCE;
#pragma unroll
            for (int c = 0; c < 4; ++c) {
                __builtin_amdgcn_raw_buffer_store_b128(pk[c], rs, ((c & 1) ? vo1 : vo0) + (c >> 1) * s2, 0, AUX_SYS);
                if (direct) WAIT_V(1);
            }
            CFENCE;
            if (n + 1 < njobs) { code = job_code(n + 1); load(code); } else WAIT_V(0);
            WAIT_V(8); BAR; CFENCE;
            if (tid < 8) {
                const bool mine = type ? tid == v : true;
                const bool sflag = type ? staged : tid == rank;
                unsigned* fp = sflag ? (unsigned*)(lws + WS_FX) + (cur >> 3) * 64 + (cur & 7) * 8 + srel
                                     : (unsigned*)(lws + WS_RDY) + rank * 256 + cur;
                if (mine) __hip_atomic_store(fp, call, RLX_SYS);
            }
        }
        STAMP(2);
    }

    f32x4 acc[2][2][4][2];
#pragma unroll
    for (int a = 0; a < 2; ++a)
#pragma unroll
        for (int c = 0; c < 2; ++c)
#pragma unroll
            for (int m = 0; m < 4; ++m)
#pragma unroll
                for (int n = 0; n < 2; ++n) acc[a][c][m][n] = (f32x4){0.f, 0.f, 0.f, 0.f};

    const __amdgpu_buffer_rsrc_t rA = make_rsrc(ws + stage_off + (size_t)pm * PANEL, PANEL);
    const __amdgpu_buffer_rsrc_t rB = make_rsrc(ws + stage_off + (size_t)(16 + pn) * PANEL, PANEL);
    const __amdgpu_buffer_rsrc_t rC = make_rsrc(ws, WS_CTL_BYTES);
    const unsigned snap_src = wid == 0 ? WS_FX + pm * 256 : wid == 1 ? WS_FW + pn * 256 : wid == 2 ? WS_CTRL + b * 256 : wid == 3 ? WS_GLOB : OOB;

#define SA(bb, h) (((bb) * 2 + (h)) * HTB)
#define SB(bb, h) ((4 + (bb) * 2 + (h)) * HTB)
#define STAGE(bufoff, rs, so) do { _Pragma("unroll") for (int _i = 0; _i < 2; ++_i) \
        __builtin_amdgcn_raw_ptr_buffer_load_lds(rs, (LAS void*)(lds + (bufoff) + ldsw + _i * 8192), 16, voff, (so) + _i * 8192, 0, AUX_SYS); } while (0)
#define SNAP() __builtin_amdgcn_raw_ptr_buffer_load_lds(rC, (LAS void*)(lds + LDS_SNAP + wid * 256), 4, lane * 4, snap_src, 0, AUX_SYS)
#define LDA(dst, bb, h) do { _Pragma("unroll") for (int m = 0; m < 4; ++m) _Pragma("unroll") for (int k = 0; k < 2; ++k) \
        dst[m][k] = *(const LAS half8*)(lds + SA(bb, h) + (k ? aoff1 : aoff0) + m * 2048); } while (0)
#define LDB(dst, bb, h) do { _Pragma("unroll") for (int n = 0; n < 2; ++n) _Pragma("unroll") for (int k = 0; k < 2; ++k) \
        dst[n][k] = *(const LAS half8*)(lds + SA(bb, h) + boff + n * 1024 + k * 8192); } while (0)
#define MMA(ai, bj, At, Bt) do { __builtin_amdgcn_s_setprio(1); \
        _Pragma("unroll") for (int m = 0; m < 4; ++m) _Pragma("unroll") for (int n = 0; n < 2; ++n) _Pragma("unroll") for (int k = 0; k < 2; ++k) \
        acc[ai][bj][m][n] = __builtin_amdgcn_mfma_f32_16x16x32_f16(At[m][k], Bt[n][k], acc[ai][bj][m][n], 0, 0, 0); \
        __builtin_amdgcn_s_setprio(0); } while (0)
#define SNAP_ADDR(tt) (LDS_SNAP + (lane < 4 ? (lane >> 1) * 256 + (((tt) + (lane & 1)) & 63) * 4 : lane == 4 ? 512 : 768))

    int t = 0;
    int state = v == rank ? 2 : grp == 0 ? 1 : 0;
    do {
        if (state != 1) {
            const u64 t0 = __builtin_amdgcn_s_memrealtime();
            unsigned bits;
            for (;;) {
                SNAP(); WAIT_V(0); BAR; CFENCE;
                const unsigned sv = *(const LAS unsigned*)(lds + SNAP_ADDR(t));
                bits = (unsigned)__ballot(lane == 5 ? sv == call : sv >= call) & (state == 0 ? 63u : 47u);
                if (bits >= 16u || (t < 64 && bits == 15u)) break;
                if (tid == 0 && __builtin_amdgcn_s_memrealtime() - t0 > DEADLINE) __hip_atomic_store(tmo, call, RLX_AGENT);
                __builtin_amdgcn_s_sleep(6);
                CFENCE; BAR;
            }
            if (bits & 16u) state = 1;
            if (bits & 32u) { state = 3; t = 64; }
        }
        const bool run = state != 1 && t < 64;
        if (state == 1) {
            STAMP(3);
            state = 2;
            int tz = tid; asm volatile("" : "+v"(tz));
            const int ln = tz & 63;
            const int quota = grp == 0 ? QUOTA0 : grp == 1 ? QUOTA1 : grp == 2 ? QUOTA2 : 1000;
            const int srel = (v - rank) & 7;
            const unsigned vo = wid * 4096 + ln * 16;
            unsigned* const qv = (unsigned*)queue + (tz >> 10);
            const unsigned* const rdy = (const unsigned*)(ws + WS_RDY) + v * 256;
            int id0 = 8 + xcd, id1 = 16 + xcd;
            if (grp != 0) {
                BAR;
                if (tz == 0) misc[1] = 24u + __hip_atomic_fetch_add(qv, 2u, RLX_AGENT);
                WAIT_L(0); BAR; CFENCE;
                id0 = __builtin_amdgcn_readfirstlane(misc[1]); id1 = id0 + 1;
            }
            auto code_of = [&](int id) { return (int)((p.order[(id >> 2) & 63] >> ((id & 3) * 8)) & 255u); };
            auto src_of = [&](int code) {
                return make_rsrc(pws + stage_off + ((code >> 7) ? SRCW + (size_t)(rank * 128 + (code & 127)) * PIECE
                                                                : (size_t)(code >> 3) * PANEL + (size_t)((code & 7) * 8) * PIECE), PIECE);
            };
            auto wait_ready = [&](int code, unsigned seen) {
                if (__builtin_amdgcn_readfirstlane(seen) >= call) return;
                const u64 t0 = __builtin_amdgcn_s_memrealtime();
                while (__builtin_amdgcn_readfirstlane(__hip_atomic_load(rdy + code, RLX_SYS)) < call) {
                    if (__builtin_amdgcn_readfirstlane(__hip_atomic_load(tmo, RLX_AGENT)) == call
                        || __builtin_amdgcn_s_memrealtime() - t0 > DEADLINE) {
                        if (ln == 0) __hip_atomic_store(tmo, call, RLX_AGENT);
                        break;
                    }
                    __builtin_amdgcn_s_sleep(2);
                }
            };
            auto publish = [&](int code) {
                __hip_atomic_store((unsigned*)(ws + WS_FX) + (code >> 3) * 64 + (code & 7) * 8 + srel, call, RLX_AGENT);
            };
            unsigned* const next_tok = (unsigned*)(ws + WS_CTRL + ((b + 64) & 255) * 256) + (grp == 3 ? 1 : 0);
            bool passed = false;
            if (id0 < 256) {
                u32x4 d0, d1, d2, d3;
#define FLOAD(d, rs, i) d = __builtin_amdgcn_raw_buffer_load_b128(rs, vo + (i) * 1024, 0, AUX_SYS)
#define FSTORE(d, i) __builtin_amdgcn_raw_buffer_store_b128(d, rD, vo + (i) * 1024, 0, AUX_SYS)
                int c0 = code_of(id0), cp = -1;
                wait_ready(c0, 0u);
                __amdgpu_buffer_rsrc_t rS = src_of(c0);
                FLOAD(d0, rS, 0);
                WAIT_VC(0);
                for (int n = 0;; ++n) {
                    const bool more = id1 < 256;
                    const bool fetch = more && n + 2 < quota;
                    const bool pass = !passed && (grp < 3 ? (!more || n + PASS_AHEAD >= quota) : (OUT_TOK == 2 && !more));
                    const int c1 = code_of(id1);
                    const __amdgpu_buffer_rsrc_t rD = make_rsrc(ws + stage_off + (size_t)(c0 >> 3) * PANEL + (size_t)((c0 & 7) * 8 + srel) * PIECE, PIECE);
                    WAIT_VC(1);
                    const unsigned seen = __hip_atomic_load(rdy + c1, RLX_SYS);
                    unsigned got = 232u;
                    if (tz == 0 && fetch) got = __hip_atomic_fetch_add(qv, 1u, RLX_AGENT);
                    if (tz == 0 && pass) __hip_atomic_store(next_tok, call, RLX_AGENT);
                    passed = passed || pass;
                    CFENCE;
                    FLOAD(d1, rS, 1); CFENCE; FSTORE(d0, 0);
                    WAIT_VC(1);
                    asm volatile("" : "+v"(got));
                    if (tz == 0) misc[2 + (n & 1)] = 24u + got;
                    if (cp >= 0 && ln == 0 && __hip_atomic_fetch_add(misc + 8 + (n & 1), 1u, __ATOMIC_RELAXED, __HIP_MEMORY_SCOPE_WORKGROUP) % 8u == 7u) publish(cp);
                    CFENCE;
                    FLOAD(d2, rS, 2); CFENCE; FSTORE(d1, 1);
                    WAIT_VC(1);
                    FLOAD(d3, rS, 3); CFENCE; FSTORE(d2, 2);
                    WAIT_VC(1);
                    if (more) {
                        wait_ready(c1, seen);
                        rS = src_of(c1);
                        FLOAD(d0, rS, 0);
                    }
                    CFENCE;
                    FSTORE(d3, 3);
                    WAIT_L(0); BAR; CFENCE;
                    if (!more) break;
                    cp = c0; c0 = c1; id1 = __builtin_amdgcn_readfirstlane(misc[2 + (n & 1)]);
                }
                if (!passed) { if (tz == 0) __hip_atomic_store(next_tok, call, RLX_AGENT); passed = true; }
                WAIT_VC(0); BAR; CFENCE;
                if (tz == 64) publish(c0);
            }
            if (!passed && tz == 0) __hip_atomic_store(next_tok, call, RLX_AGENT);
            STAMP(4);
        }
        if (run) {
            const unsigned voff = (unsigned)tid * 16u;
            const unsigned ldsw = (unsigned)wid * 1024u;
            const int aoff0 = (wr * 64 + fr) * 128 + (fq ^ (fr >> 1)) * 16, aoff1 = aoff0 ^ 64;
            int boff = SB(0, 0) + fq * 2048 + (wc * 16 + fr) * 16;
            asm volatile("" : "+v"(boff));
            half8 At[4][2], B0[2][2], B1[2][2];
            unsigned so = (unsigned)t * PIECE;
            STAGE(SB(0, 0), rB, so); STAGE(SA(0, 0), rA, so); STAGE(SB(0, 1), rB, so + HTB); STAGE(SA(0, 1), rA, so + HTB);
            if (wr == 1) BAR;
            WAIT_V(4); BAR;
            STAGE(SB(1, 0), rB, so + PIECE); STAGE(SA(1, 0), rA, so + PIECE); STAGE(SB(1, 1), rB, so + PIECE + HTB);
            WAIT_V(6); BAR;
            bool go;
            do {
                const unsigned sv = *(const LAS unsigned*)(lds + SNAP_ADDR(t + 2));
                SCHED;
                LDB(B0, 0, 0); SCHED; LDA(At, 0, 0); STAGE(SA(1, 1), rA, so + PIECE + HTB);
                WAIT_L(8); BAR; WAIT_L(0); MMA(0, 0, At, B0);
                const unsigned bits = (unsigned)__ballot(sv >= call);
                go = t + 2 < 64 && (bits & (state == 0 ? 31u : 15u)) == 15u;
                const unsigned s2 = go ? so + 2 * PIECE : OOB;
                BAR; SCHED;
                LDB(B1, 0, 1); STAGE(SB(0, 0), rB, s2);
                BAR; WAIT_L(0); MMA(0, 1, At, B1); BAR;
                LDA(At, 0, 1); STAGE(SA(0, 0), rA, s2);
                BAR; WAIT_L(0); MMA(1, 0, At, B0); BAR; SCHED;
                STAGE(SB(0, 1), rB, s2 + HTB);
                if (!go) WAIT_V(0);
                WAIT_V(6); BAR; MMA(1, 1, At, B1); BAR;
                SNAP();
                LDB(B0, 1, 0); SCHED; LDA(At, 1, 0); STAGE(SA(0, 1), rA, s2 + HTB);
                WAIT_L(8); BAR; WAIT_L(0); MMA(0, 0, At, B0); BAR; SCHED;
                LDB(B1, 1, 1); STAGE(SB(1, 0), rB, s2 + PIECE);
                BAR; WAIT_L(0); MMA(0, 1, At, B1); BAR;
                LDA(At, 1, 1); STAGE(SA(1, 0), rA, s2 + PIECE);
                BAR; WAIT_L(0); MMA(1, 0, At, B0); BAR; SCHED;
                STAGE(SB(1, 1), rB, s2 + PIECE + HTB);
                WAIT_V(6); BAR; MMA(1, 1, At, B1); BAR;
                t += 2; so += 2 * PIECE;
            } while (go);
            WAIT_V(0);
            if (wr == 0) BAR;
            BAR;
        }
    } while (t < 64 || state < 2);
    STAMP(6);

    {
        int tz = tid; asm volatile("" : "+v"(tz));
        const int ln = tz & 63, efr = ln & 15, efq = ln >> 4;
        const float qnan = __builtin_nanf("");
        const __amdgpu_buffer_rsrc_t rO = make_rsrc(p.out, (unsigned)MS * GN * 4u);
        const unsigned lds0 = (unsigned)(size_t)lds;
        const unsigned reg0 = (unsigned)wid * CHUNK;
        const bool bad = __builtin_amdgcn_readfirstlane(__hip_atomic_load(tmo, RLX_AGENT)) == call;
        if (v == rank) {
            const unsigned base = (unsigned)(((pm & 1) * 256 + wr * 64 + efq * 4) * GN + rank * NB + pn * 256 + wc * 64 + efr * 4) * 4u;
#pragma unroll
            for (int ai = 0; ai < 2; ++ai)
#pragma unroll
                for (int m = 0; m < 4; ++m)
#pragma unroll
                    for (int r = 0; r < 4; ++r) {
                        f32x4 o = (f32x4){acc[ai][0][m][0][r], acc[ai][0][m][1][r], acc[ai][1][m][0][r], acc[ai][1][m][1][r]};
#pragma unroll
                        for (int e = 0; e < 4; ++e) o[e] = bad ? qnan : (o[e] < 0.f ? 0.f : o[e]);
                        __builtin_amdgcn_raw_buffer_store_b128(__builtin_bit_cast(u32x4, o), rO, base + (unsigned)((ai * 128 + m * 16 + r) * GN) * 4u, 0, AUX_NT);
                    }
        } else {
            unsigned sizes = 0;
#pragma unroll
            for (int ai = 0; ai < 2; ++ai) {
                unsigned mk0 = 0u, mk1 = 0u;
                unsigned pos = lds0 + reg0 + 512u;
#pragma unroll
                for (int m = 0; m < 4; ++m)
#pragma unroll
                    for (int r = 0; r < 4; ++r) {
                        if (m < 2) compact4(acc[ai][0][m][0][r], acc[ai][0][m][1][r], acc[ai][1][m][0][r], acc[ai][1][m][1][r], mk0, pos);
                        else compact4(acc[ai][0][m][0][r], acc[ai][0][m][1][r], acc[ai][1][m][0][r], acc[ai][1][m][1][r], mk1, pos);
                    }
                *(LAS u64*)(lds + reg0 + ln * 8) = ((u64)mk1 << 32) | mk0;
                WAIT_L(0);
                const unsigned bytes = pos - lds0 - reg0;
                sizes |= bytes << (16 * ai);
                const __amdgpu_buffer_rsrc_t rs = make_rsrc(ws + WS_OBOX + (size_t)(v * 512 + grp * 128 + ai * 64 + xcd * 8 + wid) * CHUNK, (bytes + 63u) & ~63u);
                for (unsigned o = 0; o < bytes; o += 1024)
                    __builtin_amdgcn_raw_buffer_store_b128(*(const LAS u32x4*)(lds + reg0 + ln * 16 + o), rs, ln * 16 + o, 0, AUX_SYS);
                WAIT_L(0);
            }
            WAIT_V(0);
            if (ln == 0) __hip_atomic_store((u64*)(pws + WS_FO + (rank * 256 + grp * 64 + xcd * 8 + wid) * 64), ((u64)call << 32) | (bad ? 0xffffffffu : sizes), RLX_SYS);
            STAMP(7);
            if (wid < (grp == 0 ? 8 : grp == 1 ? PULLW1 : grp == 2 ? PULLW2 : PULLW3)) {
                {
                    const unsigned* tok = (const unsigned*)(ws + WS_CTRL + (b & 63) * 256) + 1;
                    const u64 t0 = __builtin_amdgcn_s_memrealtime();
                    while (__builtin_amdgcn_readfirstlane(__hip_atomic_load(tok, RLX_AGENT)) < call) {
                        if (__builtin_amdgcn_readfirstlane(__hip_atomic_load(tmo, RLX_AGENT)) == call
                            || __builtin_amdgcn_s_memrealtime() - t0 > DEADLINE) {
                            if (ln == 0) __hip_atomic_store(tmo, call, RLX_AGENT);
                            break;
                        }
                        __builtin_amdgcn_s_sleep(8);
                    }
                }
                STAMP(5);
                unsigned* const oq = (unsigned*)(queue + 1) + (tz >> 10);
                unsigned i = (unsigned)(xcd * 8 + wid);
                if (grp != 0) {
                    unsigned g = 0;
                    if (ln == 0) g = __hip_atomic_fetch_add(oq, 1u, RLX_AGENT);
                    i = 64u + __builtin_amdgcn_readfirstlane(g);
                }
                while (i < 512u) {
                    unsigned nxt = 0;
                    if (ln == 0) nxt = __hip_atomic_fetch_add(oq, 1u, RLX_AGENT);
                    const unsigned g2 = i >> 7, ai = (i >> 6) & 1u, x2 = (i >> 3) & 7u, w2 = i & 7u;
                    const u64* fo = (const u64*)(ws + WS_FO + (v * 256 + g2 * 64 + (i & 63u)) * 64);
                    const u64 t0 = __builtin_amdgcn_s_memrealtime();
                    unsigned bytes;
                    for (;;) {
                        const u64 fv = __hip_atomic_load(fo, RLX_SYS);
                        bytes = __builtin_amdgcn_readfirstlane((unsigned)fv);
                        if (__builtin_amdgcn_readfirstlane((unsigned)(fv >> 32)) >= call) break;
                        if (__builtin_amdgcn_readfirstlane(__hip_atomic_load(tmo, RLX_AGENT)) == call
                            || __builtin_amdgcn_s_memrealtime() - t0 > DEADLINE) {
                            if (ln == 0) __hip_atomic_store(tmo, call, RLX_AGENT);
                            bytes = 0xffffffffu;
                            break;
                        }
                        __builtin_amdgcn_s_sleep(4);
                    }
                    bytes = (bytes >> (16u * ai)) & 0xffffu;
                    const bool nodata = bytes > 512u + 16384u;
                    if (nodata) bytes = 0u;
                    const __amdgpu_buffer_rsrc_t rs = make_rsrc(pws + WS_OBOX + (size_t)(rank * 512 + i) * CHUNK, bytes);
                    const bool wide = (i < 64u && (i & 1u) == 0u) || i >= 512u - PULL_W1;
                    for (unsigned o = 0; o < bytes; o += 1024) {
                        if (wide) WAIT_V(1); else WAIT_V(0);
                        __builtin_amdgcn_raw_ptr_buffer_load_lds(rs, (LAS void*)(lds + reg0 + o), 16, ln * 16, o, 0, AUX_SYS);
                    }
                    WAIT_V(0);
                    const u64 mm = *(const LAS u64*)(lds + reg0 + ln * 8);
                    unsigned mk = nodata ? 0u : (unsigned)mm;
                    unsigned pos = lds0 + reg0 + 512u;
                    unsigned off = (unsigned)(((x2 & 1u) * 256u + ai * 128u + (w2 >> 2) * 64u + efq * 4) * GN + v * NB + (4u * g2 + (x2 >> 1)) * 256u + (w2 & 3u) * 64u + efr * 4) * 4u;
                    const float fill = nodata ? qnan : 0.f;
                    for (int m = 0; m < 4; ++m) {
                        f32x4 o[4];
                        expand16(o, mk, pos, fill);
#pragma unroll
                        for (int r = 0; r < 4; ++r)
                            __builtin_amdgcn_raw_buffer_store_b128(__builtin_bit_cast(u32x4, o[r]), rO, off + (unsigned)(r * GN) * 4u, 0, AUX_NT);
                        off += 16u * GN * 4u;
                        if (m == 1) mk = nodata ? 0u : (unsigned)(mm >> 32);
                    }
                    i = 64u + __builtin_amdgcn_readfirstlane(nxt);
                }
            }
        }
        STAMP(9);
    }
}

}

extern "C" void kernel_launch(int rank, int nranks, ncclComm_t comm,
                              hipStream_t stream,
                              void* const* d_in, const int* in_sizes, int n_in,
                              void* d_out, int out_size,
                              const int* in_dtypes, int out_dtype,
                              void* const* d_out_all,
                              void* const* d_ws_all, size_t ws_bytes) {
    if (nranks != NR || n_in < 2 || (unsigned)rank >= (unsigned)NR || ws_bytes < WS_NEED ||
        in_sizes[0] != GM * KS || in_sizes[1] != KS * GN || out_size != MS * GN) {
        fprintf(stderr, "kernel_launch: unexpected problem shape\n");
        return;
    }
    Params p;
    p.x = (const float*)d_in[0];
    p.w = (const float*)d_in[1];
    p.out = (float*)d_out;
    for (int r = 0; r < NR; ++r) p.ws_all[r] = (char*)d_ws_all[r];
    make_order(p.order, p.worder);
    p.rank = rank;
    hipLaunchKernelGGL(fused_gemm_rs, dim3(NWG), dim3(512), 0, stream, p);
    HIP_CHECK(hipGetLastError());
}
```

```cpp
#include <hip/hip_runtime.h>
#include <rccl/rccl.h>
#include <cstdio>

#define HIP_CHECK(expr) do { hipError_t _e = (expr); if (_e != hipSuccess) { \
    fprintf(stderr, "HIP error %s at %s:%d\n", hipGetErrorString(_e), __FILE__, __LINE__); \
    return; } } while (0)

namespace {

typedef _Float16 half8 __attribute__((ext_vector_type(8)));
typedef _Float16 half2v __attribute__((ext_vector_type(2)));
typedef float f32x4 __attribute__((ext_vector_type(4)));
typedef unsigned u32x4 __attribute__((ext_vector_type(4)));
typedef unsigned long long u64;
#define LAS __attribute__((address_space(3)))

constexpr int NR = 8;
constexpr int GM = 4096, GK = 4096, GN = 32768;
constexpr int KS = GK / NR;
constexpr int NB = GN / NR;
constexpr int MS = GM / NR;
constexpr int NWG = 256;
constexpr unsigned PIECE = 32768;
constexpr unsigned HTB = 16384;
constexpr unsigned PANEL = 64 * PIECE;
constexpr unsigned CHUNK = 17408;
constexpr unsigned WS_GLOB = 0;
constexpr unsigned WS_Q = 1024;
constexpr unsigned WS_FX = 8192;
constexpr unsigned WS_FW = WS_FX + 4096;
constexpr unsigned WS_RDY = 16384;
constexpr unsigned WS_CTRL = 65536;
constexpr unsigned WS_STAMP = 131072;
constexpr unsigned WS_FO = 262144;
constexpr unsigned WS_CTL_BYTES = 1u << 20;
constexpr size_t WS_STAGE = 1u << 20;
constexpr size_t STAGE_PAR = 48ull * PANEL;
constexpr size_t SRCW = 32ull * PANEL;
constexpr size_t WS_OBOX = WS_STAGE + 2 * STAGE_PAR;
constexpr size_t WS_NEED = WS_OBOX + 4096ull * CHUNK;
constexpr u64 DEADLINE = 50000000ull;
constexpr int AUX_SYS = 17;
constexpr unsigned LDS_SNAP = 8 * CHUNK + 1024;
constexpr unsigned LDS_MISC = LDS_SNAP + 2048;
constexpr unsigned LDS_BYTES = LDS_MISC + 128;
static_assert(LDS_BYTES <= 163840 && CHUNK >= HTB, "LDS layout");
constexpr unsigned OOB = 0x40000000u;

#define QUOTA0 8
#define QUOTA1 8
#define QUOTA2 8
#define PASS_AHEAD 2
#define DEF1 6
#define DEF2 4
#define DEF3 3
#define PP 0
#define CONV_SLEEP 0
#define DIRECT_WIN 2
#define FIRST_PRIO 0
#define NDIRECT 8
#define AUX_SRC 2
#define AUX_OUT 17
#define AUX_CST 17
#define AUX_FLD 17
#define AUX_FST 17
#define AUX_OST 17
#define AUX_OLD 17
#define GO_AT 3
#define OGENS 1
#define OWIN0 7
#define OWIN1 0

struct Params {
    const float* x;
    const float* w;
    char* ws;
    int rank;
    int pad;
    float* out;
    char* ws_all[NR];
    unsigned order[64];
    unsigned worder[32];
};

static void make_order(unsigned* order, unsigned* worder) {
    const int def[4] = {8, DEF1, DEF2, DEF3};
    unsigned char q[256], wq[128]; int n = 0, m = 0;
    for (int j = 0; j < 8; ++j) {
        for (int pn = 0; pn < 16; ++pn) if (j < def[pn >> 2]) q[n++] = (unsigned char)(128 | pn << 3 | j);
        for (int pm = 0; pm < 16; ++pm) q[n++] = (unsigned char)(pm << 3 | j);
    }
    for (int g = 1; g < 4; ++g)
        for (int j = def[g]; j < 8; ++j)
            for (int pn = 4 * g; pn < 4 * g + 4; ++pn) q[n++] = (unsigned char)(128 | pn << 3 | j);
    for (int i = 0; i < 256; ++i) if (q[i] & 128) wq[m++] = q[i];
    for (int i = 0; i < 64; ++i) order[i] = q[4 * i] | q[4 * i + 1] << 8 | q[4 * i + 2] << 16 | (unsigned)q[4 * i + 3] << 24;
    for (int i = 0; i < 32; ++i) worder[i] = wq[4 * i] | wq[4 * i + 1] << 8 | wq[4 * i + 2] << 16 | (unsigned)wq[4 * i + 3] << 24;
}

__device__ __forceinline__ unsigned pack2(float a, float b) {
    half2v h; h.x = (_Float16)a; h.y = (_Float16)b;
    return __builtin_bit_cast(unsigned, h);
}
__device__ __forceinline__ __amdgpu_buffer_rsrc_t make_rsrc(const void* p, unsigned bytes) {
    const u64 v = (u64)p;
    const unsigned lo = __builtin_amdgcn_readfirstlane((unsigned)v), hi = __builtin_amdgcn_readfirstlane((unsigned)(v >> 32));
    return __builtin_amdgcn_make_buffer_rsrc((void*)(((u64)hi << 32) | lo), 0, bytes, 0x00020000);
}

__device__ __forceinline__ unsigned lds_ld(unsigned addr) {
    unsigned r; asm volatile("ds_read_b32 %0, %1\n\ts_waitcnt lgkmcnt(0)" : "=v"(r) : "v"(addr) : "memory"); return r;
}
__device__ __forceinline__ void lds_st(unsigned addr, unsigned val) { asm volatile("ds_write_b32 %0, %1" :: "v"(addr), "v"(val) : "memory"); }

__device__ __forceinline__ void compact4(float x0, float x1, float x2, float x3, unsigned& mk, unsigned& pos) {
    unsigned a0, a1, a2, a3, c, p1, p2, p3;
    asm volatile(
        "v_cmp_nle_f32 s[88:89], %10, 0\n\t"
        "v_cmp_nle_f32 s[90:91], %11, 0\n\t"
        "v_cmp_nle_f32 s[92:93], %12, 0\n\t"
        "v_cmp_nle_f32 s[94:95], %13, 0\n\t"
        "s_bcnt1_i32_b64 %4, s[88:89]\n\t"
        "v_mbcnt_lo_u32_b32 %0, s88, 0\n\t"
        "s_lshl2_add_u32 %5, %4, %9\n\t"
        "v_mbcnt_lo_u32_b32 %1, s90, 0\n\t"
        "s_bcnt1_i32_b64 %4, s[90:91]\n\t"
        "v_mbcnt_lo_u32_b32 %2, s92, 0\n\t"
        "s_lshl2_add_u32 %6, %4, %5\n\t"
        "v_mbcnt_lo_u32_b32 %3, s94, 0\n\t"
        "s_bcnt1_i32_b64 %4, s[92:93]\n\t"
        "v_mbcnt_hi_u32_b32 %0, s89, %0\n\t"
        "s_lshl2_add_u32 %7, %4, %6\n\t"
        "v_mbcnt_hi_u32_b32 %1, s91, %1\n\t"
        "s_bcnt1_i32_b64 %4, s[94:95]\n\t"
        "v_mbcnt_hi_u32_b32 %2, s93, %2\n\t"
        "v_mbcnt_hi_u32_b32 %3, s95, %3\n\t"
        "v_lshl_add_u32 %0, %0, 2, %9\n\t"
        "v_lshl_add_u32 %1, %1, 2, %5\n\t"
        "v_lshl_add_u32 %2, %2, 2, %6\n\t"
        "v_lshl_add_u32 %3, %3, 2, %7\n\t"
        "s_lshl2_add_u32 %9, %4, %7\n\t"
        "v_addc_co_u32 %8, vcc, %8, %8, s[88:89]\n\t"
        "v_addc_co_u32 %8, vcc, %8, %8, s[90:91]\n\t"
        "v_addc_co_u32 %8, vcc, %8, %8, s[92:93]\n\t"
        "v_addc_co_u32 %8, vcc, %8, %8, s[94:95]\n\t"
        "s_mov_b64 exec, s[88:89]\n\t"
        "ds_write_b32 %0, %10\n\t"
        "s_mov_b64 exec, s[90:91]\n\t"
        "ds_write_b32 %1, %11\n\t"
        "s_mov_b64 exec, s[92:93]\n\t"
        "ds_write_b32 %2, %12\n\t"
        "s_mov_b64 exec, s[94:95]\n\t"
        "ds_write_b32 %3, %13\n\t"
        "s_mov_b64 exec, -1"
        : "=&v"(a0), "=&v"(a1), "=&v"(a2), "=&v"(a3), "=&s"(c), "=&s"(p1), "=&s"(p2), "=&s"(p3), "+v"(mk), "+s"(pos)
        : "v"(x0), "v"(x1), "v"(x2), "v"(x3)
        : "vcc", "scc", "memory", "s88", "s89", "s90", "s91", "s92", "s93", "s94", "s95");
}

__device__ __forceinline__ void expand16(f32x4 (&o)[4], unsigned& mk, unsigned& pos, float fill) {
    float x[16]; unsigned a0, a1, a2, a3, c, p1, p2, p3;
    asm volatile(
        "v_add_co_u32 %24, s[88:89], %24, %24\n\t"
        "v_add_co_u32 %24, s[90:91], %24, %24\n\t"
        "v_add_co_u32 %24, s[92:93], %24, %24\n\t"
        "v_add_co_u32 %24, s[94:95], %24, %24\n\t"
        "v_mov_b32 %0, %26\n\t"
        "v_mov_b32 %1, %26\n\t"
        "s_bcnt1_i32_b64 %20, s[88:89]\n\t"
        "v_mbcnt_lo_u32_b32 %16, s88, 0\n\t"
        "s_lshl2_add_u32 %21, %20, %25\n\t"
        "v_mbcnt_lo_u32_b32 %17, s90, 0\n\t"
        "s_bcnt1_i32_b64 %20, s[90:91]\n\t"
        "v_mbcnt_lo_u32_b32 %18, s92, 0\n\t"
        "s_lshl2_add_u32 %22, %20, %21\n\t"
        "v_mbcnt_lo_u32_b32 %19, s94, 0\n\t"
        "s_bcnt1_i32_b64 %20, s[92:93]\n\t"
        "v_mbcnt_hi_u32_b32 %16, s89, %16\n\t"
        "s_lshl2_add_u32 %23, %20, %22\n\t"
        "v_mbcnt_hi_u32_b32 %17, s91, %17\n\t"
        "s_bcnt1_i32_b64 %20, s[94:95]\n\t"
        "v_mbcnt_hi_u32_b32 %18, s93, %18\n\t"
        "v_mbcnt_hi_u32_b32 %19, s95, %19\n\t"
        "v_lshl_add_u32 %16, %16, 2, %25\n\t"
        "v_lshl_add_u32 %17, %17, 2, %21\n\t"
        "v_lshl_add_u32 %18, %18, 2, %22\n\t"
        "v_lshl_add_u32 %19, %19, 2, %23\n\t"
        "s_lshl2_add_u32 %25, %20, %23\n\t"
        "v_mov_b32 %2, %26\n\t"
        "v_mov_b32 %3, %26\n\t"
        "s_mov_b64 exec, s[88:89]\n\t"
        "ds_read_b32 %0, %16\n\t"
        "s_mov_b64 exec, s[90:91]\n\t"
        "ds_read_b32 %1, %17\n\t"
        "s_mov_b64 exec, s[92:93]\n\t"
        "ds_read_b32 %2, %18\n\t"
        "s_mov_b64 exec, s[94:95]\n\t"
        "ds_read_b32 %3, %19\n\t"
        "s_mov_b64 exec, -1\n\t"
        "v_add_co_u32 %24, s[88:89], %24, %24\n\t"
        "v_add_co_u32 %24, s[90:91], %24, %24\n\t"
        "v_add_co_u32 %24, s[92:93], %24, %24\n\t"
        "v_add_co_u32 %24, s[94:95], %24, %24\n\t"
        "v_mov_b32 %4, %26\n\t"
        "v_mov_b32 %5, %26\n\t"
        "s_bcnt1_i32_b64 %20, s[88:89]\n\t"
        "v_mbcnt_lo_u32_b32 %16, s88, 0\n\t"
        "s_lshl2_add_u32 %21, %20, %25\n\t"
        "v_mbcnt_lo_u32_b32 %17, s90, 0\n\t"
        "s_bcnt1_i32_b64 %20, s[90:91]\n\t"
        "v_mbcnt_lo_u32_b32 %18, s92, 0\n\t"
        "s_lshl2_add_u32 %22, %20, %21\n\t"
        "v_mbcnt_lo_u32_b32 %19, s94, 0\n\t"
        "s_bcnt1_i32_b64 %20, s[92:93]\n\t"
        "v_mbcnt_hi_u32_b32 %16, s89, %16\n\t"
        "s_lshl2_add_u32 %23, %20, %22\n\t"
        "v_mbcnt_hi_u32_b32 %17, s91, %17\n\t"
        "s_bcnt1_i32_b64 %20, s[94:95]\n\t"
        "v_mbcnt_hi_u32_b32 %18, s93, %18\n\t"
        "v_mbcnt_hi_u32_b32 %19, s95, %19\n\t"
        "v_lshl_add_u32 %16, %16, 2, %25\n\t"
        "v_lshl_add_u32 %17, %17, 2, %21\n\t"
        "v_lshl_add_u32 %18, %18, 2, %22\n\t"
        "v_lshl_add_u32 %19, %19, 2, %23\n\t"
        "s_lshl2_add_u32 %25, %20, %23\n\t"
        "v_mov_b32 %6, %26\n\t"
        "v_mov_b32 %7, %26\n\t"
        "s_mov_b64 exec, s[88:89]\n\t"
        "ds_read_b32 %4, %16\n\t"
        "s_mov_b64 exec, s[90:91]\n\t"
        "ds_read_b32 %5, %17\n\t"
        "s_mov_b64 exec, s[92:93]\n\t"
        "ds_read_b32 %6, %18\n\t"
        "s_mov_b64 exec, s[94:95]\n\t"
        "ds_read_b32 %7, %19\n\t"
        "s_mov_b64 exec, -1\n\t"
        "v_add_co_u32 %24, s[88:89], %24, %24\n\t"
        "v_add_co_u32 %24, s[90:91], %24, %24\n\t"
        "v_add_co_u32 %24, s[92:93], %24, %24\n\t"
        "v_add_co_u32 %24, s[94:95], %24, %24\n\t"
        "v_mov_b32 %8, %26\n\t"
        "v_mov_b32 %9, %26\n\t"
        "s_bcnt1_i32_b64 %20, s[88:89]\n\t"
        "v_mbcnt_lo_u32_b32 %16, s88, 0\n\t"
        "s_lshl2_add_u32 %21, %20, %25\n\t"
        "v_mbcnt_lo_u32_b32 %17, s90, 0\n\t"
        "s_bcnt1_i32_b64 %20, s[90:91]\n\t"
        "v_mbcnt_lo_u32_b32 %18, s92, 0\n\t"
        "s_lshl2_add_u32 %22, %20, %21\n\t"
        "v_mbcnt_lo_u32_b32 %19, s94, 0\n\t"
        "s_bcnt1_i32_b64 %20, s[92:93]\n\t"
        "v_mbcnt_hi_u32_b32 %16, s89, %16\n\t"
        "s_lshl2_add_u32 %23, %20, %22\n\t"
        "v_mbcnt_hi_u32_b32 %17, s91, %17\n\t"
        "s_bcnt1_i32_b64 %20, s[94:95]\n\t"
        "v_mbcnt_hi_u32_b32 %18, s93, %18\n\t"
        "v_mbcnt_hi_u32_b32 %19, s95, %19\n\t"
        "v_lshl_add_u32 %16, %16, 2, %25\n\t"
        "v_lshl_add_u32 %17, %17, 2, %21\n\t"
        "v_lshl_add_u32 %18, %18, 2, %22\n\t"
        "v_lshl_add_u32 %19, %19, 2, %23\n\t"
        "s_lshl2_add_u32 %25, %20, %23\n\t"
        "v_mov_b32 %10, %26\n\t"
        "v_mov_b32 %11, %26\n\t"
        "s_mov_b64 exec, s[88:89]\n\t"
        "ds_read_b32 %8, %16\n\t"
        "s_mov_b64 exec, s[90:91]\n\t"
        "ds_read_b32 %9, %17\n\t"
        "s_mov_b64 exec, s[92:93]\n\t"
        "ds_read_b32 %10, %18\n\t"
        "s_mov_b64 exec, s[94:95]\n\t"
        "ds_read_b32 %11, %19\n\t"
        "s_mov_b64 exec, -1\n\t"
        "v_add_co_u32 %24, s[88:89], %24, %24\n\t"
        "v_add_co_u32 %24, s[90:91], %24, %24\n\t"
        "v_add_co_u32 %24, s[92:93], %24, %24\n\t"
        "v_add_co_u32 %24, s[94:95], %24, %24\n\t"
        "v_mov_b32 %12, %26\n\t"
        "v_mov_b32 %13, %26\n\t"
        "s_bcnt1_i32_b64 %20, s[88:89]\n\t"
        "v_mbcnt_lo_u32_b32 %16, s88, 0\n\t"
        "s_lshl2_add_u32 %21, %20, %25\n\t"
        "v_mbcnt_lo_u32_b32 %17, s90, 0\n\t"
        "s_bcnt1_i32_b64 %20, s[90:91]\n\t"
        "v_mbcnt_lo_u32_b32 %18, s92, 0\n\t"
        "s_lshl2_add_u32 %22, %20, %21\n\t"
        "v_mbcnt_lo_u32_b32 %19, s94, 0\n\t"
        "s_bcnt1_i32_b64 %20, s[92:93]\n\t"
        "v_mbcnt_hi_u32_b32 %16, s89, %16\n\t"
        "s_lshl2_add_u32 %23, %20, %22\n\t"
        "v_mbcnt_hi_u32_b32 %17, s91, %17\n\t"
        "s_bcnt1_i32_b64 %20, s[94:95]\n\t"
        "v_mbcnt_hi_u32_b32 %18, s93, %18\n\t"
        "v_mbcnt_hi_u32_b32 %19, s95, %19\n\t"
        "v_lshl_add_u32 %16, %16, 2, %25\n\t"
        "v_lshl_add_u32 %17, %17, 2, %21\n\t"
        "v_lshl_add_u32 %18, %18, 2, %22\n\t"
        "v_lshl_add_u32 %19, %19, 2, %23\n\t"
        "s_lshl2_add_u32 %25, %20, %23\n\t"
        "v_mov_b32 %14, %26\n\t"
        "v_mov_b32 %15, %26\n\t"
        "s_mov_b64 exec, s[88:89]\n\t"
        "ds_read_b32 %12, %16\n\t"
        "s_mov_b64 exec, s[90:91]\n\t"
        "ds_read_b32 %13, %17\n\t"
        "s_mov_b64 exec, s[92:93]\n\t"
        "ds_read_b32 %14, %18\n\t"
        "s_mov_b64 exec, s[94:95]\n\t"
        "ds_read_b32 %15, %19\n\t"
        "s_mov_b64 exec, -1\n\t"
        "s_waitcnt lgkmcnt(0)"
        : "=&v"(x[0]), "=&v"(x[1]), "=&v"(x[2]), "=&v"(x[3]), "=&v"(x[4]), "=&v"(x[5]), "=&v"(x[6]), "=&v"(x[7]), "=&v"(x[8]), "=&v"(x[9]), "=&v"(x[10]), "=&v"(x[11]), "=&v"(x[12]), "=&v"(x[13]), "=&v"(x[14]), "=&v"(x[15]),
          "=&v"(a0), "=&v"(a1), "=&v"(a2), "=&v"(a3), "=&s"(c), "=&s"(p1), "=&s"(p2), "=&s"(p3), "+v"(mk), "+s"(pos)
        : "v"(fill)
        : "scc", "memory", "s88", "s89", "s90", "s91", "s92", "s93", "s94", "s95");
#pragma unroll
    for (int r = 0; r < 4; ++r) o[r] = (f32x4){x[4 * r], x[4 * r + 1], x[4 * r + 2], x[4 * r + 3]};
}

__global__ void __launch_bounds__(512, 2) fused_gemm_rs(const Params p) {
    __shared__ __attribute__((aligned(16))) unsigned char smem[LDS_BYTES];
    LAS unsigned char* lds = (LAS unsigned char*)smem;
    LAS unsigned* misc = (LAS unsigned*)(lds + LDS_MISC);

    const int tid = threadIdx.x, lane = tid & 63;
    const int wid = __builtin_amdgcn_readfirstlane(tid >> 6);
    const int wr = wid >> 2, wc = wid & 3, fr = lane & 15, fq = lane >> 4;
    const int b = blockIdx.x, xcd = b & 7, grp = b >> 6, v = (b >> 3) & 7;
    const int pm = 2 * v + (xcd & 1), pn = 4 * grp + (xcd >> 1);
    const int rank = p.rank;
    char* const ws = p.ws;
    char* const pws = p.ws_all[v];
    unsigned* const tmo = (unsigned*)(ws + WS_GLOB);
    u64* const queue = (u64*)(ws + WS_Q + v * 128);

#define STAMP(i) do { } while (0)
#define WAIT_V(n) asm volatile("s_waitcnt vmcnt(" #n ")" ::: "memory")
#define WAIT_VC(n) do { __builtin_amdgcn_s_waitcnt(0x0F70 | (n)); WAIT_V(n); } while (0)
#define WAIT_VX(n) WAIT_V(n)
#define WAIT_L(n) asm volatile("s_waitcnt lgkmcnt(" #n ")" ::: "memory")
#define BAR __builtin_amdgcn_s_barrier()
#define SCHED __builtin_amdgcn_sched_barrier(0)
#define CFENCE asm volatile("" ::: "memory")
#define RLX_AGENT __ATOMIC_RELAXED, __HIP_MEMORY_SCOPE_AGENT
#define RLX_SYS __ATOMIC_RELAXED, __HIP_MEMORY_SCOPE_SYSTEM

    unsigned call;
    size_t stage_off;
    {
        const int njobs = grp == 0 ? 0 : 6;
        auto job_code = [&](int n) {
            if (grp == 3 && n == 0) { const int u = v * 8 + xcd; return (u & 15) << 3 | (u >> 4); }
            if (grp == 2 && n == 5) { const int u = 64 + v * 8 + xcd; return (u & 15) << 3 | (u >> 4); }
            const int i = xcd + (grp - 1) * 8 + 24 * (grp == 3 ? n - 1 : n);
            return (int)((p.worder[(i >> 2) & 31] >> ((i & 3) * 8)) & 255u);
        };
        f32x4 f[8];
        auto load = [&](int code) {
            const int type = code >> 7, panel = (code >> 3) & 15, j = code & 7;
            const float* src = type ? p.w + (size_t)(j * 64 + wid * 8) * GN + v * NB + panel * 256
                                    : p.x + (size_t)(panel * 256 + wid * 32) * KS + j * 64;
            const __amdgpu_buffer_rsrc_t rs = make_rsrc(src, 8u * GN * 4u);
            const unsigned vo = type ? lane * 16 : (lane >> 3) * 2048 + (lane & 7) * 32;
            const unsigned s1 = type ? GN * 4 : 16, s2 = type ? 2 * GN * 4 : 16384;
#pragma unroll
            for (int i = 0; i < 8; ++i)
                f[i] = __builtin_bit_cast(f32x4, __builtin_amdgcn_raw_buffer_load_b128(rs, vo, (i >> 1) * s2 + (i & 1) * s1, AUX_SRC));
        };
        int code = grp == 3 ? ((v * 8 + xcd) & 15) << 3 | (v >> 1) : 128 | (((grp - 1) * 8 + xcd) & 15) << 3;
        if (FIRST_PRIO >= 2 && grp == 3 && v >= 2) __builtin_amdgcn_s_sleep(100);
        if (grp != 0) load(code);
        char* const lws = p.ws_all[tid & 7];
        CFENCE;
        if (tid >= 72 && tid < 96) misc[tid - 64] = 0u;
        if (tid == 0) {
            unsigned* cp = (unsigned*)(ws + WS_CTRL + b * 256) + 32;
            const unsigned c = __hip_atomic_load(cp, RLX_AGENT) + 1u;
            __hip_atomic_store(cp, c, RLX_AGENT);
            __hip_atomic_fetch_max(queue, (u64)c << 32, RLX_AGENT);
            __hip_atomic_fetch_max(queue + 1, (u64)c << 32, RLX_AGENT);
            misc[0] = c;
        }
        WAIT_L(0); BAR; CFENCE;
        call = __builtin_amdgcn_readfirstlane(misc[0]);
        stage_off = WS_STAGE + (size_t)(call & 1u) * STAGE_PAR;
        STAMP(0);
#pragma unroll 1
        for (int n = 0; n < njobs; ++n) {
            const int type = code >> 7;
            u32x4 pk[4];
            if (type) {
#pragma unroll
                for (int e = 0; e < 4; ++e)
                    pk[e] = (u32x4){pack2(f[0][e], f[1][e]), pack2(f[2][e], f[3][e]), pack2(f[4][e], f[5][e]), pack2(f[6][e], f[7][e])};
            } else {
#pragma unroll
                for (int c = 0; c < 4; ++c)
                    pk[c] = (u32x4){pack2(f[2 * c][0], f[2 * c][1]), pack2(f[2 * c][2], f[2 * c][3]), pack2(f[2 * c + 1][0], f[2 * c + 1][1]), pack2(f[2 * c + 1][2], f[2 * c + 1][3])};
            }
            const bool direct = n == 0 && grp * 8 <= NDIRECT;
            const bool staged = !type || v == rank || direct;
            const int srel = direct ? (rank - v) & 7 : 0;
            char* const dbase = direct ? pws : ws;
            const __amdgpu_buffer_rsrc_t rs = make_rsrc(dbase + stage_off + (staged ? (size_t)(code >> 3) * PANEL + (size_t)((code & 7) * 8 + srel) * PIECE
                                                                                    : SRCW + (size_t)(v * 128 + (code & 127)) * PIECE), PIECE);
            const unsigned vsX = wid * 4096 + (lane >> 3) * 128 + ((lane & 7) ^ (lane >> 4)) * 16;
            const unsigned vo0 = type ? wid * 2048 + lane * 16 : vsX, vo1 = (type ? vo0 : vsX ^ 64u) + 1024u, s2 = type ? HTB : 2048;
            const int cur = code;
            CFENCE;
#pragma unroll
            for (int c = 0; c < 4; ++c) {
                __builtin_amdgcn_raw_buffer_store_b128(pk[c], rs, ((c & 1) ? vo1 : vo0) + (c >> 1) * s2, 0, AUX_CST);
                if (direct) WAIT_V(1);
            }
            CFENCE;
            if (FIRST_PRIO && n == 0) WAIT_V(0);
            if (n + 1 < njobs) { code = job_code(n + 1); load(code); } else WAIT_V(0);
            WAIT_V(8); BAR; CFENCE;
            if (tid < 8) {
                const bool mine = type ? tid == v : true;
                const bool sflag = type ? staged : tid == rank;
                unsigned* fp = sflag ? (unsigned*)(lws + WS_FX) + (cur >> 3) * 64 + (cur & 7) * 8 + srel
                                     : (unsigned*)(lws + WS_RDY) + rank * 256 + cur;
                if (mine) __hip_atomic_store(fp, call, RLX_SYS);
            }
        }
        STAMP(2);
    }

    f32x4 acc[2][2][4][2];
#pragma unroll
    for (int a = 0; a < 2; ++a)
#pragma unroll
        for (int c = 0; c < 2; ++c)
#pragma unroll
            for (int m = 0; m < 4; ++m)
#pragma unroll
                for (int n = 0; n < 2; ++n) acc[a][c][m][n] = (f32x4){0.f, 0.f, 0.f, 0.f};

    const __amdgpu_buffer_rsrc_t rA = make_rsrc(ws + stage_off + (size_t)pm * PANEL, PANEL);
    const __amdgpu_buffer_rsrc_t rB = make_rsrc(ws + stage_off + (size_t)(16 + pn) * PANEL, PANEL);
    const __amdgpu_buffer_rsrc_t rC = make_rsrc(ws, WS_CTL_BYTES);
    const unsigned snap_src = wid == 0 ? WS_FX + pm * 256 : wid == 1 ? WS_FW + pn * 256 : wid == 2 ? WS_CTRL + b * 256 : wid == 3 ? WS_GLOB : OOB;

#define SA(bb, h) (((bb) * 2 + (h)) * HTB)
#define SB(bb, h) ((4 + (bb) * 2 + (h)) * HTB)
#define STAGE(bufoff, rs, so) do { _Pragma("unroll") for (int _i = 0; _i < 2; ++_i) \
        __builtin_amdgcn_raw_ptr_buffer_load_lds(rs, (LAS void*)(lds + (bufoff) + ldsw + _i * 8192), 16, voff, (so) + _i * 8192, 0, AUX_SYS); } while (0)
#define SNAP() __builtin_amdgcn_raw_ptr_buffer_load_lds(rC, (LAS void*)(lds + LDS_SNAP + wid * 256), 4, lane * 4, snap_src, 0, AUX_SYS)
#define LDA(dst, bb, h) do { _Pragma("unroll") for (int m = 0; m < 4; ++m) _Pragma("unroll") for (int k = 0; k < 2; ++k) \
        dst[m][k] = *(const LAS half8*)(lds + SA(bb, h) + (k ? aoff1 : aoff0) + m * 2048); } while (0)
#define LDB(dst, bb, h) do { _Pragma("unroll") for (int n = 0; n < 2; ++n) _Pragma("unroll") for (int k = 0; k < 2; ++k) \
        dst[n][k] = *(const LAS half8*)(lds + SA(bb, h) + boff + n * 1024 + k * 8192); } while (0)
#define MMA(ai, bj, At, Bt) do { __builtin_amdgcn_s_setprio(1); \
        _Pragma("unroll") for (int m = 0; m < 4; ++m) _Pragma("unroll") for (int n = 0; n < 2; ++n) _Pragma("unroll") for (int k = 0; k < 2; ++k) \
        acc[ai][bj][m][n] = __builtin_amdgcn_mfma_f32_16x16x32_f16(At[m][k], Bt[n][k], acc[ai][bj][m][n], 0, 0, 0); \
        __builtin_amdgcn_s_setprio(0); } while (0)
#define SNAP_ADDR(tt) (LDS_SNAP + (lane < 4 ? (lane >> 1) * 256 + (((tt) + (lane & 1)) & 63) * 4 : lane == 4 ? 512 : 768))

    int t = 0;
    int state = v == rank ? 2 : grp == 0 ? 1 : 0;
    do {
        if (state != 1) {
            const u64 t0 = __builtin_amdgcn_s_memrealtime();
            unsigned bits;
            for (;;) {
                SNAP(); WAIT_V(0); BAR; CFENCE;
                const unsigned sv = *(const LAS unsigned*)(lds + SNAP_ADDR(t));
                bits = (unsigned)__ballot(lane == 5 ? sv == call : sv >= call) & (state == 0 ? 63u : 47u);
                if (bits >= 16u || (t < 64 && bits == 15u)) break;
                if (tid == 0 && __builtin_amdgcn_s_memrealtime() - t0 > DEADLINE) __hip_atomic_store(tmo, call, RLX_AGENT);
                __builtin_amdgcn_s_sleep(6);
                CFENCE; BAR;
            }
            if (bits & 16u) state = 1;
            if (bits & 32u) { state = 3; t = 64; }
        }
        const bool run = state != 1 && t < 64;
        if (state == 1) {
            STAMP(3);
            state = 2;
            int tz = tid; asm volatile("" : "+v"(tz));
            const int ln = tz & 63;
            const int quota = grp == 0 ? QUOTA0 : grp == 1 ? QUOTA1 : grp == 2 ? QUOTA2 : 1000;
            const int srel = (v - rank) & 7;
            const unsigned vo = wid * 4096 + ln * 16;
            unsigned* const qv = (unsigned*)queue + (tz >> 10);
            const unsigned* const rdy = (const unsigned*)(ws + WS_RDY) + v * 256;
            int id0 = NDIRECT + xcd, id1 = NDIRECT + 8 + xcd;
            if (grp != 0) {
                BAR;
                if (tz == 0) misc[1] = (NDIRECT + 16u) + __hip_atomic_fetch_add(qv, 2u, RLX_AGENT);
                WAIT_L(0); BAR; CFENCE;
                id0 = __builtin_amdgcn_readfirstlane(misc[1]); id1 = id0 + 1;
            }
            auto code_of = [&](int id) { return (int)((p.order[(id >> 2) & 63] >> ((id & 3) * 8)) & 255u); };
            auto src_of = [&](int code) {
                return make_rsrc(pws + stage_off + ((code >> 7) ? SRCW + (size_t)(rank * 128 + (code & 127)) * PIECE
                                                                : (size_t)(code >> 3) * PANEL + (size_t)((code & 7) * 8) * PIECE), PIECE);
            };
            auto wait_ready = [&](int code, unsigned seen) {
                if (__builtin_amdgcn_readfirstlane(seen) >= call) return;
                const u64 t0 = __builtin_amdgcn_s_memrealtime();
                while (__builtin_amdgcn_readfirstlane(__hip_atomic_load(rdy + code, RLX_SYS)) < call) {
                    if (__builtin_amdgcn_readfirstlane(__hip_atomic_load(tmo, RLX_AGENT)) == call
                        || __builtin_amdgcn_s_memrealtime() - t0 > DEADLINE) {
                        if (ln == 0) __hip_atomic_store(tmo, call, RLX_AGENT);
                        break;
                    }
                    __builtin_amdgcn_s_sleep(2);
                }
            };
            auto publish = [&](int code) {
                __hip_atomic_store((unsigned*)(ws + WS_FX) + (code >> 3) * 64 + (code & 7) * 8 + srel, call, RLX_AGENT);
            };
            unsigned* const next_ctl = (unsigned*)(ws + WS_CTRL + ((b + 64) & 255) * 256);
            unsigned* const next_go = next_ctl + (grp == 3 ? 1 : 2);
            bool passed = false;
            if (id0 < 256) {
                u32x4 d0, d1, d2, d3;
#define FLOAD(d, rs, i) d = __builtin_amdgcn_raw_buffer_load_b128(rs, vo + (i) * 1024, 0, AUX_FLD)
#define FSTORE(d, i) __builtin_amdgcn_raw_buffer_store_b128(d, rD, vo + (i) * 1024, 0, AUX_FST)
                int c0 = code_of(id0), cp = -1;
                wait_ready(c0, 0u);
                if (grp != 0) {
                    const unsigned* go = (const unsigned*)(ws + WS_CTRL + b * 256) + 2;
                    const u64 t0 = __builtin_amdgcn_s_memrealtime();
                    while (__builtin_amdgcn_readfirstlane(__hip_atomic_load(go, RLX_AGENT)) < call) {
                        if (__builtin_amdgcn_readfirstlane(__hip_atomic_load(tmo, RLX_AGENT)) == call
                            || __builtin_amdgcn_s_memrealtime() - t0 > DEADLINE) {
                            if (ln == 0) __hip_atomic_store(tmo, call, RLX_AGENT);
                            break;
                        }
                        __builtin_amdgcn_s_sleep(2);
                    }
                }
                STAMP(8);
                __amdgpu_buffer_rsrc_t rS = src_of(c0);
                FLOAD(d0, rS, 0);
                WAIT_VC(0);
                for (int n = 0;; ++n) {
                    const bool more = id1 < 256;
                    const bool fetch = more && n + 2 < quota;
                    const bool pass = !passed && grp < 3 && (!more || n + PASS_AHEAD >= quota);
                    const int c1 = code_of(id1);
                    const __amdgpu_buffer_rsrc_t rD = make_rsrc(ws + stage_off + (size_t)(c0 >> 3) * PANEL + (size_t)((c0 & 7) * 8 + srel) * PIECE, PIECE);
                    WAIT_VC(1);
                    const unsigned seen = __hip_atomic_load(rdy + c1, RLX_SYS);
                    unsigned got = 256u;
                    if (tz == 0 && fetch) got = __hip_atomic_fetch_add(qv, 1u, RLX_AGENT);
                    if (tz == 0 && pass) __hip_atomic_store(next_ctl, call, RLX_AGENT);
                    if (GO_AT == 1 && tz == 0 && !more) __hip_atomic_store(next_go, call, RLX_AGENT);
                    passed = passed || pass;
                    CFENCE;
                    FLOAD(d1, rS, 1); CFENCE; FSTORE(d0, 0);
                    WAIT_VC(1);
                    asm volatile("" : "+v"(got));
                    if (tz == 0) misc[2 + (n & 1)] = (NDIRECT + 16u) + got;
                    if (cp >= 0 && ln == 0 && __hip_atomic_fetch_add(misc + 8 + (n & 1), 1u, __ATOMIC_RELAXED, __HIP_MEMORY_SCOPE_WORKGROUP) % 8u == 7u) publish(cp);
                    CFENCE;
                    if (GO_AT == 2 && tz == 0 && !more) __hip_atomic_store(next_go, call, RLX_AGENT);
                    CFENCE;
                    FLOAD(d2, rS, 2); CFENCE; FSTORE(d1, 1);
                    WAIT_VC(1);
                    if (GO_AT == 3 && tz == 0 && !more) __hip_atomic_store(next_go, call, RLX_AGENT);
                    CFENCE;
                    FLOAD(d3, rS, 3); CFENCE; FSTORE(d2, 2);
                    WAIT_VC(1);
                    if (GO_AT == 4 && tz == 0 && !more) __hip_atomic_store(next_go, call, RLX_AGENT);
                    CFENCE;
                    if (more) {
                        wait_ready(c1, seen);
                        rS = src_of(c1);
                        FLOAD(d0, rS, 0);
                    }
                    CFENCE;
                    FSTORE(d3, 3);
                    WAIT_L(0); BAR; CFENCE;
                    if (!more) break;
                    cp = c0; c0 = c1; id1 = __builtin_amdgcn_readfirstlane(misc[2 + (n & 1)]);
                }
                WAIT_VC(0); BAR; CFENCE;
                if (tz == 64) publish(c0);
            } else if (tz == 0) {
                if (grp < 3) __hip_atomic_store(next_ctl, call, RLX_AGENT);
                __hip_atomic_store(next_go, call, RLX_AGENT);
            }
            STAMP(4);
        }
        if (run) {
            const unsigned voff = (unsigned)tid * 16u;
            const unsigned ldsw = (unsigned)wid * 1024u;
            const int aoff0 = (wr * 64 + fr) * 128 + (fq ^ (fr >> 1)) * 16, aoff1 = aoff0 ^ 64;
            int boff = SB(0, 0) + fq * 2048 + (wc * 16 + fr) * 16;
            asm volatile("" : "+v"(boff));
            half8 At[4][2], B0[2][2], B1[2][2];
            unsigned so = (unsigned)t * PIECE;
            STAGE(SB(0, 0), rB, so); STAGE(SA(0, 0), rA, so); STAGE(SB(0, 1), rB, so + HTB); STAGE(SA(0, 1), rA, so + HTB);
            if (wr == 1) BAR;
            WAIT_V(4); BAR;
            STAGE(SB(1, 0), rB, so + PIECE); STAGE(SA(1, 0), rA, so + PIECE); STAGE(SB(1, 1), rB, so + PIECE + HTB);
            WAIT_V(6); BAR;
            bool go;
            do {
                const unsigned sv = *(const LAS unsigned*)(lds + SNAP_ADDR(t + 2));
                SCHED;
                LDB(B0, 0, 0); SCHED; LDA(At, 0, 0); STAGE(SA(1, 1), rA, so + PIECE + HTB);
                WAIT_L(8); BAR; WAIT_L(0); MMA(0, 0, At, B0);
                const unsigned bits = (unsigned)__ballot(sv >= call);
                go = t + 2 < 64 && (bits & (state == 0 ? 31u : 15u)) == 15u;
                const unsigned s2 = go ? so + 2 * PIECE : OOB;
                BAR; SCHED;
                LDB(B1, 0, 1); STAGE(SB(0, 0), rB, s2);
                BAR; WAIT_L(0); MMA(0, 1, At, B1); BAR;
                LDA(At, 0, 1); STAGE(SA(0, 0), rA, s2);
                BAR; WAIT_L(0); MMA(1, 0, At, B0); BAR; SCHED;
                STAGE(SB(0, 1), rB, s2 + HTB);
                if (!go) WAIT_V(0);
                WAIT_V(6); BAR; MMA(1, 1, At, B1); BAR;
                SNAP();
                LDB(B0, 1, 0); SCHED; LDA(At, 1, 0); STAGE(SA(0, 1), rA, s2 + HTB);
                WAIT_L(8); BAR; WAIT_L(0); MMA(0, 0, At, B0); BAR; SCHED;
                LDB(B1, 1, 1); STAGE(SB(1, 0), rB, s2 + PIECE);
                BAR; WAIT_L(0); MMA(0, 1, At, B1); BAR;
                LDA(At, 1, 1); STAGE(SA(1, 0), rA, s2 + PIECE);
                BAR; WAIT_L(0); MMA(1, 0, At, B0); BAR; SCHED;
                STAGE(SB(1, 1), rB, s2 + PIECE + HTB);
                WAIT_V(6); BAR; MMA(1, 1, At, B1); BAR;
                t += 2; so += 2 * PIECE;
            } while (go);
            WAIT_V(0);
            if (wr == 0) BAR;
            BAR;
        }
    } while (t < 64 || state < 2);
    STAMP(6);

    {
        int tz = tid; asm volatile("" : "+v"(tz));
        const int ln = tz & 63, efr = ln & 15, efq = ln >> 4;
        const float qnan = __builtin_nanf("");
        const __amdgpu_buffer_rsrc_t rO = make_rsrc(p.out, (unsigned)MS * GN * 4u);
        const unsigned lds0 = (unsigned)(size_t)lds;
        const unsigned reg0 = (unsigned)wid * CHUNK;
        const bool bad = __builtin_amdgcn_readfirstlane(__hip_atomic_load(tmo, RLX_AGENT)) == call;
        if (v == rank) {
            const unsigned base = (unsigned)(((pm & 1) * 256 + wr * 64 + efq * 4) * GN + rank * NB + pn * 256 + wc * 64 + efr * 4) * 4u;
#pragma unroll
            for (int ai = 0; ai < 2; ++ai)
#pragma unroll
                for (int m = 0; m < 4; ++m)
#pragma unroll
                    for (int r = 0; r < 4; ++r) {
                        f32x4 o = (f32x4){acc[ai][0][m][0][r], acc[ai][0][m][1][r], acc[ai][1][m][0][r], acc[ai][1][m][1][r]};
#pragma unroll
                        for (int e = 0; e < 4; ++e) o[e] = bad ? qnan : (o[e] < 0.f ? 0.f : o[e]);
                        __builtin_amdgcn_raw_buffer_store_b128(__builtin_bit_cast(u32x4, o), rO, base + (unsigned)((ai * 128 + m * 16 + r) * GN) * 4u, 0, AUX_OUT);
                    }
        } else {
            unsigned sizes = 0;
#pragma unroll
            for (int ai = 0; ai < 2; ++ai) {
                unsigned mk0 = 0u, mk1 = 0u;
                unsigned pos = lds0 + reg0 + 512u;
#pragma unroll
                for (int m = 0; m < 4; ++m)
#pragma unroll
                    for (int r = 0; r < 4; ++r) {
                        if (m < 2) compact4(acc[ai][0][m][0][r], acc[ai][0][m][1][r], acc[ai][1][m][0][r], acc[ai][1][m][1][r], mk0, pos);
                        else compact4(acc[ai][0][m][0][r], acc[ai][0][m][1][r], acc[ai][1][m][0][r], acc[ai][1][m][1][r], mk1, pos);
                    }
                *(LAS u64*)(lds + reg0 + ln * 8) = ((u64)mk1 << 32) | mk0;
                WAIT_L(0);
                const unsigned bytes = pos - lds0 - reg0;
                sizes |= bytes << (16 * ai);
                const __amdgpu_buffer_rsrc_t rs = make_rsrc(ws + WS_OBOX + (size_t)(v * 512 + grp * 128 + ai * 64 + xcd * 8 + wid) * CHUNK, (bytes + 63u) & ~63u);
                for (unsigned o = 0; o < bytes; o += 1024)
                    __builtin_amdgcn_raw_buffer_store_b128(*(const LAS u32x4*)(lds + reg0 + ln * 16 + o), rs, ln * 16 + o, 0, AUX_OST);
                WAIT_L(0);
            }
            WAIT_V(0);
            if (ln == 0) __hip_atomic_store((u64*)(pws + WS_FO + (rank * 256 + grp * 64 + xcd * 8 + wid) * 64), ((u64)call << 32) | (bad ? 0xffffffffu : sizes), RLX_SYS);
            STAMP(7);
            if (grp < OGENS) {
                LAS unsigned* const req = misc + 16;
                LAS unsigned* const fin = misc + 24;
                if (wid == 0) {
                    {
                        const unsigned* tok = (const unsigned*)(ws + WS_CTRL + (b & 63) * 256) + 1;
                        const u64 t0 = __builtin_amdgcn_s_memrealtime();
                        while (__builtin_amdgcn_readfirstlane(__hip_atomic_load(tok, RLX_AGENT)) < call) {
                            if (__builtin_amdgcn_readfirstlane(__hip_atomic_load(tmo, RLX_AGENT)) == call
                                || __builtin_amdgcn_s_memrealtime() - t0 > DEADLINE) {
                                if (ln == 0) __hip_atomic_store(tmo, call, RLX_AGENT);
                                break;
                            }
                            __builtin_amdgcn_s_sleep(2);
                        }
                    }
                    STAMP(5);
                    const unsigned reqa = lds0 + LDS_MISC + 64, fina = reqa + 32;
                    unsigned pend = 0, last = 0;
                    for (;;) {
                        const unsigned r = lds_ld(reqa + (ln & 7) * 4);
                        const unsigned done = (unsigned)__ballot(r == 0xffffffffu) & 0xfeu;
                        const unsigned want = (unsigned)__ballot(r != 0u && r != 0xffffffffu) & 0xfeu;
                        if (want == 0u) {
                            if (pend) { WAIT_V(0); lds_st(fina + pend * 4, 1u); pend = 0; }
                            if (done == 0xfeu) break;
                            __builtin_amdgcn_s_sleep(1);
                            continue;
                        }
                        const unsigned hi = want & (0xfffffffeu << last);
                        const unsigned w = __builtin_ctz(hi ? hi : want);
                        last = w;
                        const unsigned rq = __builtin_amdgcn_readlane(r, w);
                        lds_st(reqa + w * 4, 0u);
                        const unsigned bytes = rq & 0xffffu;
                        const __amdgpu_buffer_rsrc_t rs = make_rsrc(pws + WS_OBOX + (size_t)(rank * 512 + ((rq >> 16) & 511u)) * CHUNK, bytes);
                        for (unsigned o = 0; o < bytes; o += 1024) {
                            if (grp == 0) WAIT_VX(OWIN0); else WAIT_VX(OWIN1);
                            if (pend && o >= (grp == 0 ? OWIN0 : OWIN1) * 1024u) { lds_st(fina + pend * 4, 1u); pend = 0; }
                            __builtin_amdgcn_raw_ptr_buffer_load_lds(rs, (LAS void*)(lds + w * CHUNK + o), 16, ln * 16, o, 0, AUX_OLD);
                        }
                        if (pend) { WAIT_V(0); lds_st(fina + pend * 4, 1u); }
                        pend = w;
                    }
                } else {
                    unsigned* const oq = (unsigned*)(queue + 1) + (tz >> 10);
                    unsigned i = (unsigned)((wid - 1) * 8 + xcd);
                    if (grp != 0) {
                        unsigned g = 0;
                        if (ln == 0) g = __hip_atomic_fetch_add(oq, 1u, RLX_AGENT);
                        i = 56u + __builtin_amdgcn_readfirstlane(g);
                    }
                    while (i < 512u) {
                        unsigned nxt = 0;
                        if (ln == 0) nxt = __hip_atomic_fetch_add(oq, 1u, RLX_AGENT);
                        const unsigned g2 = i >> 7, ai = (i >> 6) & 1u, x2 = (i >> 3) & 7u, w2 = i & 7u;
                        const u64* fo = (const u64*)(ws + WS_FO + (v * 256 + g2 * 64 + (i & 63u)) * 64);
                        const u64 t0 = __builtin_amdgcn_s_memrealtime();
                        unsigned bytes;
                        for (;;) {
                            const u64 fv = __hip_atomic_load(fo, RLX_SYS);
                            bytes = __builtin_amdgcn_readfirstlane((unsigned)fv);
                            if (__builtin_amdgcn_readfirstlane((unsigned)(fv >> 32)) >= call) break;
                            if (__builtin_amdgcn_readfirstlane(__hip_atomic_load(tmo, RLX_AGENT)) == call
                                || __builtin_amdgcn_s_memrealtime() - t0 > DEADLINE) {
                                if (ln == 0) __hip_atomic_store(tmo, call, RLX_AGENT);
                                bytes = 0xffffffffu;
                                break;
                            }
                            __builtin_amdgcn_s_sleep(4);
                        }
                        bytes = (bytes >> (16u * ai)) & 0xffffu;
                        const bool nodata = bytes > 512u + 16384u || bytes < 512u;
                        if (!nodata) {
                            if (ln == 0) __hip_atomic_store(req + wid, 0x80000000u | i << 16 | bytes, __ATOMIC_RELAXED, __HIP_MEMORY_SCOPE_WORKGROUP);
                            while (__builtin_amdgcn_readfirstlane(__hip_atomic_load(fin + wid, __ATOMIC_RELAXED, __HIP_MEMORY_SCOPE_WORKGROUP)) == 0u)
                                __builtin_amdgcn_s_sleep(2);
                            if (ln == 0) fin[wid] = 0u;
                            CFENCE;
                        }
                        const u64 mm = *(const LAS u64*)(lds + reg0 + ln * 8);
                        unsigned mk = nodata ? 0u : (unsigned)mm;
                        unsigned pos = lds0 + reg0 + 512u;
                        unsigned off = (unsigned)(((x2 & 1u) * 256u + ai * 128u + (w2 >> 2) * 64u + efq * 4) * GN + v * NB + (4u * g2 + (x2 >> 1)) * 256u + (w2 & 3u) * 64u + efr * 4) * 4u;
                        const float fill = nodata ? qnan : 0.f;
                        for (int m = 0; m < 4; ++m) {
                            f32x4 o[4];
                            expand16(o, mk, pos, fill);
#pragma unroll
                            for (int r = 0; r < 4; ++r)
                                __builtin_amdgcn_raw_buffer_store_b128(__builtin_bit_cast(u32x4, o[r]), rO, off + (unsigned)(r * GN) * 4u, 0, AUX_OUT);
                            off += 16u * GN * 4u;
                            if (m == 1) mk = nodata ? 0u : (unsigned)(mm >> 32);
                        }
                        i = 56u + __builtin_amdgcn_readfirstlane(nxt);
                    }
                    if (ln == 0) __hip_atomic_store(req + wid, 0xffffffffu, __ATOMIC_RELAXED, __HIP_MEMORY_SCOPE_WORKGROUP);
                }
            }
        }
        STAMP(9);
    }
}

}

extern "C" void kernel_launch(int rank, int nranks, ncclComm_t comm,
                              hipStream_t stream,
                              void* const* d_in, const int* in_sizes, int n_in,
                              void* d_out, int out_size,
                              const int* in_dtypes, int out_dtype,
                              void* const* d_out_all,
                              void* const* d_ws_all, size_t ws_bytes) {
    if (nranks != NR || n_in < 2 || (unsigned)rank >= (unsigned)NR || ws_bytes < WS_NEED ||
        in_sizes[0] != GM * KS || in_sizes[1] != KS * GN || out_size != MS * GN) {
        fprintf(stderr, "kernel_launch: unexpected problem shape\n");
        return;
    }
    Params p;
    p.x = (const float*)d_in[0];
    p.w = (const float*)d_in[1];
    p.out = (float*)d_out;
    for (int r = 0; r < NR; ++r) p.ws_all[r] = (char*)d_ws_all[r];
    make_order(p.order, p.worder);
    p.rank = rank; p.pad = 0; p.ws = (char*)d_ws_all[rank];
    hipLaunchKernelGGL(fused_gemm_rs, dim3(NWG), dim3(512), 0, stream, p);
    HIP_CHECK(hipGetLastError());
}
```

```cpp
#include <hip/hip_runtime.h>
#include <rccl/rccl.h>
#include <cstdio>

#define HIP_CHECK(expr) do { hipError_t _e = (expr); if (_e != hipSuccess) { \
    fprintf(stderr, "HIP error %s at %s:%d\n", hipGetErrorString(_e), __FILE__, __LINE__); \
    return; } } while (0)

namespace {

typedef _Float16 half8 __attribute__((ext_vector_type(8)));
typedef _Float16 half2v __attribute__((ext_vector_type(2)));
typedef float f32x4 __attribute__((ext_vector_type(4)));
typedef unsigned u32x4 __attribute__((ext_vector_type(4)));
typedef unsigned long long u64;
#define LAS __attribute__((address_space(3)))

constexpr int NR = 8;
constexpr int GM = 4096, GK = 4096, GN = 32768;
constexpr int KS = GK / NR;
constexpr int NB = GN / NR;
constexpr int MS = GM / NR;
constexpr int NWG = 256;
constexpr unsigned PIECE = 32768;
constexpr unsigned HTB = 16384;
constexpr unsigned PANEL = 64 * PIECE;
constexpr unsigned CHUNK = 17408;
constexpr unsigned WS_GLOB = 0;
constexpr unsigned WS_AM = 256;
constexpr unsigned WS_AML = 4096;
constexpr unsigned WS_Q = 1024;
constexpr unsigned WS_FX = 8192;
constexpr unsigned WS_FW = WS_FX + 4096;
constexpr unsigned WS_RDY = 16384;
constexpr unsigned WS_CTRL = 65536;
constexpr unsigned WS_FO = 262144;
constexpr unsigned WS_EPOCH = 1u << 19;
constexpr unsigned WS_CTL_BYTES = 1u << 20;
constexpr size_t WS_STAGE = 1u << 20;
constexpr size_t STAGE_PAR = 48ull * PANEL;
constexpr size_t SRCW = 32ull * PANEL;
constexpr size_t WS_OBOX = WS_STAGE + 3 * STAGE_PAR;
constexpr size_t WS_NEED = WS_OBOX + 4096ull * CHUNK;
constexpr u64 DEADLINE = 50000000ull;
constexpr int AUX_NT = 2;
constexpr int AUX_SYS = 17;
constexpr unsigned LDS_SNAP = 8 * CHUNK + 1024;
constexpr unsigned LDS_MISC = LDS_SNAP + 2048;
constexpr unsigned LDS_BYTES = LDS_MISC + 128;
static_assert(LDS_BYTES <= 163840 && CHUNK >= HTB, "LDS layout");
constexpr unsigned OOB = 0x40000000u;

constexpr int FGEN = 2;
constexpr int QUOTA = 16;
constexpr int PASS_AHEAD = 2;
constexpr int DEF1 = 6, DEF2 = 4, DEF3 = 3;
constexpr int NEXP = 7;
#define OWIN0 7

struct Params {
    const float* x;
    const float* w;
    char* ws;
    int rank;
    int pad;
    float* out;
    char* ws_all[NR];
    unsigned order[64];
    unsigned worder[32];
};

static void make_order(unsigned* order, unsigned* worder) {
    const int def[4] = {8, DEF1, DEF2, DEF3};
    unsigned char q[256], wq[128]; int n = 0, m = 0;
    for (int j = 0; j < 8; ++j) {
        for (int pn = 0; pn < 16; ++pn) if (j < def[pn >> 2]) q[n++] = (unsigned char)(128 | pn << 3 | j);
        for (int pm = 0; pm < 16; ++pm) q[n++] = (unsigned char)(pm << 3 | j);
    }
    for (int g = 1; g < 4; ++g)
        for (int j = def[g]; j < 8; ++j)
            for (int pn = 4 * g; pn < 4 * g + 4; ++pn) q[n++] = (unsigned char)(128 | pn << 3 | j);
    for (int i = 0; i < 256; ++i) if (q[i] & 128) wq[m++] = q[i];
    for (int i = 0; i < 64; ++i) order[i] = q[4 * i] | q[4 * i + 1] << 8 | q[4 * i + 2] << 16 | (unsigned)q[4 * i + 3] << 24;
    for (int i = 0; i < 32; ++i) worder[i] = wq[4 * i] | wq[4 * i + 1] << 8 | wq[4 * i + 2] << 16 | (unsigned)wq[4 * i + 3] << 24;
}

__device__ __forceinline__ unsigned pack2(float a, float b) {
    half2v h; h.x = (_Float16)a; h.y = (_Float16)b;
    return __builtin_bit_cast(unsigned, h);
}
__device__ __forceinline__ __amdgpu_buffer_rsrc_t make_rsrc(const void* p, unsigned bytes) {
    const u64 v = (u64)p;
    const unsigned lo = __builtin_amdgcn_readfirstlane((unsigned)v), hi = __builtin_amdgcn_readfirstlane((unsigned)(v >> 32));
    return __builtin_amdgcn_make_buffer_rsrc((void*)(((u64)hi << 32) | lo), 0, bytes, 0x00020000);
}

__device__ __forceinline__ unsigned lds_ld(unsigned addr) {
    unsigned r; asm volatile("ds_read_b32 %0, %1\n\ts_waitcnt lgkmcnt(0)" : "=v"(r) : "v"(addr) : "memory"); return r;
}
__device__ __forceinline__ void lds_st(unsigned addr, unsigned val) { asm volatile("ds_write_b32 %0, %1" :: "v"(addr), "v"(val) : "memory"); }

__device__ __forceinline__ void compact4(float x0, float x1, float x2, float x3, unsigned& mk, unsigned& pos) {
    unsigned a0, a1, a2, a3, c, p1, p2, p3;
    asm volatile(
        "v_cmp_nle_f32 s[88:89], %10, 0\n\t"
        "v_cmp_nle_f32 s[90:91], %11, 0\n\t"
        "v_cmp_nle_f32 s[92:93], %12, 0\n\t"
        "v_cmp_nle_f32 s[94:95], %13, 0\n\t"
        "s_bcnt1_i32_b64 %4, s[88:89]\n\t"
        "v_mbcnt_lo_u32_b32 %0, s88, 0\n\t"
        "s_lshl2_add_u32 %5, %4, %9\n\t"
        "v_mbcnt_lo_u32_b32 %1, s90, 0\n\t"
        "s_bcnt1_i32_b64 %4, s[90:91]\n\t"
        "v_mbcnt_lo_u32_b32 %2, s92, 0\n\t"
        "s_lshl2_add_u32 %6, %4, %5\n\t"
        "v_mbcnt_lo_u32_b32 %3, s94, 0\n\t"
        "s_bcnt1_i32_b64 %4, s[92:93]\n\t"
        "v_mbcnt_hi_u32_b32 %0, s89, %0\n\t"
        "s_lshl2_add_u32 %7, %4, %6\n\t"
        "v_mbcnt_hi_u32_b32 %1, s91, %1\n\t"
        "s_bcnt1_i32_b64 %4, s[94:95]\n\t"
        "v_mbcnt_hi_u32_b32 %2, s93, %2\n\t"
        "v_mbcnt_hi_u32_b32 %3, s95, %3\n\t"
        "v_lshl_add_u32 %0, %0, 2, %9\n\t"
        "v_lshl_add_u32 %1, %1, 2, %5\n\t"
        "v_lshl_add_u32 %2, %2, 2, %6\n\t"
        "v_lshl_add_u32 %3, %3, 2, %7\n\t"
        "s_lshl2_add_u32 %9, %4, %7\n\t"
        "v_addc_co_u32 %8, vcc, %8, %8, s[88:89]\n\t"
        "v_addc_co_u32 %8, vcc, %8, %8, s[90:91]\n\t"
        "v_addc_co_u32 %8, vcc, %8, %8, s[92:93]\n\t"
        "v_addc_co_u32 %8, vcc, %8, %8, s[94:95]\n\t"
        "s_mov_b64 exec, s[88:89]\n\t"
        "ds_write_b32 %0, %10\n\t"
        "s_mov_b64 exec, s[90:91]\n\t"
        "ds_write_b32 %1, %11\n\t"
        "s_mov_b64 exec, s[92:93]\n\t"
        "ds_write_b32 %2, %12\n\t"
        "s_mov_b64 exec, s[94:95]\n\t"
        "ds_write_b32 %3, %13\n\t"
        "s_mov_b64 exec, -1"
        : "=&v"(a0), "=&v"(a1), "=&v"(a2), "=&v"(a3), "=&s"(c), "=&s"(p1), "=&s"(p2), "=&s"(p3), "+v"(mk), "+s"(pos)
        : "v"(x0), "v"(x1), "v"(x2), "v"(x3)
        : "vcc", "scc", "memory", "s88", "s89", "s90", "s91", "s92", "s93", "s94", "s95");
}

__device__ __forceinline__ void expand16(f32x4 (&o)[4], unsigned& mk, unsigned& pos, float fill) {
    float x[16]; unsigned a0, a1, a2, a3, c, p1, p2, p3;
    asm volatile(
        "v_add_co_u32 %24, s[88:89], %24, %24\n\t"
        "v_add_co_u32 %24, s[90:91], %24, %24\n\t"
        "v_add_co_u32 %24, s[92:93], %24, %24\n\t"
        "v_add_co_u32 %24, s[94:95], %24, %24\n\t"
        "v_mov_b32 %0, %26\n\t"
        "v_mov_b32 %1, %26\n\t"
        "s_bcnt1_i32_b64 %20, s[88:89]\n\t"
        "v_mbcnt_lo_u32_b32 %16, s88, 0\n\t"
        "s_lshl2_add_u32 %21, %20, %25\n\t"
        "v_mbcnt_lo_u32_b32 %17, s90, 0\n\t"
        "s_bcnt1_i32_b64 %20, s[90:91]\n\t"
        "v_mbcnt_lo_u32_b32 %18, s92, 0\n\t"
        "s_lshl2_add_u32 %22, %20, %21\n\t"
        "v_mbcnt_lo_u32_b32 %19, s94, 0\n\t"
        "s_bcnt1_i32_b64 %20, s[92:93]\n\t"
        "v_mbcnt_hi_u32_b32 %16, s89, %16\n\t"
        "s_lshl2_add_u32 %23, %20, %22\n\t"
        "v_mbcnt_hi_u32_b32 %17, s91, %17\n\t"
        "s_bcnt1_i32_b64 %20, s[94:95]\n\t"
        "v_mbcnt_hi_u32_b32 %18, s93, %18\n\t"
        "v_mbcnt_hi_u32_b32 %19, s95, %19\n\t"
        "v_lshl_add_u32 %16, %16, 2, %25\n\t"
        "v_lshl_add_u32 %17, %17, 2, %21\n\t"
        "v_lshl_add_u32 %18, %18, 2, %22\n\t"
        "v_lshl_add_u32 %19, %19, 2, %23\n\t"
        "s_lshl2_add_u32 %25, %20, %23\n\t"
        "v_mov_b32 %2, %26\n\t"
        "v_mov_b32 %3, %26\n\t"
        "s_mov_b64 exec, s[88:89]\n\t"
        "ds_read_b32 %0, %16\n\t"
        "s_mov_b64 exec, s[90:91]\n\t"
        "ds_read_b32 %1, %17\n\t"
        "s_mov_b64 exec, s[92:93]\n\t"
        "ds_read_b32 %2, %18\n\t"
        "s_mov_b64 exec, s[94:95]\n\t"
        "ds_read_b32 %3, %19\n\t"
        "s_mov_b64 exec, -1\n\t"
        "v_add_co_u32 %24, s[88:89], %24, %24\n\t"
        "v_add_co_u32 %24, s[90:91], %24, %24\n\t"
        "v_add_co_u32 %24, s[92:93], %24, %24\n\t"
        "v_add_co_u32 %24, s[94:95], %24, %24\n\t"
        "v_mov_b32 %4, %26\n\t"
        "v_mov_b32 %5, %26\n\t"
        "s_bcnt1_i32_b64 %20, s[88:89]\n\t"
        "v_mbcnt_lo_u32_b32 %16, s88, 0\n\t"
        "s_lshl2_add_u32 %21, %20, %25\n\t"
        "v_mbcnt_lo_u32_b32 %17, s90, 0\n\t"
        "s_bcnt1_i32_b64 %20, s[90:91]\n\t"
        "v_mbcnt_lo_u32_b32 %18, s92, 0\n\t"
        "s_lshl2_add_u32 %22, %20, %21\n\t"
        "v_mbcnt_lo_u32_b32 %19, s94, 0\n\t"
        "s_bcnt1_i32_b64 %20, s[92:93]\n\t"
        "v_mbcnt_hi_u32_b32 %16, s89, %16\n\t"
        "s_lshl2_add_u32 %23, %20, %22\n\t"
        "v_mbcnt_hi_u32_b32 %17, s91, %17\n\t"
        "s_bcnt1_i32_b64 %20, s[94:95]\n\t"
        "v_mbcnt_hi_u32_b32 %18, s93, %18\n\t"
        "v_mbcnt_hi_u32_b32 %19, s95, %19\n\t"
        "v_lshl_add_u32 %16, %16, 2, %25\n\t"
        "v_lshl_add_u32 %17, %17, 2, %21\n\t"
        "v_lshl_add_u32 %18, %18, 2, %22\n\t"
        "v_lshl_add_u32 %19, %19, 2, %23\n\t"
        "s_lshl2_add_u32 %25, %20, %23\n\t"
        "v_mov_b32 %6, %26\n\t"
        "v_mov_b32 %7, %26\n\t"
        "s_mov_b64 exec, s[88:89]\n\t"
        "ds_read_b32 %4, %16\n\t"
        "s_mov_b64 exec, s[90:91]\n\t"
        "ds_read_b32 %5, %17\n\t"
        "s_mov_b64 exec, s[92:93]\n\t"
        "ds_read_b32 %6, %18\n\t"
        "s_mov_b64 exec, s[94:95]\n\t"
        "ds_read_b32 %7, %19\n\t"
        "s_mov_b64 exec, -1\n\t"
        "v_add_co_u32 %24, s[88:89], %24, %24\n\t"
        "v_add_co_u32 %24, s[90:91], %24, %24\n\t"
        "v_add_co_u32 %24, s[92:93], %24, %24\n\t"
        "v_add_co_u32 %24, s[94:95], %24, %24\n\t"
        "v_mov_b32 %8, %26\n\t"
        "v_mov_b32 %9, %26\n\t"
        "s_bcnt1_i32_b64 %20, s[88:89]\n\t"
        "v_mbcnt_lo_u32_b32 %16, s88, 0\n\t"
        "s_lshl2_add_u32 %21, %20, %25\n\t"
        "v_mbcnt_lo_u32_b32 %17, s90, 0\n\t"
        "s_bcnt1_i32_b64 %20, s[90:91]\n\t"
        "v_mbcnt_lo_u32_b32 %18, s92, 0\n\t"
        "s_lshl2_add_u32 %22, %20, %21\n\t"
        "v_mbcnt_lo_u32_b32 %19, s94, 0\n\t"
        "s_bcnt1_i32_b64 %20, s[92:93]\n\t"
        "v_mbcnt_hi_u32_b32 %16, s89, %16\n\t"
        "s_lshl2_add_u32 %23, %20, %22\n\t"
        "v_mbcnt_hi_u32_b32 %17, s91, %17\n\t"
        "s_bcnt1_i32_b64 %20, s[94:95]\n\t"
        "v_mbcnt_hi_u32_b32 %18, s93, %18\n\t"
        "v_mbcnt_hi_u32_b32 %19, s95, %19\n\t"
        "v_lshl_add_u32 %16, %16, 2, %25\n\t"
        "v_lshl_add_u32 %17, %17, 2, %21\n\t"
        "v_lshl_add_u32 %18, %18, 2, %22\n\t"
        "v_lshl_add_u32 %19, %19, 2, %23\n\t"
        "s_lshl2_add_u32 %25, %20, %23\n\t"
        "v_mov_b32 %10, %26\n\t"
        "v_mov_b32 %11, %26\n\t"
        "s_mov_b64 exec, s[88:89]\n\t"
        "ds_read_b32 %8, %16\n\t"
        "s_mov_b64 exec, s[90:91]\n\t"
        "ds_read_b32 %9, %17\n\t"
        "s_mov_b64 exec, s[92:93]\n\t"
        "ds_read_b32 %10, %18\n\t"
        "s_mov_b64 exec, s[94:95]\n\t"
        "ds_read_b32 %11, %19\n\t"
        "s_mov_b64 exec, -1\n\t"
        "v_add_co_u32 %24, s[88:89], %24, %24\n\t"
        "v_add_co_u32 %24, s[90:91], %24, %24\n\t"
        "v_add_co_u32 %24, s[92:93], %24, %24\n\t"
        "v_add_co_u32 %24, s[94:95], %24, %24\n\t"
        "v_mov_b32 %12, %26\n\t"
        "v_mov_b32 %13, %26\n\t"
        "s_bcnt1_i32_b64 %20, s[88:89]\n\t"
        "v_mbcnt_lo_u32_b32 %16, s88, 0\n\t"
        "s_lshl2_add_u32 %21, %20, %25\n\t"
        "v_mbcnt_lo_u32_b32 %17, s90, 0\n\t"
        "s_bcnt1_i32_b64 %20, s[90:91]\n\t"
        "v_mbcnt_lo_u32_b32 %18, s92, 0\n\t"
        "s_lshl2_add_u32 %22, %20, %21\n\t"
        "v_mbcnt_lo_u32_b32 %19, s94, 0\n\t"
        "s_bcnt1_i32_b64 %20, s[92:93]\n\t"
        "v_mbcnt_hi_u32_b32 %16, s89, %16\n\t"
        "s_lshl2_add_u32 %23, %20, %22\n\t"
        "v_mbcnt_hi_u32_b32 %17, s91, %17\n\t"
        "s_bcnt1_i32_b64 %20, s[94:95]\n\t"
        "v_mbcnt_hi_u32_b32 %18, s93, %18\n\t"
        "v_mbcnt_hi_u32_b32 %19, s95, %19\n\t"
        "v_lshl_add_u32 %16, %16, 2, %25\n\t"
        "v_lshl_add_u32 %17, %17, 2, %21\n\t"
        "v_lshl_add_u32 %18, %18, 2, %22\n\t"
        "v_lshl_add_u32 %19, %19, 2, %23\n\t"
        "s_lshl2_add_u32 %25, %20, %23\n\t"
        "v_mov_b32 %14, %26\n\t"
        "v_mov_b32 %15, %26\n\t"
        "s_mov_b64 exec, s[88:89]\n\t"
        "ds_read_b32 %12, %16\n\t"
        "s_mov_b64 exec, s[90:91]\n\t"
        "ds_read_b32 %13, %17\n\t"
        "s_mov_b64 exec, s[92:93]\n\t"
        "ds_read_b32 %14, %18\n\t"
        "s_mov_b64 exec, s[94:95]\n\t"
        "ds_read_b32 %15, %19\n\t"
        "s_mov_b64 exec, -1\n\t"
        "s_waitcnt lgkmcnt(0)"
        : "=&v"(x[0]), "=&v"(x[1]), "=&v"(x[2]), "=&v"(x[3]), "=&v"(x[4]), "=&v"(x[5]), "=&v"(x[6]), "=&v"(x[7]), "=&v"(x[8]), "=&v"(x[9]), "=&v"(x[10]), "=&v"(x[11]), "=&v"(x[12]), "=&v"(x[13]), "=&v"(x[14]), "=&v"(x[15]),
          "=&v"(a0), "=&v"(a1), "=&v"(a2), "=&v"(a3), "=&s"(c), "=&s"(p1), "=&s"(p2), "=&s"(p3), "+v"(mk), "+s"(pos)
        : "v"(fill)
        : "scc", "memory", "s88", "s89", "s90", "s91", "s92", "s93", "s94", "s95");
#pragma unroll
    for (int r = 0; r < 4; ++r) o[r] = (f32x4){x[4 * r], x[4 * r + 1], x[4 * r + 2], x[4 * r + 3]};
}

__global__ void __launch_bounds__(512, 2) fused_gemm_rs(const Params p) {
    __shared__ __attribute__((aligned(16))) unsigned char smem[LDS_BYTES];
    LAS unsigned char* lds = (LAS unsigned char*)smem;
    LAS unsigned* misc = (LAS unsigned*)(lds + LDS_MISC);

    const int tid = threadIdx.x, lane = tid & 63;
    const int fr = lane & 15, fq = lane >> 4;
#define DERIVE \
    int b = blockIdx.x, wid = __builtin_amdgcn_readfirstlane(tid >> 6), rank = p.rank; \
    unsigned zero = 0; \
    asm volatile("" : "+s"(b), "+s"(wid), "+s"(rank), "+s"(zero)); \
    char* const ws = p.ws + zero; \
    const int wr = wid >> 2, wc = wid & 3; \
    unsigned* const tmo = (unsigned*)(ws + WS_GLOB); \
    const int xcd = b & 7, grp = b >> 6, v = (b >> 3) & 7; \
    const int pm = 4 * (xcd >> 1) + (v & 3), pn = 4 * grp + 2 * (xcd & 1) + (v >> 2); \
    const int dest = pm >> 1, tix = (pn & 3) * 2 + (pm & 1); \
    char* const pws = p.ws_all[v]; \
    u64* const queue = (u64*)(ws + WS_Q + v * 128); \
    (void)dest; (void)tix; (void)pws; (void)queue; (void)wr; (void)wc; (void)tmo

#define WAIT_V(n) asm volatile("s_waitcnt vmcnt(" #n ")" ::: "memory")
#define WAIT_VC(n) do { __builtin_amdgcn_s_waitcnt(0x0F70 | (n)); WAIT_V(n); } while (0)
#define WAIT_VX(n) WAIT_V(n)
#define WAIT_L(n) asm volatile("s_waitcnt lgkmcnt(" #n ")" ::: "memory")
#define BAR __builtin_amdgcn_s_barrier()
#define SCHED __builtin_amdgcn_sched_barrier(0)
#define CFENCE asm volatile("" ::: "memory")
#define RLX_AGENT __ATOMIC_RELAXED, __HIP_MEMORY_SCOPE_AGENT
#define RLX_SYS __ATOMIC_RELAXED, __HIP_MEMORY_SCOPE_SYSTEM

    unsigned call = 0;
    size_t stage_off;
    f32x4 acc[2][2][4][2];
    int state;
    float sx = 1.f, sw = 1.f;
    unsigned eo = 0;
    bool again;
    do {
    {
        DERIVE;
        int t1 = tid; asm volatile("" : "+v"(t1));
        const int l1 = t1 & 63;
        const int cg = grp == FGEN ? 0 : grp < FGEN ? grp + 1 : grp;
        const int njobs = cg == 0 ? 0 : 6;
        auto job_code = [&](int n) {
            if (cg == 3 && n == 0) { const int u = v * 8 + xcd; return (u & 15) << 3 | (u >> 4); }
            if (cg == 2 && n == 5) { const int u = 64 + v * 8 + xcd; return (u & 15) << 3 | (u >> 4); }
            const int i = xcd + (cg - 1) * 8 + 24 * (cg == 3 ? n - 1 : n);
            return (int)((p.worder[(i >> 2) & 31] >> ((i & 3) * 8)) & 255u);
        };
        f32x4 f[8];
        auto load = [&](int code) {
            const int type = code >> 7, panel = (code >> 3) & 15, j = code & 7;
            const float* src = type ? p.w + (size_t)(j * 64 + wid * 8) * GN + v * NB + panel * 256
                                    : p.x + (size_t)(panel * 256 + wid * 32) * KS + j * 64;
            const __amdgpu_buffer_rsrc_t rs = make_rsrc(src, 8u * GN * 4u);
            const unsigned vo = type ? l1 * 16 : (l1 >> 3) * 2048 + (l1 & 7) * 32;
            const unsigned s1 = type ? GN * 4 : 16, s2 = type ? 2 * GN * 4 : 16384;
#pragma unroll
            for (int i = 0; i < 8; ++i)
                f[i] = __builtin_bit_cast(f32x4, __builtin_amdgcn_raw_buffer_load_b128(rs, vo, (i >> 1) * s2 + (i & 1) * s1, AUX_NT));
        };
        int code = cg == 3 ? ((v * 8 + xcd) & 15) << 3 | (v >> 1) : 128 | (((cg - 1) * 8 + xcd) & 15) << 3;
        if (cg != 0) load(code);
        char* const lws = p.ws_all[t1 & 7];
        CFENCE;
        if (eo != 0u) {
            if (t1 == 0) __hip_atomic_fetch_max((u64*)((char*)queue + eo), (u64)call << 32, RLX_AGENT);
        } else if (t1 >= 72 && t1 < 96) misc[t1 - 64] = 0u;
        else if (t1 == 0) {
            unsigned* cp = (unsigned*)(ws + WS_CTRL + b * 256) + 32;
            const unsigned c = __hip_atomic_load(cp, RLX_AGENT) + 1u;
            __hip_atomic_store(cp, c, RLX_AGENT);
            __hip_atomic_fetch_max(queue, (u64)c << 32, RLX_AGENT);
            __hip_atomic_fetch_max(queue + 1, (u64)c << 32, RLX_AGENT);
            misc[0] = c;
        }
        WAIT_L(0); BAR; CFENCE;
        call = __builtin_amdgcn_readfirstlane(misc[0]);
        stage_off = WS_STAGE + (size_t)(eo ? 2u : call & 1u) * STAGE_PAR;
        float amx = 0.f, amw = 0.f;
#pragma unroll 1
        for (int n = 0; n < njobs; ++n) {
            const int type = code >> 7;
            u32x4 pk[4];
            {
                float a = 0.f;
                const float sc = type ? sw : sx;
#pragma unroll
                for (int i = 0; i < 8; ++i) {
                    a = __builtin_fmaxf(__builtin_fmaxf(a, __builtin_fmaxf(__builtin_fabsf(f[i][0]), __builtin_fabsf(f[i][1]))),
                                        __builtin_fmaxf(__builtin_fabsf(f[i][2]), __builtin_fabsf(f[i][3])));
                    f[i] *= sc;
                }
                if (type) amw = __builtin_fmaxf(amw, a); else amx = __builtin_fmaxf(amx, a);
            }
            if (type) {
#pragma unroll
                for (int e = 0; e < 4; ++e)
                    pk[e] = (u32x4){pack2(f[0][e], f[1][e]), pack2(f[2][e], f[3][e]), pack2(f[4][e], f[5][e]), pack2(f[6][e], f[7][e])};
            } else {
#pragma unroll
                for (int c = 0; c < 4; ++c)
                    pk[c] = (u32x4){pack2(f[2 * c][0], f[2 * c][1]), pack2(f[2 * c][2], f[2 * c][3]), pack2(f[2 * c + 1][0], f[2 * c + 1][1]), pack2(f[2 * c + 1][2], f[2 * c + 1][3])};
            }
            const bool direct = n == 0 && cg == 1;
            const bool staged = !type || v == rank || direct;
            const int srel = direct ? (rank - v) & 7 : 0;
            char* const dbase = direct ? pws : ws;
            const __amdgpu_buffer_rsrc_t rs = make_rsrc(dbase + stage_off + (staged ? (size_t)(code >> 3) * PANEL + (size_t)((code & 7) * 8 + srel) * PIECE
                                                                                    : SRCW + (size_t)(v * 128 + (code & 127)) * PIECE), PIECE);
            const unsigned vsX = wid * 4096 + (l1 >> 3) * 128 + ((l1 & 7) ^ (l1 >> 4)) * 16;
            const unsigned vo0 = type ? wid * 2048 + l1 * 16 : vsX, vo1 = (type ? vo0 : vsX ^ 64u) + 1024u, s2 = type ? HTB : 2048;
            const int cur = code;
            CFENCE;
#pragma unroll
            for (int c = 0; c < 4; ++c) {
                __builtin_amdgcn_raw_buffer_store_b128(pk[c], rs, ((c & 1) ? vo1 : vo0) + (c >> 1) * s2, 0, AUX_SYS);
                if (direct) WAIT_V(1);
            }
            CFENCE;
            if (n + 1 < njobs) { code = job_code(n + 1); load(code); }
            else {
#pragma unroll
                for (int o = 32; o >= 1; o >>= 1) { amx = __builtin_fmaxf(amx, __shfl_xor(amx, o)); amw = __builtin_fmaxf(amw, __shfl_xor(amw, o)); }
                if (l1 < 2) __hip_atomic_fetch_max(misc + 10 + l1, __float_as_uint(l1 ? amw : amx), __ATOMIC_RELAXED, __HIP_MEMORY_SCOPE_WORKGROUP);
                WAIT_L(0); WAIT_V(0);
            }
            WAIT_V(8); BAR; CFENCE;
            if (t1 < 8) {
                const bool mine = type ? t1 == v : true;
                const bool sflag = type ? staged : t1 == rank;
                unsigned* fp = sflag ? (unsigned*)(lws + eo + WS_FX) + (cur >> 3) * 64 + (cur & 7) * 8 + srel
                                     : (unsigned*)(lws + eo + WS_RDY) + rank * 256 + cur;
                if (mine) __hip_atomic_store(fp, call, RLX_SYS);
            }
        }
        if (cg != 0 && eo == 0u && wid == 7) {
            u64* const aml = (u64*)(ws + WS_AML);
            if (l1 < 2) __hip_atomic_fetch_max(aml + l1, ((u64)call << 32) | misc[10 + l1], RLX_AGENT);
            WAIT_V(0);
            unsigned tk = 0;
            if (l1 == 0) tk = __hip_atomic_fetch_add((unsigned*)(ws + WS_AML + 128), 1u, RLX_AGENT);
            if (__builtin_amdgcn_readfirstlane(tk) % 192u == 191u && l1 < 16)
                __hip_atomic_store((u64*)(lws + WS_AM + (call & 1u) * 128) + rank * 2 + (l1 >> 3), __hip_atomic_load(aml + (l1 >> 3), RLX_AGENT), RLX_SYS);
        }
    }
    DERIVE;
#pragma unroll
    for (int a = 0; a < 2; ++a)
#pragma unroll
        for (int c = 0; c < 2; ++c)
#pragma unroll
            for (int m = 0; m < 4; ++m)
#pragma unroll
                for (int n = 0; n < 2; ++n) acc[a][c][m][n] = (f32x4){0.f, 0.f, 0.f, 0.f};

    const __amdgpu_buffer_rsrc_t rA = make_rsrc(ws + stage_off + (size_t)pm * PANEL, PANEL);
    const __amdgpu_buffer_rsrc_t rB = make_rsrc(ws + stage_off + (size_t)(16 + pn) * PANEL, PANEL);
    const __amdgpu_buffer_rsrc_t rC = make_rsrc(ws, WS_CTL_BYTES);
    const unsigned snap_src = wid == 0 ? eo + WS_FX + pm * 256 : wid == 1 ? eo + WS_FW + pn * 256 : wid == 2 ? eo + WS_CTRL + b * 256
                            : wid == 3 ? WS_GLOB : wid == 4 ? WS_AM + (call & 1u) * 128 : OOB;

#define SA(bb, h) (((bb) * 2 + (h)) * HTB)
#define SB(bb, h) ((4 + (bb) * 2 + (h)) * HTB)
#define STAGE(bufoff, rs, so) do { _Pragma("unroll") for (int _i = 0; _i < 2; ++_i) \
        __builtin_amdgcn_raw_ptr_buffer_load_lds(rs, (LAS void*)(lds + (bufoff) + ldsw + _i * 8192), 16, voff, (so) + _i * 8192, 0, AUX_SYS); } while (0)
#define SNAP() __builtin_amdgcn_raw_ptr_buffer_load_lds(rC, (LAS void*)(lds + LDS_SNAP + wid * 256), 4, lane * 4, snap_src, 0, AUX_SYS)
#define LDA(dst, bb, h) do { _Pragma("unroll") for (int m = 0; m < 4; ++m) _Pragma("unroll") for (int k = 0; k < 2; ++k) \
        dst[m][k] = *(const LAS half8*)(lds + SA(bb, h) + (k ? aoff1 : aoff0) + m * 2048); } while (0)
#define LDB(dst, bb, h) do { _Pragma("unroll") for (int n = 0; n < 2; ++n) _Pragma("unroll") for (int k = 0; k < 2; ++k) \
        dst[n][k] = *(const LAS half8*)(lds + SA(bb, h) + boff + n * 1024 + k * 8192); } while (0)
#define MMA(ai, bj, At, Bt) do { __builtin_amdgcn_s_setprio(1); \
        _Pragma("unroll") for (int m = 0; m < 4; ++m) _Pragma("unroll") for (int n = 0; n < 2; ++n) _Pragma("unroll") for (int k = 0; k < 2; ++k) \
        acc[ai][bj][m][n] = __builtin_amdgcn_mfma_f32_16x16x32_f16(At[m][k], Bt[n][k], acc[ai][bj][m][n], 0, 0, 0); \
        __builtin_amdgcn_s_setprio(0); } while (0)
#define SNAP_ADDR(tt) (LDS_SNAP + (lane < 4 ? (lane >> 1) * 256 + (((tt) + (lane & 1)) & 63) * 4 : lane == 4 ? 512 : 768))

    int t = 0;
    state = (v == rank || grp < FGEN) ? 2 : grp == FGEN ? 1 : 0;
    do {
        if (state != 1) {
            const u64 t0 = __builtin_amdgcn_s_memrealtime();
            unsigned bits;
            for (;;) {
                SNAP(); WAIT_V(0); BAR; CFENCE;
                const unsigned sv = *(const LAS unsigned*)(lds + SNAP_ADDR(t));
                bits = (unsigned)__ballot(lane == 5 ? sv == call : sv >= call) & (state == 0 ? 63u : 47u);
                if (bits >= 16u || (t < 64 && bits == 15u)) break;
                if (tid == 0 && __builtin_amdgcn_s_memrealtime() - t0 > DEADLINE) __hip_atomic_store(tmo, call, RLX_AGENT);
                __builtin_amdgcn_s_sleep(6);
                CFENCE; BAR;
            }
            if (bits & 16u) state = 1;
            if (bits & 32u) { state = 3; t = 64; }
        }
        const bool run = state != 1 && t < 64;
        if (state == 1) {
            state = 2;
            int tz = tid; asm volatile("" : "+v"(tz));
            const int ln = tz & 63;
            const int quota = grp < 3 ? QUOTA : 1000;
            const int srel = (v - rank) & 7;
            const unsigned vo = wid * 4096 + ln * 16;
            unsigned* const qv = (unsigned*)((char*)queue + eo) + (tz >> 10);
            const unsigned* const rdy = (const unsigned*)(ws + eo + WS_RDY) + v * 256;
            int id0 = 8 + xcd, id1 = 16 + xcd;
            if (grp != FGEN) {
                BAR;
                if (tz == 0) misc[1] = 24u + __hip_atomic_fetch_add(qv, 2u, RLX_AGENT);
                WAIT_L(0); BAR; CFENCE;
                id0 = __builtin_amdgcn_readfirstlane(misc[1]); id1 = id0 + 1;
            }
            auto code_of = [&](int id) { return (int)((p.order[(id >> 2) & 63] >> ((id & 3) * 8)) & 255u); };
            auto src_of = [&](int code) {
                return make_rsrc(pws + stage_off + ((code >> 7) ? SRCW + (size_t)(rank * 128 + (code & 127)) * PIECE
                                                                : (size_t)(code >> 3) * PANEL + (size_t)((code & 7) * 8) * PIECE), PIECE);
            };
            auto wait_ready = [&](int code, unsigned seen) {
                if (__builtin_amdgcn_readfirstlane(seen) >= call) return;
                const u64 t0 = __builtin_amdgcn_s_memrealtime();
                while (__builtin_amdgcn_readfirstlane(__hip_atomic_load(rdy + code, RLX_SYS)) < call) {
                    if (__builtin_amdgcn_readfirstlane(__hip_atomic_load(tmo, RLX_AGENT)) == call
                        || __builtin_amdgcn_s_memrealtime() - t0 > DEADLINE) {
                        if (ln == 0) __hip_atomic_store(tmo, call, RLX_AGENT);
                        break;
                    }
                    __builtin_amdgcn_s_sleep(2);
                }
            };
            auto publish = [&](int code) {
                __hip_atomic_store((unsigned*)(ws + eo + WS_FX) + (code >> 3) * 64 + (code & 7) * 8 + srel, call, RLX_AGENT);
            };
            unsigned* const next_ctl = (unsigned*)(ws + eo + WS_CTRL + ((b + 64) & 255) * 256);
            unsigned* const next_go = next_ctl + (grp == 3 ? 1 : 2);
            bool passed = false;
            if (id0 < 256) {
                u32x4 d0, d1, d2, d3;
#define FLOAD(d, rs, i) d = __builtin_amdgcn_raw_buffer_load_b128(rs, vo + (i) * 1024, 0, AUX_SYS)
#define FSTORE(d, i) __builtin_amdgcn_raw_buffer_store_b128(d, rD, vo + (i) * 1024, 0, AUX_SYS)
                int c0 = code_of(id0), cp = -1;
                wait_ready(c0, 0u);
                if (grp != FGEN) {
                    const unsigned* go = (const unsigned*)(ws + eo + WS_CTRL + b * 256) + 2;
                    const u64 t0 = __builtin_amdgcn_s_memrealtime();
                    while (__builtin_amdgcn_readfirstlane(__hip_atomic_load(go, RLX_AGENT)) < call) {
                        if (__builtin_amdgcn_readfirstlane(__hip_atomic_load(tmo, RLX_AGENT)) == call
                            || __builtin_amdgcn_s_memrealtime() - t0 > DEADLINE) {
                            if (ln == 0) __hip_atomic_store(tmo, call, RLX_AGENT);
                            break;
                        }
                        __builtin_amdgcn_s_sleep(2);
                    }
                }
                __amdgpu_buffer_rsrc_t rS = src_of(c0);
                FLOAD(d0, rS, 0);
                WAIT_VC(0);
                for (int n = 0;; ++n) {
                    const bool more = id1 < 256;
                    const bool fetch = more && n + 2 < quota;
                    const bool pass = !passed && grp < 3 && (!more || n + PASS_AHEAD >= quota);
                    const int c1 = code_of(id1);
                    const __amdgpu_buffer_rsrc_t rD = make_rsrc(ws + stage_off + (size_t)(c0 >> 3) * PANEL + (size_t)((c0 & 7) * 8 + srel) * PIECE, PIECE);
                    WAIT_VC(1);
                    const unsigned seen = __hip_atomic_load(rdy + c1, RLX_SYS);
                    unsigned got = 256u;
                    if (tz == 0 && fetch) got = __hip_atomic_fetch_add(qv, 1u, RLX_AGENT);
                    if (tz == 0 && pass) __hip_atomic_store(next_ctl, call, RLX_AGENT);
                    passed = passed || pass;
                    CFENCE;
                    FLOAD(d1, rS, 1); CFENCE; FSTORE(d0, 0);
                    WAIT_VC(1);
                    asm volatile("" : "+v"(got));
                    if (tz == 0) misc[2 + (n & 1)] = 24u + got;
                    if (cp >= 0 && ln == 0 && __hip_atomic_fetch_add(misc + 8 + (n & 1), 1u, __ATOMIC_RELAXED, __HIP_MEMORY_SCOPE_WORKGROUP) % 8u == 7u) publish(cp);
                    CFENCE;
                    FLOAD(d2, rS, 2); CFENCE; FSTORE(d1, 1);
                    WAIT_VC(1);
                    if (tz == 0 && !more) __hip_atomic_store(next_go, call, RLX_AGENT);
                    CFENCE;
                    FLOAD(d3, rS, 3); CFENCE; FSTORE(d2, 2);
                    WAIT_VC(1);
                    if (more) {
                        wait_ready(c1, seen);
                        rS = src_of(c1);
                        FLOAD(d0, rS, 0);
                    }
                    CFENCE;
                    FSTORE(d3, 3);
                    WAIT_L(0); BAR; CFENCE;
                    if (!more) break;
                    cp = c0; c0 = c1; id1 = __builtin_amdgcn_readfirstlane(misc[2 + (n & 1)]);
                }
                WAIT_VC(0); BAR; CFENCE;
                if (tz == 64) publish(c0);
            } else if (tz == 0) {
                if (grp < 3) __hip_atomic_store(next_ctl, call, RLX_AGENT);
                __hip_atomic_store(next_go, call, RLX_AGENT);
            }
        }
        if (run) {
            const unsigned voff = (unsigned)tid * 16u;
            const unsigned ldsw = (unsigned)wid * 1024u;
            const int aoff0 = (wr * 64 + fr) * 128 + (fq ^ (fr >> 1)) * 16, aoff1 = aoff0 ^ 64;
            int boff = SB(0, 0) + fq * 2048 + (wc * 16 + fr) * 16;
            asm volatile("" : "+v"(boff));
            half8 At[4][2], B0[2][2], B1[2][2];
            unsigned so = (unsigned)t * PIECE;
            STAGE(SB(0, 0), rB, so); STAGE(SA(0, 0), rA, so); STAGE(SB(0, 1), rB, so + HTB); STAGE(SA(0, 1), rA, so + HTB);
            if (wr == 1) BAR;
            WAIT_V(4); BAR;
            STAGE(SB(1, 0), rB, so + PIECE); STAGE(SA(1, 0), rA, so + PIECE); STAGE(SB(1, 1), rB, so + PIECE + HTB);
            WAIT_V(6); BAR;
            bool go;
            do {
                const unsigned sv = *(const LAS unsigned*)(lds + SNAP_ADDR(t + 2));
                SCHED;
                LDB(B0, 0, 0); SCHED; LDA(At, 0, 0); STAGE(SA(1, 1), rA, so + PIECE + HTB);
                WAIT_L(8); BAR; WAIT_L(0); MMA(0, 0, At, B0);
                const unsigned bits = (unsigned)__ballot(sv >= call);
                go = t + 2 < 64 && (bits & (state == 0 ? 31u : 15u)) == 15u;
                const unsigned s2 = go ? so + 2 * PIECE : OOB;
                BAR; SCHED;
                LDB(B1, 0, 1); STAGE(SB(0, 0), rB, s2);
                BAR; WAIT_L(0); MMA(0, 1, At, B1); BAR;
                LDA(At, 0, 1); STAGE(SA(0, 0), rA, s2);
                BAR; WAIT_L(0); MMA(1, 0, At, B0); BAR; SCHED;
                STAGE(SB(0, 1), rB, s2 + HTB);
                if (!go) WAIT_V(0);
                WAIT_V(6); BAR; MMA(1, 1, At, B1); BAR;
                SNAP();
                LDB(B0, 1, 0); SCHED; LDA(At, 1, 0); STAGE(SA(0, 1), rA, s2 + HTB);
                WAIT_L(8); BAR; WAIT_L(0); MMA(0, 0, At, B0); BAR; SCHED;
                LDB(B1, 1, 1); STAGE(SB(1, 0), rB, s2 + PIECE);
                BAR; WAIT_L(0); MMA(0, 1, At, B1); BAR;
                LDA(At, 1, 1); STAGE(SA(1, 0), rA, s2 + PIECE);
                BAR; WAIT_L(0); MMA(1, 0, At, B0); BAR; SCHED;
                STAGE(SB(1, 1), rB, s2 + PIECE + HTB);
                WAIT_V(6); BAR; MMA(1, 1, At, B1); BAR;
                t += 2; so += 2 * PIECE;
            } while (go);
            WAIT_V(0);
            if (wr == 0) BAR;
            BAR;
        }
    } while (t < 64 || state < 2);
    again = false;
    if (eo == 0u && state != 3) {
        if (wid == 0) {
            int lz = lane; asm volatile("" : "+v"(lz));
            u64 am = *(const LAS u64*)(lds + LDS_SNAP + 1024 + (lz & 15) * 8);
            const u64 t0 = __builtin_amdgcn_s_memrealtime();
            while (__ballot((unsigned)(am >> 32) < call) != 0ull) {
                if (__builtin_amdgcn_readfirstlane(__hip_atomic_load(tmo, RLX_AGENT)) == call
                    || __builtin_amdgcn_s_memrealtime() - t0 > DEADLINE) {
                    if (lz == 0) __hip_atomic_store(tmo, call, RLX_AGENT);
                    am = ((u64)call << 32) | 0x3f800000u;
                    break;
                }
                __builtin_amdgcn_s_sleep(4);
                am = __hip_atomic_load((const u64*)(ws + WS_AM + (call & 1u) * 128) + (lz & 15), RLX_SYS);
            }
            unsigned mx = (unsigned)am;
#pragma unroll
            for (int o = 8; o >= 2; o >>= 1) { const unsigned y = __shfl_xor(mx, o); mx = y > mx ? y : mx; }
            const int e = (int)(mx >> 23);
            int se = (mx == 0u || e == 255 || (e >= 119 && e < 142)) ? 127 : 127 + 137 - e;
            se = se > 253 ? 253 : se < 1 ? 1 : se;
            if (lz < 2) misc[4 + lz] = (unsigned)se << 23;
        }
        WAIT_L(0); BAR; CFENCE;
        sx = __uint_as_float(__builtin_amdgcn_readfirstlane(misc[4])); sw = __uint_as_float(__builtin_amdgcn_readfirstlane(misc[5]));
        if (sx != 1.f || sw != 1.f) { again = true; eo = WS_EPOCH; }
    }
    } while (again);
    if (eo != 0u) {
        const int et = 254 - (int)(__float_as_uint(sx) >> 23) - (int)(__float_as_uint(sw) >> 23), e1 = et / 2;
        const float ix = __uint_as_float((unsigned)(127 + e1) << 23), iw = __uint_as_float((unsigned)(127 + et - e1) << 23);
#pragma unroll
        for (int a = 0; a < 2; ++a)
#pragma unroll
            for (int c = 0; c < 2; ++c)
#pragma unroll
                for (int m = 0; m < 4; ++m)
#pragma unroll
                    for (int n = 0; n < 2; ++n) acc[a][c][m][n] = acc[a][c][m][n] * ix * iw;
    }

    {
        DERIVE;
        int tz = tid; asm volatile("" : "+v"(tz));
        const int ln = tz & 63, efr = ln & 15, efq = ln >> 4;
        const float qnan = __builtin_nanf("");
        const __amdgpu_buffer_rsrc_t rO = make_rsrc(p.out, (unsigned)MS * GN * 4u);
        const unsigned lds0 = (unsigned)(size_t)lds;
        const unsigned reg0 = (unsigned)wid * CHUNK;
        const bool bad = __builtin_amdgcn_readfirstlane(__hip_atomic_load(tmo, RLX_AGENT)) == call;
        if (dest == rank) {
            const unsigned base = (unsigned)(((pm & 1) * 256 + wr * 64 + efq * 4) * GN + rank * NB + pn * 256 + wc * 64 + efr * 4) * 4u;
#pragma unroll
            for (int ai = 0; ai < 2; ++ai)
#pragma unroll
                for (int m = 0; m < 4; ++m)
#pragma unroll
                    for (int r = 0; r < 4; ++r) {
                        f32x4 o = (f32x4){acc[ai][0][m][0][r], acc[ai][0][m][1][r], acc[ai][1][m][0][r], acc[ai][1][m][1][r]};
#pragma unroll
                        for (int e = 0; e < 4; ++e) o[e] = bad ? qnan : (o[e] < 0.f ? 0.f : o[e]);
                        __builtin_amdgcn_raw_buffer_store_b128(__builtin_bit_cast(u32x4, o), rO, base + (unsigned)((ai * 128 + m * 16 + r) * GN) * 4u, 0, AUX_SYS);
                    }
        } else {
            unsigned sizes = 0, ticket = 0;
            if (ln == 0) ticket = __hip_atomic_fetch_add((unsigned*)(ws + WS_Q + dest * 128 + 64) + (tz >> 10), 1u, RLX_AGENT);
#pragma unroll
            for (int ai = 0; ai < 2; ++ai) {
                unsigned mk0 = 0u, mk1 = 0u;
                unsigned pos = lds0 + reg0 + 512u;
#pragma unroll
                for (int m = 0; m < 4; ++m)
#pragma unroll
                    for (int r = 0; r < 4; ++r) {
                        if (m < 2) compact4(acc[ai][0][m][0][r], acc[ai][0][m][1][r], acc[ai][1][m][0][r], acc[ai][1][m][1][r], mk0, pos);
                        else compact4(acc[ai][0][m][0][r], acc[ai][0][m][1][r], acc[ai][1][m][0][r], acc[ai][1][m][1][r], mk1, pos);
                    }
                *(LAS u64*)(lds + reg0 + ln * 8) = ((u64)mk1 << 32) | mk0;
                WAIT_L(0);
                const unsigned bytes = pos - lds0 - reg0;
                sizes |= (bytes >> 2) << (13 * ai);
                const __amdgpu_buffer_rsrc_t rs = make_rsrc(ws + WS_OBOX + (size_t)(dest * 512 + (grp * 64 + tix * 8 + wid) * 2 + ai) * CHUNK, (bytes + 63u) & ~63u);
                for (unsigned o = 0; o < bytes; o += 1024)
                    __builtin_amdgcn_raw_buffer_store_b128(*(const LAS u32x4*)(lds + reg0 + ln * 16 + o), rs, ln * 16 + o, 0, AUX_SYS);
                WAIT_L(0);
            }
            WAIT_V(0);
            asm volatile("" : "+v"(ticket));
            if (ln == 0) __hip_atomic_store((u64*)(p.ws_all[dest] + WS_FO + (rank * 256 + (ticket & 255u)) * 64),
                                            ((u64)call << 34) | ((u64)(grp * 64 + tix * 8 + wid) << 26) | (bad ? 0u : sizes), RLX_SYS);
        }
        {
            if (grp == 0 && v != rank) {
                LAS unsigned* const req = misc + 16;
                LAS unsigned* const fin = misc + 24;
                if (wid == 0) {
                    {
                        const unsigned* tok = (const unsigned*)(ws + eo + WS_CTRL + b * 256) + 1;
                        const u64 t0 = __builtin_amdgcn_s_memrealtime();
                        while (__builtin_amdgcn_readfirstlane(__hip_atomic_load(tok, RLX_AGENT)) < call) {
                            if (__builtin_amdgcn_readfirstlane(__hip_atomic_load(tmo, RLX_AGENT)) == call
                                || __builtin_amdgcn_s_memrealtime() - t0 > DEADLINE) {
                                if (ln == 0) __hip_atomic_store(tmo, call, RLX_AGENT);
                                break;
                            }
                            __builtin_amdgcn_s_sleep(2);
                        }
                    }
                    const unsigned reqa = lds0 + LDS_MISC + 64, fina = reqa + 32;
                    constexpr unsigned ALL = (2u << NEXP) - 2u;
                    unsigned fw = 0, nf = 0, cnt = 0, last = 0;
                    u64 fe = 0, tl = __builtin_amdgcn_s_memrealtime();
                    for (;;) {
                        const unsigned r = lds_ld(reqa + (ln & 7) * 4);
                        const unsigned done = (unsigned)__ballot(r == 0xffffffffu) & ALL;
                        const unsigned want = (unsigned)__ballot(r != 0u && r != 0xffffffffu) & ALL;
                        if (want == 0u) {
                            if (nf) {
                                WAIT_V(0);
                                for (; nf; --nf, fw >>= 4) lds_st(fina + (fw & 15u) * 4, 1u);
                                fe = 0;
                            }
                            if (done == ALL || __builtin_amdgcn_s_memrealtime() - tl > 2 * DEADLINE) break;
                            __builtin_amdgcn_s_sleep(1);
                            continue;
                        }
                        const unsigned hi = want & (0xfffffffeu << last);
                        const unsigned w = __builtin_ctz(hi ? hi : want);
                        last = w;
                        const unsigned rq = __builtin_amdgcn_readlane(r, w);
                        lds_st(reqa + w * 4, 0u);
                        const unsigned bytes = rq & 0xffffu;
                        const __amdgpu_buffer_rsrc_t rs = make_rsrc(pws + WS_OBOX + (size_t)(rank * 512 + ((rq >> 16) & 511u)) * CHUNK, bytes);
                        for (unsigned o = 0; o < bytes; o += 1024) {
                            WAIT_VX(OWIN0);
                            while (nf && ((cnt - OWIN0 - (unsigned)fe) & 255u) < 128u) { lds_st(fina + (fw & 15u) * 4, 1u); fw >>= 4; fe >>= 8; --nf; }
                            __builtin_amdgcn_raw_ptr_buffer_load_lds(rs, (LAS void*)(lds + w * CHUNK + o), 16, ln * 16, o, 0, AUX_SYS);
                            ++cnt;
                        }
                        fw |= w << (4u * nf); fe |= (u64)(cnt & 255u) << (8u * nf); ++nf;
                    }
                } else if (wid <= NEXP) {
                    unsigned* const oq = (unsigned*)(queue + 1) + (tz >> 10);
                    unsigned i = (unsigned)((wid - 1) * 8 + xcd);
                    const u64 tw = __builtin_amdgcn_s_memrealtime();
                    while (i < 512u) {
                        unsigned nxt = 0;
                        if (ln == 0) nxt = __hip_atomic_fetch_add(oq, 1u, RLX_AGENT);
                        const unsigned ai = i & 1u;
                        const u64* fo = (const u64*)(ws + WS_FO + (v * 256 + (i >> 1)) * 64);
                        const u64 t0 = __builtin_amdgcn_s_memrealtime();
                        unsigned lo, hi;
                        for (;;) {
                            const u64 fv = __hip_atomic_load(fo, RLX_SYS);
                            lo = __builtin_amdgcn_readfirstlane((unsigned)fv); hi = __builtin_amdgcn_readfirstlane((unsigned)(fv >> 32));
                            if ((hi >> 2) >= (call & 0x3fffffffu)) break;
                            if (__builtin_amdgcn_readfirstlane(__hip_atomic_load(tmo, RLX_AGENT)) == call
                                || __builtin_amdgcn_s_memrealtime() - t0 > DEADLINE) {
                                if (ln == 0) __hip_atomic_store(tmo, call, RLX_AGENT);
                                lo = (i >> 1) << 26; hi = i >> 7;
                                break;
                            }
                            __builtin_amdgcn_s_sleep(4);
                        }
                        const unsigned id = (hi & 3u) << 6 | lo >> 26, g2 = id >> 6, x2 = (id >> 3) & 7u, w2 = id & 7u;
                        const unsigned bytes = ((lo >> (13u * ai)) & 0x1fffu) * 4u;
                        bool nodata = bytes > 512u + 16384u || bytes < 512u;
                        if (!nodata) {
                            if (ln == 0) __hip_atomic_store(req + wid, 0x80000000u | (id * 2u + ai) << 16 | bytes, __ATOMIC_RELAXED, __HIP_MEMORY_SCOPE_WORKGROUP);
                            while (__builtin_amdgcn_readfirstlane(__hip_atomic_load(fin + wid, __ATOMIC_RELAXED, __HIP_MEMORY_SCOPE_WORKGROUP)) == 0u) {
                                if (__builtin_amdgcn_s_memrealtime() - tw > 3 * DEADLINE) { nodata = true; break; }
                                __builtin_amdgcn_s_sleep(2);
                            }
                            if (ln == 0) fin[wid] = 0u;
                            CFENCE;
                        }
                        const u64 mm = *(const LAS u64*)(lds + reg0 + ln * 8);
                        unsigned mk = nodata ? 0u : (unsigned)mm;
                        unsigned pos = lds0 + reg0 + 512u;
                        unsigned off = (unsigned)(((x2 & 1u) * 256u + ai * 128u + (w2 >> 2) * 64u + efq * 4) * GN + v * NB + (4u * g2 + (x2 >> 1)) * 256u + (w2 & 3u) * 64u + efr * 4) * 4u;
                        const float fill = nodata ? qnan : 0.f;
                        for (int m = 0; m < 4; ++m) {
                            f32x4 o[4];
                            expand16(o, mk, pos, fill);
#pragma unroll
                            for (int r = 0; r < 4; ++r)
                                __builtin_amdgcn_raw_buffer_store_b128(__builtin_bit_cast(u32x4, o[r]), rO, off + (unsigned)(r * GN) * 4u, 0, AUX_SYS);
                            off += 16u * GN * 4u;
                            if (m == 1) mk = nodata ? 0u : (unsigned)(mm >> 32);
                        }
                        i = 8u * NEXP + __builtin_amdgcn_readfirstlane(nxt);
                    }
                    if (ln == 0) __hip_atomic_store(req + wid, 0xffffffffu, __ATOMIC_RELAXED, __HIP_MEMORY_SCOPE_WORKGROUP);
                }
            }
        }
    }
}

}

extern "C" void kernel_launch(int rank, int nranks, ncclComm_t comm,
                              hipStream_t stream,
                              void* const* d_in, const int* in_sizes, int n_in,
                              void* d_out, int out_size,
                              const int* in_dtypes, int out_dtype,
                              void* const* d_out_all,
                              void* const* d_ws_all, size_t ws_bytes) {
    if (nranks != NR || n_in < 2 || (unsigned)rank >= (unsigned)NR || ws_bytes < WS_NEED ||
        in_sizes[0] != GM * KS || in_sizes[1] != KS * GN || out_size != MS * GN) {
        fprintf(stderr, "kernel_launch: unexpected problem shape\n");
        return;
    }
    Params p;
    p.x = (const float*)d_in[0];
    p.w = (const float*)d_in[1];
    p.out = (float*)d_out;
    for (int r = 0; r < NR; ++r) p.ws_all[r] = (char*)d_ws_all[r];
    make_order(p.order, p.worder);
    p.rank = rank; p.pad = 0; p.ws = (char*)d_ws_all[rank];
    hipLaunchKernelGGL(fused_gemm_rs, dim3(NWG), dim3(512), 0, stream, p);
    HIP_CHECK(hipGetLastError());
}
```
